```python
import numpy as np

kernel_name = "gpu_a2a_gemm_m8192_k8192_n8192_f32_silu"


ABSMAX_RTOL = 0.001

COMPUTE_DTYPE = "float32"
EPILOGUE = "silu"
IN_QUANT = None
OUT_QUANT = None
IN_DTYPES = {"x": "float32", "w_mat": "float32"}
WORLD_SIZE = 8

def setup_inputs():

    x = np.random.randn(8192, 8192).astype(np.float32)
    w_mat = (np.random.randn(8192, 8192) / np.sqrt(8192)).astype(np.float32)
    return {"x": x, "w_mat": w_mat}

def _epilogue(y):
    return (y / (1.0 + np.exp(-np.clip(y, -60.0, 60.0)))).astype(np.float32)

def reference(x, w_mat):
    y = (x @ w_mat).astype(np.float32)
    return _epilogue(y)
```

```cpp
#include <hip/hip_runtime.h>
#include <rccl/rccl.h>
#include <cstdio>
#include <cstdint>

#define HIP_CHECK(expr) do { hipError_t _e = (expr); if (_e != hipSuccess) { \
    fprintf(stderr, "HIP error %s at %s:%d\n", hipGetErrorString(_e), __FILE__, __LINE__); \
    return; } } while (0)

typedef _Float16 f16x8 __attribute__((ext_vector_type(8)));
typedef float f32x4 __attribute__((ext_vector_type(4)));
typedef unsigned u32x4 __attribute__((ext_vector_type(4)));

constexpr int NR = 8;
constexpr int MG = 8192, KG = 8192, NG = 8192;
constexpr int ML = MG / NR;
constexpr int KL = KG / NR;
constexpr size_t X16_BYTES = (size_t)ML * KG * 2;
constexpr size_t WS_CTRL = 1 << 20;
constexpr size_t WS_X16 = WS_CTRL;
constexpr size_t WS_NEED = WS_X16 + 2 * X16_BYTES;
constexpr unsigned long long DEADLINE_TICKS = 100000000ull;

constexpr int CW_TICKET0 = 0, CW_TICKET1 = 1, CW_TMO = 2, CW_FLAG = 8;

struct Ptrs { char* ws[NR]; };

__global__ void __launch_bounds__(256, 1)
fused(int phase, int rank, const float* __restrict__ x, const float* __restrict__ w, float* __restrict__ out, Ptrs P) {
  __shared__ __attribute__((aligned(16))) char smem[2 * 8192 + 16];
  char* myws = P.ws[rank];
  unsigned* ctrl = (unsigned*)myws;
  const int tid = threadIdx.x, lane = tid & 63, wid = tid >> 6;
  unsigned* scall = (unsigned*)(smem + 2 * 8192);
  if (tid == 0) {
    unsigned t = __hip_atomic_fetch_add(ctrl + 32 * (phase == 0 ? CW_TICKET0 : CW_TICKET1), 1u, __ATOMIC_RELAXED, __HIP_MEMORY_SCOPE_AGENT);
    scall[0] = t / gridDim.x + 1;
  }
  __syncthreads();
  const unsigned call = scall[0];
  const size_t par = (size_t)(call & 1) * X16_BYTES;
  if (phase == 0) {
    const int d = blockIdx.x & 7;
    const int sub = blockIdx.x >> 3;
    char* dst = P.ws[d] + WS_X16 + par;
    for (int i = 0; i < 16; ++i) {
      const int u = sub * 64 + wid * 16 + i;
      const int m16g = u >> 5, ks = u & 31;
      const int row = d * ML + m16g * 16 + (lane & 15);
      const int k = ks * 32 + (lane >> 4) * 8;
      const f32x4 a = *(const f32x4*)(x + (size_t)row * KL + k);
      const f32x4 b = *(const f32x4*)(x + (size_t)row * KL + k + 4);
      f16x8 h;
      h[0] = (_Float16)a[0]; h[1] = (_Float16)a[1]; h[2] = (_Float16)a[2]; h[3] = (_Float16)a[3];
      h[4] = (_Float16)b[0]; h[5] = (_Float16)b[1]; h[6] = (_Float16)b[2]; h[7] = (_Float16)b[3];
      const int mt = m16g >> 4, m16 = m16g & 15, kg = rank * 32 + ks;
      const size_t off = ((size_t)((mt * 256 + kg) * 16 + m16)) * 1024 + lane * 16;
      *(f16x8*)(dst + off) = h;
    }
    asm volatile("s_waitcnt vmcnt(0)" ::: "memory");
    __syncthreads();
    if (tid == 0) {
      __builtin_amdgcn_fence(__ATOMIC_RELEASE, "");
      asm volatile("s_waitcnt vmcnt(0)" ::: "memory");
      __hip_atomic_fetch_add((unsigned*)P.ws[d] + 32 * (CW_FLAG + rank), 1u, __ATOMIC_RELAXED, __HIP_MEMORY_SCOPE_SYSTEM);
    }
    return;
  }
  unsigned* tmo = ctrl + 32 * CW_TMO;
  if (tid == 0) {
    const unsigned long long t0 = __builtin_amdgcn_s_memrealtime();
    for (int p = 0; p < NR; ++p) {
      if (p == rank) continue;
      while (__hip_atomic_load(ctrl + 32 * (CW_FLAG + p), __ATOMIC_RELAXED, __HIP_MEMORY_SCOPE_SYSTEM) < call * 32u) {
        if (__hip_atomic_load(tmo, __ATOMIC_RELAXED, __HIP_MEMORY_SCOPE_AGENT) == call ||
            __builtin_amdgcn_s_memrealtime() - t0 > DEADLINE_TICKS) {
          __hip_atomic_store(tmo, call, __ATOMIC_RELAXED, __HIP_MEMORY_SCOPE_AGENT);
          break;
        }
        __builtin_amdgcn_s_sleep(2);
      }
    }
    __builtin_amdgcn_fence(__ATOMIC_ACQUIRE, "");
    asm volatile("s_waitcnt vmcnt(0)" ::: "memory");
    scall[1] = (__hip_atomic_load(tmo, __ATOMIC_RELAXED, __HIP_MEMORY_SCOPE_AGENT) == call) ? 1u : 0u;
  }
  __syncthreads();
  const bool bad = scall[1] != 0;
  const int xcd = blockIdx.x & 7, idx = blockIdx.x >> 3;
  const int pn = xcd * 4 + (idx & 3), pm = idx >> 2;
  const char* x16 = myws + WS_X16 + par;
  const int mt = pm >> 1, m16base = (pm & 1) * 8;
  f32x4 acc[8][4];
#pragma unroll
  for (int m = 0; m < 8; ++m)
#pragma unroll
    for (int j = 0; j < 4; ++j) acc[m][j] = (f32x4){0.f, 0.f, 0.f, 0.f};
  const int g = lane >> 4, c = lane & 15;
  const float* wl = w + (size_t)(8 * g) * NG + pn * 256 + wid * 64 + 4 * c;
  if (!bad) {
    u32x4 xr[2];
    f32x4 wr[8];
    {
      const char* src = x16 + ((size_t)((mt * 256 + 0) * 16 + m16base)) * 1024;
      xr[0] = *(const u32x4*)(src + tid * 16);
      xr[1] = *(const u32x4*)(src + 4096 + tid * 16);
#pragma unroll
      for (int i = 0; i < 8; ++i) wr[i] = *(const f32x4*)(wl + (size_t)i * NG);
      *(u32x4*)(smem + tid * 16) = xr[0];
      *(u32x4*)(smem + 4096 + tid * 16) = xr[1];
    }
    __syncthreads();
    for (int ks = 0; ks < 256; ++ks) {
      const int cur = ks & 1;
      f16x8 bf[4];
#pragma unroll
      for (int j = 0; j < 4; ++j)
#pragma unroll
        for (int i = 0; i < 8; ++i) bf[j][i] = (_Float16)wr[i][j];
      if (ks + 1 < 256) {
        const char* src = x16 + ((size_t)((mt * 256 + ks + 1) * 16 + m16base)) * 1024;
        xr[0] = *(const u32x4*)(src + tid * 16);
        xr[1] = *(const u32x4*)(src + 4096 + tid * 16);
#pragma unroll
        for (int i = 0; i < 8; ++i) wr[i] = *(const f32x4*)(wl + (size_t)((ks + 1) * 32 + i) * NG);
      }
#pragma unroll
      for (int m = 0; m < 8; ++m) {
        const f16x8 af = *(const f16x8*)(smem + cur * 8192 + m * 1024 + lane * 16);
#pragma unroll
        for (int j = 0; j < 4; ++j) acc[m][j] = __builtin_amdgcn_mfma_f32_16x16x32_f16(af, bf[j], acc[m][j], 0, 0, 0);
      }
      if (ks + 1 < 256) {
        *(u32x4*)(smem + (cur ^ 1) * 8192 + tid * 16) = xr[0];
        *(u32x4*)(smem + (cur ^ 1) * 8192 + 4096 + tid * 16) = xr[1];
      }
      __syncthreads();
    }
  }
  const float qnan = __builtin_nanf("");
#pragma unroll
  for (int m = 0; m < 8; ++m)
#pragma unroll
    for (int r = 0; r < 4; ++r) {
      f32x4 o;
#pragma unroll
      for (int j = 0; j < 4; ++j) {
        const float y = acc[m][j][r];
        const float yc = fminf(fmaxf(y, -60.f), 60.f);
        o[j] = bad ? qnan : y / (1.0f + __expf(-yc));
      }
      const int row = pm * 128 + m * 16 + 4 * g + r;
      *(f32x4*)(out + (size_t)row * NG + pn * 256 + wid * 64 + 4 * c) = o;
    }
}

extern "C" void kernel_launch(int rank, int nranks, ncclComm_t comm, hipStream_t stream,
                              void* const* d_in, const int* in_sizes, int n_in,
                              void* d_out, int out_size, const int* in_dtypes, int out_dtype,
                              void* const* d_out_all, void* const* d_ws_all, size_t ws_bytes) {
  if (nranks != NR || (unsigned)rank >= (unsigned)NR || ws_bytes < WS_NEED || n_in < 2 ||
      in_sizes[0] != MG * KL || in_sizes[1] != KG * NG || out_size != ML * NG) {
    fprintf(stderr, "unexpected problem shape\n");
    return;
  }
  Ptrs P;
  for (int p = 0; p < NR; ++p) P.ws[p] = (char*)d_ws_all[p];
  const float* x = (const float*)d_in[0];
  const float* w = (const float*)d_in[1];
  fused<<<256, 256, 0, stream>>>(0, rank, x, w, (float*)d_out, P);
  fused<<<256, 256, 0, stream>>>(1, rank, x, w, (float*)d_out, P);
  HIP_CHECK(hipGetLastError());
}
```

```cpp
#include <hip/hip_runtime.h>
#include <rccl/rccl.h>
#include <cstdio>
#include <cstdint>

#define HIP_CHECK(expr) do { hipError_t _e = (expr); if (_e != hipSuccess) { \
    fprintf(stderr, "HIP error %s at %s:%d\n", hipGetErrorString(_e), __FILE__, __LINE__); \
    return; } } while (0)

typedef unsigned u32x4 __attribute__((ext_vector_type(4)));

constexpr int NR = 8;
constexpr int MG = 8192, KG = 8192, NG = 8192;
constexpr int ML = MG / NR, KL = KG / NR;
constexpr unsigned WS_TMO = 256;
constexpr unsigned WS_CTR = 4096;
constexpr unsigned WS_XFLG = 65536;
constexpr unsigned WS_PFLG = 131072;
constexpr unsigned WS_CTRL_BYTES = 1u << 20;
constexpr unsigned WS_X16 = 1u << 20;
constexpr unsigned X16_BYTES = (unsigned)ML * KG * 2;
constexpr unsigned WS_XCH = WS_X16 + 2 * X16_BYTES;
constexpr unsigned WS_NEED = WS_XCH + 256u * 131072u;
constexpr unsigned OOB = 0x40000000u;
constexpr int DAB = 2, DBC = 2;

struct Ptrs { char* ws[NR]; };

#define VCLOB "v16","v17","v18","v19","v20","v21","v22","v23","v24","v25","v26","v27","v28","v29","v30","v31","v32","v33","v34","v35","v36","v37","v38","v39","v40","v41","v42","v43","v44","v45","v46","v47","v48","v49","v50","v51","v52","v53","v54","v55","v56","v57","v58","v59","v60","v61","v62","v63","v64","v65","v66","v67","v68","v69","v70","v71","v72","v73","v74","v75","v76","v77","v78","v79","v80","v81","v82","v83","v84","v85","v86","v87","v88","v89","v90","v91","v92","v93","v94","v95","v96","v97","v98","v99","v100","v101","v102","v103","v104","v105","v106","v107","v108","v109","v110","v111","v112","v113","v114","v115","v116","v117","v118","v119","v120","v121","v122","v123","v124","v125","v126","v127","v128","v129","v130","v131","v132","v133","v134","v135","v136","v137","v138","v139","v140","v141","v142","v143","v144","v145","v146","v147","v148","v149","v150","v151","v152","v153","v154","v155","v156","v157","v158","v159","v160","v161","v162","v163","v164","v165","v166","v167","v168","v169","v170","v171","v172","v173","v174","v175","v176","v177","v178","v179","v180","v181","v182","v183","v184","v185","v186","v187","v188","v189","v190","v191","v192","v193","v194","v195","v196","v197","v198","v199","v200","v201","v202","v203","v204","v205","v206","v207","v208","v209","v210","v211","v212","v213","v214","v215","v216","v217","v218","v219","v220","v221","v222","v223","v224","v225","v226","v227","v228","v229","v230","v231","v232","v233","v234","v235","v236","v237","v238","v239","v240","v241","v242","v243","v244","v245","v246","v247","v248","v249","v250","v251","v252","v253","v254","v255"
#define VCLOB8 "v8","v9","v10","v11","v12","v13","v14","v15","v16","v17","v18","v19","v20","v21","v22","v23","v24","v25","v26","v27","v28","v29","v30","v31","v32","v33","v34","v35","v36","v37","v38","v39","v40","v41","v42","v43","v44","v45","v46","v47","v48","v49","v50","v51","v52","v53","v54","v55","v56","v57","v58","v59","v60","v61","v62","v63","v64","v65","v66","v67","v68","v69","v70","v71","v72","v73","v74","v75","v76","v77","v78","v79","v80","v81","v82","v83","v84","v85","v86","v87","v88","v89","v90","v91","v92","v93","v94","v95","v96","v97","v98","v99","v100","v101","v102","v103","v104","v105","v106","v107","v108","v109","v110","v111","v112","v113","v114","v115","v116","v117","v118","v119","v120","v121","v122","v123","v124","v125","v126","v127","v128","v129","v130","v131","v132","v133","v134","v135","v136","v137","v138","v139","v140","v141","v142","v143","v144","v145","v146","v147","v148","v149","v150","v151","v152","v153","v154","v155","v156","v157","v158","v159","v160","v161","v162","v163","v164","v165","v166","v167","v168","v169","v170","v171","v172","v173","v174","v175","v176","v177","v178","v179","v180","v181","v182","v183","v184","v185","v186","v187","v188","v189","v190","v191","v192","v193","v194","v195","v196","v197","v198","v199","v200","v201","v202","v203","v204","v205","v206","v207","v208","v209","v210","v211","v212","v213","v214","v215","v216","v217","v218","v219","v220","v221","v222","v223","v224","v225","v226","v227","v228","v229","v230","v231","v232","v233","v234","v235","v236","v237","v238","v239","v240","v241","v242","v243","v244","v245","v246","v247","v248","v249","v250","v251","v252","v253","v254","v255"
#define ACLOB "a0","a1","a2","a3","a4","a5","a6","a7","a8","a9","a10","a11","a12","a13","a14","a15","a16","a17","a18","a19","a20","a21","a22","a23","a24","a25","a26","a27","a28","a29","a30","a31","a32","a33","a34","a35","a36","a37","a38","a39","a40","a41","a42","a43","a44","a45","a46","a47","a48","a49","a50","a51","a52","a53","a54","a55","a56","a57","a58","a59","a60","a61","a62","a63","a64","a65","a66","a67","a68","a69","a70","a71","a72","a73","a74","a75","a76","a77","a78","a79","a80","a81","a82","a83","a84","a85","a86","a87","a88","a89","a90","a91","a92","a93","a94","a95","a96","a97","a98","a99","a100","a101","a102","a103","a104","a105","a106","a107","a108","a109","a110","a111","a112","a113","a114","a115","a116","a117","a118","a119","a120","a121","a122","a123","a124","a125","a126","a127","a128","a129","a130","a131","a132","a133","a134","a135","a136","a137","a138","a139","a140","a141","a142","a143","a144","a145","a146","a147","a148","a149","a150","a151","a152","a153","a154","a155","a156","a157","a158","a159","a160","a161","a162","a163","a164","a165","a166","a167","a168","a169","a170","a171","a172","a173","a174","a175","a176","a177","a178","a179","a180","a181","a182","a183","a184","a185","a186","a187","a188","a189","a190","a191","a192","a193","a194","a195","a196","a197","a198","a199","a200","a201","a202","a203","a204","a205","a206","a207","a208","a209","a210","a211","a212","a213","a214","a215","a216","a217","a218","a219","a220","a221","a222","a223","a224","a225","a226","a227","a228","a229","a230","a231","a232","a233","a234","a235","a236","a237","a238","a239","a240","a241","a242","a243","a244","a245","a246","a247","a248","a249","a250","a251","a252","a253","a254","a255"
#define SCLOB "s40","s41","s42","s43","s44","s45","s46","s47","s48","s49","s50","s51","s52","s53","s54","s55","s56","s57","s58","s59","s60","s61","s62","s63","s64","s65","s66","s67","s68","s69","s70","s71","s72","s73","s74","s75","s76","s77","s78","s79","s80","s81","s82","s83","s84","s85","s86","s87","s88","s89","s90","s91","s92","s93","s94","s95"

__device__ __forceinline__ u32x4 mksrd(const void* p, unsigned bytes) {
  const uint64_t a = (uint64_t)p;
  u32x4 r;
  r[0] = __builtin_amdgcn_readfirstlane((unsigned)a);
  r[1] = __builtin_amdgcn_readfirstlane((unsigned)(a >> 32)) & 0xffffu;
  r[2] = __builtin_amdgcn_readfirstlane(bytes);
  r[3] = 0x00020000u;
  return r;
}
__device__ __forceinline__ int swz(int rq) { return (((rq & 1) << 1) | (rq >> 1)) ^ ((rq >> 1) << 1); }
__device__ __forceinline__ int step_b(int b, int s) { return (((256 * s + b) / 6) >> 2) - 3; }

__global__ void __launch_bounds__(256, 1)
fused(int rank, const float* __restrict__ x, const float* __restrict__ w, float* __restrict__ out, char* myws, Ptrs P
      ) {
  __shared__ __attribute__((aligned(1024))) char smem[131072 + 1024];
  const int tid = threadIdx.x, lane = tid & 63;
  const int wid = __builtin_amdgcn_readfirstlane(tid >> 6);
  const int b = blockIdx.x, j8 = b & 7, idx = b >> 3;
  const int h = j8 & 1, pn = (j8 >> 1) * 8 + (idx & 7), mt = idx >> 3, partner = b ^ 1;
  unsigned call;
  const int g = lane >> 4, c16 = lane & 15;
  const u32x4 srdx = mksrd(x, (unsigned)MG * KL * 4u);
  const u32x4 srdC = mksrd(myws, WS_CTRL_BYTES);
  const unsigned wcol = (unsigned)(pn * 256 + wid * 64) * 4u;
  const u32x4 srdW = mksrd((const char*)w + wcol, 0x10000000u - wcol);
  const unsigned sctr = WS_CTR + (b * 4 + wid) * 4, sk0 = rank * 32 + h * 16;
  {
    const int pr = lane >> 2, pg = (lane & 3) ^ swz(pr >> 2);
    const unsigned vxsrc = pr * 4096 + pg * 16, vwoff = (4 * g) * 32768 + 16 * c16;
    const int lu = b & 31, lhd = lu >> 4, lmt = (lu >> 2) & 3, lm16 = (lu & 3) * 4 + wid, lit = lhd * 8 + (b >> 5);
    const unsigned slsrc = (unsigned)(((rank * ML + lmt * 256 + lm16 * 16) * KL + lit * 64) * 4), slmt = lmt, swrot = mt;
    const unsigned sldst = WS_X16 + (unsigned)(((lmt * 256 + rank * 32 + 2 * lit) * 16 + lm16) * 1024), vl16 = lane * 16;
    const u32x4 srdWS = mksrd(myws, WS_XCH);
    asm volatile(
"buffer_load_dword v52, off, %[srdC], %[sctr] sc1\n\t"
"v_mov_b32 v40, %[vxsrc]\n\t"
"s_add_u32 s89, %[slmt], 0\n\t"
"s_and_b32 s89, s89, 3\n\t"
"s_lshl_b32 s89, s89, 6\n\t"
"s_add_u32 s89, s89, %[slsrc]\n\t"
"buffer_load_dwordx4 v[64:67], v40, %[srdx], s89 offen\n\t"
"s_add_u32 s89, %[slmt], 1\n\t"
"s_and_b32 s89, s89, 3\n\t"
"s_lshl_b32 s89, s89, 6\n\t"
"s_add_u32 s89, s89, %[slsrc]\n\t"
"buffer_load_dwordx4 v[68:71], v40, %[srdx], s89 offen\n\t"
"s_add_u32 s89, %[slmt], 2\n\t"
"s_and_b32 s89, s89, 3\n\t"
"s_lshl_b32 s89, s89, 6\n\t"
"s_add_u32 s89, s89, %[slsrc]\n\t"
"buffer_load_dwordx4 v[72:75], v40, %[srdx], s89 offen\n\t"
"s_add_u32 s89, %[slmt], 3\n\t"
"s_and_b32 s89, s89, 3\n\t"
"s_lshl_b32 s89, s89, 6\n\t"
"s_add_u32 s89, s89, %[slsrc]\n\t"
"buffer_load_dwordx4 v[76:79], v40, %[srdx], s89 offen\n\t"
"v_add_u32 v60, 0, %[vwoff]\n\t"
"v_add_u32 v61, 32768, %[vwoff]\n\t"
"v_add_u32 v62, 65536, %[vwoff]\n\t"
"v_add_u32 v63, 98304, %[vwoff]\n\t"
"s_add_u32 s49, %[swrot], 0\n\t"
"s_and_b32 s49, s49, 3\n\t"
"s_lshl_b32 s50, s49, 19\n\t"
"s_add_u32 s49, %[swrot], 1\n\t"
"s_and_b32 s49, s49, 3\n\t"
"s_lshl_b32 s51, s49, 19\n\t"
"s_add_u32 s49, %[swrot], 2\n\t"
"s_and_b32 s49, s49, 3\n\t"
"s_lshl_b32 s52, s49, 19\n\t"
"s_add_u32 s49, %[swrot], 3\n\t"
"s_and_b32 s49, s49, 3\n\t"
"s_lshl_b32 s53, s49, 19\n\t"
"s_lshl_b32 s63, %[sk0], 20\n\t"
"s_lshl_b32 s64, %[sk0], 14\n\t"
"s_add_u32 s89, s63, 0\n\t"
"s_add_u32 s40, s89, s50\n\t"
"buffer_load_dwordx4 v[128:131], v60, %[srdW], s40 offen\n\t"
"buffer_load_dwordx4 v[132:135], v61, %[srdW], s40 offen\n\t"
"buffer_load_dwordx4 v[136:139], v62, %[srdW], s40 offen\n\t"
"buffer_load_dwordx4 v[140:143], v63, %[srdW], s40 offen\n\t"
"s_add_u32 s89, s63, 0\n\t"
"s_add_u32 s41, s89, s51\n\t"
"buffer_load_dwordx4 v[144:147], v60, %[srdW], s41 offen\n\t"
"buffer_load_dwordx4 v[148:151], v61, %[srdW], s41 offen\n\t"
"buffer_load_dwordx4 v[152:155], v62, %[srdW], s41 offen\n\t"
"buffer_load_dwordx4 v[156:159], v63, %[srdW], s41 offen\n\t"
"v_mov_b32 v41, %[vl16]\n\t"
"s_waitcnt vmcnt(12)\n\t"
"v_add_u32 v52, 1, v52\n\t"
"v_mov_b32 v42, 4\n\t"
"v_readfirstlane_b32 s94, v52\n\t"
"s_and_b32 s83, s94, 1\n\t"
"s_lshl_b32 s83, s83, 24\n\t"
"s_add_u32 s84, %[sldst], s83\n\t"
"s_add_u32 s85, s84, 0x4000\n\t"
"s_waitcnt vmcnt(8)\n\t"
"v_cvt_pk_f16_f32 v80, v64, v65\n\t"
"v_cvt_pk_f16_f32 v81, v66, v67\n\t"
"v_cvt_pk_f16_f32 v82, v68, v69\n\t"
"v_cvt_pk_f16_f32 v83, v70, v71\n\t"
"buffer_store_dwordx4 v[80:83], v41, %[srdWS], s84 offen sc0 sc1\n\t"
"v_cvt_pk_f16_f32 v84, v72, v73\n\t"
"v_cvt_pk_f16_f32 v85, v74, v75\n\t"
"v_cvt_pk_f16_f32 v86, v76, v77\n\t"
"v_cvt_pk_f16_f32 v87, v78, v79\n\t"
"buffer_store_dwordx4 v[84:87], v41, %[srdWS], s85 offen sc0 sc1\n\t"
"buffer_store_dword v52, off, %[srdC], %[sctr] sc1\n\t"
"s_mov_b32 %[ocall], s94\n\t"

      : [ocall] "=s"(call) : [srdW] "s"(srdW), [srdx] "s"(srdx), [srdC] "s"(srdC), [sctr] "s"(sctr), [sk0] "s"(sk0), [swrot] "s"(swrot), [slsrc] "s"(slsrc), [slmt] "s"(slmt),
         [vxsrc] "v"(vxsrc), [vwoff] "v"(vwoff), [vl16] "v"(vl16), [sldst] "s"(sldst), [srdWS] "s"(srdWS)
      : "memory", "scc", VCLOB, SCLOB);
  }
  unsigned* order = (unsigned*)(smem + 131072);
  if (tid < 96) {
    unsigned ks[2];
#pragma unroll
    for (int e = 0; e < 2; ++e) {
      const int i = tid + e;
      unsigned v = 256;
      if (i < 8) v = rank * 32 + h * 16 + 2 * i;
      else if (i < 64) { const int c = (i - 8) / 7, s = (i - 8) % 7; v = ((rank + 1 + s) & 7) * 32 + h * 16 + 2 * c; }
      ks[e] = v;
    }
    order[2 * tid] = ks[0];
    order[2 * tid + 1] = WS_XFLG + mt * 16384 + (ks[1] < 256 ? ks[1] : (unsigned)(rank * 32 + h * 16)) * 64;
  }
  unsigned t0 = OOB, t1 = OOB, t2 = OOB, t5 = 0xffffff00u;
  int d = rank;
  {
    int c = 0, u = 0; bool valid = false;
    if (lane < 9) {
      const int kk = 256 * lane + b, T = kk / 6, j = kk % 6;
      const int B = T / 6 - j; u = T % 6 + 6 * j;
      valid = (B >= 0 && B < 56 && u < 32);
      if (valid) { c = B / 7; d = (rank + 7 - B % 7) & 7; }
      const int sb = step_b(b, lane), sc = (lane == 0 && sb < 0) ? DBC : sb + DBC;
      const int delta = lane < 8 ? step_b(b, lane + 1) - DAB - sc - 1 : 0xfffff;
      t5 = (unsigned)delta << 8;
    }
    if (valid) {
      const int hd = u >> 4, mtd = (u >> 2) & 3, m16 = (u & 3) * 4 + wid, it = hd * 8 + c;
      t0 = (unsigned)(((d * ML + mtd * 256 + m16 * 16) * KL + it * 64) * 4);
      t1 = WS_X16 + (unsigned)(((mtd * 256 + rank * 32 + 2 * it) * 16 + m16) * 1024);
      t2 = WS_XFLG + (unsigned)(mtd * 16384 + (rank * 32 + 2 * it) * 64);
      t5 = (t5 & 0xffffff00u) | (unsigned)mtd;
    }
  }
  unsigned t3 = 0, t4 = 0;
#pragma unroll
  for (int p = 0; p < NR; ++p) {
    const uint64_t a = (uint64_t)P.ws[p];
    const unsigned lo = __builtin_amdgcn_readfirstlane((unsigned)a), hi = __builtin_amdgcn_readfirstlane((unsigned)(a >> 32)) & 0xffffu;
    t3 = d == p ? lo : t3; t4 = d == p ? hi : t4;
  }
  {
    const uint64_t xb = (uint64_t)(myws + WS_X16 + (size_t)mt * (256 * 16384));
    const unsigned sxlo = __builtin_amdgcn_readfirstlane((unsigned)xb), sxhi = __builtin_amdgcn_readfirstlane((unsigned)(xb >> 32)) & 0xffffu;
    const unsigned sldsb = __builtin_amdgcn_readfirstlane((unsigned)(size_t)smem), swv8k = wid * 8192;
    const unsigned swrot2 = mt, sxrot = idx & 7, sm0w = (wid ^ h) * 8192;
    const unsigned vla = (4 * c16 + (g ^ swz(c16 >> 2))) * 16;
    const unsigned sf0 = __builtin_amdgcn_readfirstlane(WS_XFLG + mt * 16384 + (rank * 32 + h * 16) * 64);
    const int snext0 = step_b(b, 0);
    const unsigned slflg = WS_XFLG + (unsigned)(((b >> 2) & 3) * 16384 + (rank * 32 + 2 * (((b >> 4) & 1) * 8 + (b >> 5))) * 64);
    const unsigned swsz = WS_XCH;
    const unsigned space = 200, sslope = 16;
    asm volatile(

"s_waitcnt vmcnt(1)\n\t"
"s_barrier\n\t"
"s_cmp_lg_u32 %[swv8k], 0\n\t"
"s_cbranch_scc1 L_nw0_%=\n\t"
"s_mov_b64 exec, 1\n\t"
"buffer_atomic_add v42, off, %[srdC], %[slflg] sc1\n\t"
"s_mov_b64 exec, -1\n\t"
"L_nw0_%=:\n\t"
"s_mov_b32 s94, %[scall]\n\t"
"s_and_b32 s83, s94, 1\n\t"
"s_lshl_b32 s83, s83, 24\n\t"
"s_add_u32 s49, %[swrot], 0\n\t"
"s_and_b32 s49, s49, 3\n\t"
"s_lshl_b32 s50, s49, 19\n\t"
"s_add_u32 s49, %[swrot], 1\n\t"
"s_and_b32 s49, s49, 3\n\t"
"s_lshl_b32 s51, s49, 19\n\t"
"s_add_u32 s49, %[swrot], 2\n\t"
"s_and_b32 s49, s49, 3\n\t"
"s_lshl_b32 s52, s49, 19\n\t"
"s_add_u32 s49, %[swrot], 3\n\t"
"s_and_b32 s49, s49, 3\n\t"
"s_lshl_b32 s53, s49, 19\n\t"
"s_lshl_b32 s63, %[sk0], 20\n\t"
"s_lshl_b32 s64, %[sk0], 14\n\t"
"s_mov_b32 s74, %[swsz]\n\t"
"s_mov_b32 s75, 0x20000\n\t"
"s_add_u32 s89, s63, 0\n\t"
"s_add_u32 s42, s89, s52\n\t"
"buffer_load_dwordx4 v[160:163], v60, %[srdW], s42 offen\n\t"
"buffer_load_dwordx4 v[164:167], v61, %[srdW], s42 offen\n\t"
"buffer_load_dwordx4 v[168:171], v62, %[srdW], s42 offen\n\t"
"buffer_load_dwordx4 v[172:175], v63, %[srdW], s42 offen\n\t"
"s_add_u32 s89, s63, 0\n\t"
"s_add_u32 s43, s89, s53\n\t"
"buffer_load_dwordx4 v[176:179], v60, %[srdW], s43 offen\n\t"
"buffer_load_dwordx4 v[180:183], v61, %[srdW], s43 offen\n\t"
"buffer_load_dwordx4 v[184:187], v62, %[srdW], s43 offen\n\t"
"buffer_load_dwordx4 v[188:191], v63, %[srdW], s43 offen\n\t"
"v_readlane_b32 s84, %[vt0], 0\n\t"
"v_readlane_b32 s85, %[vt5], 0\n\t"
"s_and_b32 s85, s85, 3\n\t"
"s_add_u32 s89, s85, 0\n\t"
"s_and_b32 s89, s89, 3\n\t"
"s_lshl_b32 s89, s89, 6\n\t"
"s_add_u32 s89, s89, s84\n\t"
"buffer_load_dwordx4 v[16:19], v40, %[srdx], s89 offen\n\t"
"s_add_u32 s89, s85, 1\n\t"
"s_and_b32 s89, s89, 3\n\t"
"s_lshl_b32 s89, s89, 6\n\t"
"s_add_u32 s89, s89, s84\n\t"
"buffer_load_dwordx4 v[20:23], v40, %[srdx], s89 offen\n\t"
"s_add_u32 s89, s85, 2\n\t"
"s_and_b32 s89, s89, 3\n\t"
"s_lshl_b32 s89, s89, 6\n\t"
"s_add_u32 s89, s89, s84\n\t"
"buffer_load_dwordx4 v[24:27], v40, %[srdx], s89 offen\n\t"
"s_add_u32 s89, s85, 3\n\t"
"s_and_b32 s89, s89, 3\n\t"
"s_lshl_b32 s89, s89, 6\n\t"
"s_add_u32 s89, s89, s84\n\t"
"buffer_load_dwordx4 v[28:31], v40, %[srdx], s89 offen\n\t"
"s_lshl_b32 s71, s94, 4\n\t"
"s_add_u32 s76, %[sxlo], s83\n\t"
"s_addc_u32 s77, %[sxhi], 0\n\t"
"s_mov_b32 s78, 0x400000\n\t"
"s_mov_b32 s79, 0x20000\n\t"
"v_add_u32 v59, %[swv8k], v41\n\t"
"v_add_u32 v58, %[sldsb], %[vla]\n\t"
"v_add_u32 v57, 0x10000, v58\n\t"
"s_add_u32 s47, %[sldsb], %[sm0w]\n\t"
"s_add_u32 s49, %[sxrot], 0\n\t"
"s_and_b32 s49, s49, 7\n\t"
"s_lshl_b32 s54, s49, 10\n\t"
"s_add_u32 s49, %[sxrot], 1\n\t"
"s_and_b32 s49, s49, 7\n\t"
"s_lshl_b32 s55, s49, 10\n\t"
"s_add_u32 s49, %[sxrot], 2\n\t"
"s_and_b32 s49, s49, 7\n\t"
"s_lshl_b32 s56, s49, 10\n\t"
"s_add_u32 s49, %[sxrot], 3\n\t"
"s_and_b32 s49, s49, 7\n\t"
"s_lshl_b32 s57, s49, 10\n\t"
"s_add_u32 s49, %[sxrot], 4\n\t"
"s_and_b32 s49, s49, 7\n\t"
"s_lshl_b32 s58, s49, 10\n\t"
"s_add_u32 s49, %[sxrot], 5\n\t"
"s_and_b32 s49, s49, 7\n\t"
"s_lshl_b32 s59, s49, 10\n\t"
"s_add_u32 s49, %[sxrot], 6\n\t"
"s_and_b32 s49, s49, 7\n\t"
"s_lshl_b32 s60, s49, 10\n\t"
"s_add_u32 s49, %[sxrot], 7\n\t"
"s_and_b32 s49, s49, 7\n\t"
"s_lshl_b32 s61, s49, 10\n\t"
"s_add_u32 s49, %[sldsb], 131128\n\t"
"v_mov_b32 v53, s49\n\t"
"s_mov_b32 s88, 0\n\t"
"s_mov_b32 s70, %[sf0]\n\t"
"v_and_b32 v44, 7, %[vlane]\n\t"
"v_lshlrev_b32 v44, 7, v44\n\t"
"v_mov_b32 v42, 1\n\t"
"v_cvt_pk_f16_f32 v96, v128, v132\n\t"
"v_cvt_pk_f16_f32 v100, v129, v133\n\t"
"v_cvt_pk_f16_f32 v104, v130, v134\n\t"
"v_cvt_pk_f16_f32 v108, v131, v135\n\t"
"v_cvt_pk_f16_f32 v97, v136, v140\n\t"
"v_cvt_pk_f16_f32 v101, v137, v141\n\t"
"v_cvt_pk_f16_f32 v105, v138, v142\n\t"
"v_cvt_pk_f16_f32 v109, v139, v143\n\t"
"v_cvt_pk_f16_f32 v98, v144, v148\n\t"
"v_cvt_pk_f16_f32 v102, v145, v149\n\t"
"v_cvt_pk_f16_f32 v106, v146, v150\n\t"
"v_cvt_pk_f16_f32 v110, v147, v151\n\t"
"v_cvt_pk_f16_f32 v99, v152, v156\n\t"
"v_cvt_pk_f16_f32 v103, v153, v157\n\t"
"v_cvt_pk_f16_f32 v107, v154, v158\n\t"
"v_cvt_pk_f16_f32 v111, v155, v159\n\t"
"s_cmp_lg_u32 %[swv8k], 0\n\t"
"s_cbranch_scc0 L_pp_%=\n\t"
"L_ppb_%=:\n\t"
"s_barrier\n\t"
"s_add_u32 s46, s64, 0\n\t"
"s_add_u32 s62, s46, s54\n\t"
"s_add_u32 s49, s47, s54\n\t"
"s_add_u32 m0, s49, 0\n\t"
"s_nop 0\n\t"
"buffer_load_dwordx4 v59, s[76:79], s62 offen sc1 lds\n\t"
"s_add_u32 s62, s46, s55\n\t"
"s_add_u32 s49, s47, s55\n\t"
"s_add_u32 m0, s49, 0\n\t"
"s_nop 0\n\t"
"buffer_load_dwordx4 v59, s[76:79], s62 offen sc1 lds\n\t"
"s_add_u32 s62, s46, s56\n\t"
"s_add_u32 s49, s47, s56\n\t"
"s_add_u32 m0, s49, 0\n\t"
"s_nop 0\n\t"
"buffer_load_dwordx4 v59, s[76:79], s62 offen sc1 lds\n\t"
"s_add_u32 s62, s46, s57\n\t"
"s_add_u32 s49, s47, s57\n\t"
"s_add_u32 m0, s49, 0\n\t"
"s_nop 0\n\t"
"buffer_load_dwordx4 v59, s[76:79], s62 offen sc1 lds\n\t"
"s_add_u32 s62, s46, s58\n\t"
"s_add_u32 s49, s47, s58\n\t"
"s_add_u32 m0, s49, 0\n\t"
"s_nop 0\n\t"
"buffer_load_dwordx4 v59, s[76:79], s62 offen sc1 lds\n\t"
"s_add_u32 s62, s46, s59\n\t"
"s_add_u32 s49, s47, s59\n\t"
"s_add_u32 m0, s49, 0\n\t"
"s_nop 0\n\t"
"buffer_load_dwordx4 v59, s[76:79], s62 offen sc1 lds\n\t"
"s_add_u32 s62, s46, s60\n\t"
"s_add_u32 s49, s47, s60\n\t"
"s_add_u32 m0, s49, 0\n\t"
"s_nop 0\n\t"
"buffer_load_dwordx4 v59, s[76:79], s62 offen sc1 lds\n\t"
"s_add_u32 s62, s46, s61\n\t"
"s_add_u32 s49, s47, s61\n\t"
"s_add_u32 m0, s49, 0\n\t"
"s_nop 0\n\t"
"buffer_load_dwordx4 v59, s[76:79], s62 offen sc1 lds\n\t"
"s_add_u32 s89, s63, 2097152\n\t"
"s_add_u32 s40, s89, s50\n\t"
"buffer_load_dwordx4 v[192:195], v60, %[srdW], s40 offen\n\t"
"buffer_load_dwordx4 v[196:199], v61, %[srdW], s40 offen\n\t"
"buffer_load_dwordx4 v[200:203], v62, %[srdW], s40 offen\n\t"
"buffer_load_dwordx4 v[204:207], v63, %[srdW], s40 offen\n\t"
"s_add_u32 s89, s63, 2097152\n\t"
"s_add_u32 s41, s89, s51\n\t"
"buffer_load_dwordx4 v[208:211], v60, %[srdW], s41 offen\n\t"
"buffer_load_dwordx4 v[212:215], v61, %[srdW], s41 offen\n\t"
"buffer_load_dwordx4 v[216:219], v62, %[srdW], s41 offen\n\t"
"buffer_load_dwordx4 v[220:223], v63, %[srdW], s41 offen\n\t"
"s_mov_b32 s82, 0\n\t"
"s_mov_b32 s81, 1\n\t"
"s_mov_b32 s80, %[snext0]\n\t"
"s_cmp_lt_i32 s80, 0\n\t"
"s_cbranch_scc0 L_nob0_%=\n\t"
"s_waitcnt vmcnt(16)\n\t"
"v_readlane_b32 s72, %[vt3], 0\n\t"
"v_readlane_b32 s73, %[vt4], 0\n\t"
"v_readlane_b32 s84, %[vt1], 0\n\t"
"v_cvt_pk_f16_f32 v32, v16, v17\n\t"
"v_cvt_pk_f16_f32 v33, v18, v19\n\t"
"v_cvt_pk_f16_f32 v34, v20, v21\n\t"
"v_cvt_pk_f16_f32 v35, v22, v23\n\t"
"v_cvt_pk_f16_f32 v36, v24, v25\n\t"
"v_cvt_pk_f16_f32 v37, v26, v27\n\t"
"v_cvt_pk_f16_f32 v38, v28, v29\n\t"
"v_cvt_pk_f16_f32 v39, v30, v31\n\t"
"s_add_u32 s84, s84, s83\n\t"
"s_add_u32 s85, s84, 0x4000\n\t"
"buffer_store_dwordx4 v[32:35], v41, s[72:75], s84 offen sc0 sc1\n\t"
"buffer_store_dwordx4 v[36:39], v41, s[72:75], s85 offen sc0 sc1\n\t"
"s_mov_b32 s81, 2\n\t"
"s_mov_b32 s80, 2\n\t"
"L_nob0_%=:\n\t"
"s_add_u32 s70, %[sf0], 896\n\t"
"s_waitcnt vmcnt(8) lgkmcnt(0)\n\t"
"s_memrealtime s[90:91]\n\t"
"s_waitcnt lgkmcnt(0)\n\t"
"s_mov_b32 s65, s90\n\t"
"s_lshl_b32 s95, %[space], 3\n\t"
"s_barrier\n\t"
"ds_read_b128 v[64:67], v58 offset:0\n\t"
"ds_read_b128 v[68:71], v58 offset:1024\n\t"
"ds_read_b128 v[72:75], v58 offset:2048\n\t"
"ds_read_b128 v[76:79], v58 offset:3072\n\t"
"s_mov_b32 s48, 16\n\t"
"s_waitcnt lgkmcnt(3)\n\t"
"v_mfma_f32_16x16x32_f16 a[0:3], v[64:67], v[96:99], 0\n\t"
"ds_read_b128 v[80:83], v58 offset:4096\n\t"
"s_add_u32 s46, s64, 32768\n\t"
"s_add_u32 s62, s46, s54\n\t"
"s_add_u32 s49, s47, s54\n\t"
"s_add_u32 m0, s49, 32768\n\t"
"s_waitcnt lgkmcnt(3)\n\t"
"v_mfma_f32_16x16x32_f16 a[16:19], v[68:71], v[96:99], 0\n\t"
"buffer_load_dwordx4 v59, s[76:79], s62 offen sc1 lds\n\t"
"v_mfma_f32_16x16x32_f16 a[20:23], v[68:71], v[100:103], 0\n\t"
"s_waitcnt vmcnt(27)\n\t"
"v_cvt_pk_f16_f32 v112, v160, v164\n\t"
"v_mfma_f32_16x16x32_f16 a[4:7], v[64:67], v[100:103], 0\n\t"
"v_cvt_pk_f16_f32 v116, v161, v165\n\t"
"s_add_u32 s62, s46, s55\n\t"
"s_add_u32 s49, s47, s55\n\t"
"s_add_u32 m0, s49, 32768\n\t"
"v_mfma_f32_16x16x32_f16 a[8:11], v[64:67], v[104:107], 0\n\t"
"ds_read_b128 v[84:87], v58 offset:5120\n\t"
"v_cvt_pk_f16_f32 v120, v162, v166\n\t"
"buffer_load_dwordx4 v59, s[76:79], s62 offen sc1 lds\n\t"
"v_mfma_f32_16x16x32_f16 a[24:27], v[68:71], v[104:107], 0\n\t"
"v_cvt_pk_f16_f32 v124, v163, v167\n\t"
"v_mfma_f32_16x16x32_f16 a[28:31], v[68:71], v[108:111], 0\n\t"
"v_mfma_f32_16x16x32_f16 a[12:15], v[64:67], v[108:111], 0\n\t"
"s_add_u32 s62, s46, s56\n\t"
"s_add_u32 s49, s47, s56\n\t"
"s_add_u32 m0, s49, 32768\n\t"
"s_waitcnt lgkmcnt(3)\n\t"
"v_mfma_f32_16x16x32_f16 a[44:47], v[72:75], v[108:111], 0\n\t"
"ds_read_b128 v[88:91], v58 offset:6144\n\t"
"buffer_load_dwordx4 v59, s[76:79], s62 offen sc1 lds\n\t"
"s_waitcnt lgkmcnt(3)\n\t"
"v_mfma_f32_16x16x32_f16 a[60:63], v[76:79], v[108:111], 0\n\t"
"v_mfma_f32_16x16x32_f16 a[56:59], v[76:79], v[104:107], 0\n\t"
"v_mfma_f32_16x16x32_f16 a[40:43], v[72:75], v[104:107], 0\n\t"
"s_add_u32 s62, s46, s57\n\t"
"s_add_u32 s49, s47, s57\n\t"
"s_add_u32 m0, s49, 32768\n\t"
"v_mfma_f32_16x16x32_f16 a[36:39], v[72:75], v[100:103], 0\n\t"
"ds_read_b128 v[92:95], v58 offset:7168\n\t"
"buffer_load_dwordx4 v59, s[76:79], s62 offen sc1 lds\n\t"
"v_mfma_f32_16x16x32_f16 a[52:55], v[76:79], v[100:103], 0\n\t"
"v_mfma_f32_16x16x32_f16 a[48:51], v[76:79], v[96:99], 0\n\t"
"v_mfma_f32_16x16x32_f16 a[32:35], v[72:75], v[96:99], 0\n\t"
"s_add_u32 s62, s46, s58\n\t"
"s_add_u32 s49, s47, s58\n\t"
"s_add_u32 m0, s49, 32768\n\t"
"s_waitcnt lgkmcnt(3)\n\t"
"v_mfma_f32_16x16x32_f16 a[64:67], v[80:83], v[96:99], 0\n\t"
"ds_read_b128 v[64:67], v58 offset:8192\n\t"
"buffer_load_dwordx4 v59, s[76:79], s62 offen sc1 lds\n\t"
"s_waitcnt lgkmcnt(3)\n\t"
"v_mfma_f32_16x16x32_f16 a[80:83], v[84:87], v[96:99], 0\n\t"
"v_mfma_f32_16x16x32_f16 a[84:87], v[84:87], v[100:103], 0\n\t"
"s_waitcnt vmcnt(29)\n\t"
"v_cvt_pk_f16_f32 v113, v168, v172\n\t"
"v_mfma_f32_16x16x32_f16 a[68:71], v[80:83], v[100:103], 0\n\t"
"v_cvt_pk_f16_f32 v117, v169, v173\n\t"
"s_add_u32 s62, s46, s59\n\t"
"s_add_u32 s49, s47, s59\n\t"
"s_add_u32 m0, s49, 32768\n\t"
"v_mfma_f32_16x16x32_f16 a[72:75], v[80:83], v[104:107], 0\n\t"
"ds_read_b128 v[68:71], v58 offset:9216\n\t"
"v_cvt_pk_f16_f32 v121, v170, v174\n\t"
"buffer_load_dwordx4 v59, s[76:79], s62 offen sc1 lds\n\t"
"v_mfma_f32_16x16x32_f16 a[88:91], v[84:87], v[104:107], 0\n\t"
"v_cvt_pk_f16_f32 v125, v171, v175\n\t"
"v_mfma_f32_16x16x32_f16 a[92:95], v[84:87], v[108:111], 0\n\t"
"v_mfma_f32_16x16x32_f16 a[76:79], v[80:83], v[108:111], 0\n\t"
"s_add_u32 s62, s46, s60\n\t"
"s_add_u32 s49, s47, s60\n\t"
"s_add_u32 m0, s49, 32768\n\t"
"s_waitcnt lgkmcnt(3)\n\t"
"v_mfma_f32_16x16x32_f16 a[108:111], v[88:91], v[108:111], 0\n\t"
"ds_read_b128 v[72:75], v58 offset:10240\n\t"
"buffer_load_dwordx4 v59, s[76:79], s62 offen sc1 lds\n\t"
"s_waitcnt lgkmcnt(3)\n\t"
"v_mfma_f32_16x16x32_f16 a[124:127], v[92:95], v[108:111], 0\n\t"
"v_mfma_f32_16x16x32_f16 a[120:123], v[92:95], v[104:107], 0\n\t"
"v_mfma_f32_16x16x32_f16 a[104:107], v[88:91], v[104:107], 0\n\t"
"s_add_u32 s62, s46, s61\n\t"
"s_add_u32 s49, s47, s61\n\t"
"s_add_u32 m0, s49, 32768\n\t"
"v_mfma_f32_16x16x32_f16 a[100:103], v[88:91], v[100:103], 0\n\t"
"ds_read_b128 v[76:79], v58 offset:11264\n\t"
"buffer_load_dwordx4 v59, s[76:79], s62 offen sc1 lds\n\t"
"v_mfma_f32_16x16x32_f16 a[116:119], v[92:95], v[100:103], 0\n\t"
"v_mfma_f32_16x16x32_f16 a[112:115], v[92:95], v[96:99], 0\n\t"
"v_mfma_f32_16x16x32_f16 a[96:99], v[88:91], v[96:99], 0\n\t"
"s_add_u32 s89, s63, 2097152\n\t"
"s_add_u32 s42, s89, s52\n\t"
"s_waitcnt lgkmcnt(3)\n\t"
"v_mfma_f32_16x16x32_f16 a[128:131], v[64:67], v[96:99], 0\n\t"
"ds_read_b128 v[80:83], v58 offset:12288\n\t"
"buffer_load_dwordx4 v[224:227], v60, %[srdW], s42 offen\n\t"
"s_waitcnt lgkmcnt(3)\n\t"
"v_mfma_f32_16x16x32_f16 a[144:147], v[68:71], v[96:99], 0\n\t"
"v_mfma_f32_16x16x32_f16 a[148:151], v[68:71], v[100:103], 0\n\t"
"s_waitcnt vmcnt(31)\n\t"
"v_cvt_pk_f16_f32 v114, v176, v180\n\t"
"v_mfma_f32_16x16x32_f16 a[132:135], v[64:67], v[100:103], 0\n\t"
"v_cvt_pk_f16_f32 v118, v177, v181\n\t"
"v_mfma_f32_16x16x32_f16 a[136:139], v[64:67], v[104:107], 0\n\t"
"ds_read_b128 v[84:87], v58 offset:13312\n\t"
"v_cvt_pk_f16_f32 v122, v178, v182\n\t"
"buffer_load_dwordx4 v[228:231], v61, %[srdW], s42 offen\n\t"
"v_mfma_f32_16x16x32_f16 a[152:155], v[68:71], v[104:107], 0\n\t"
"v_cvt_pk_f16_f32 v126, v179, v183\n\t"
"v_mfma_f32_16x16x32_f16 a[156:159], v[68:71], v[108:111], 0\n\t"
"v_mfma_f32_16x16x32_f16 a[140:143], v[64:67], v[108:111], 0\n\t"
"s_waitcnt lgkmcnt(3)\n\t"
"v_mfma_f32_16x16x32_f16 a[172:175], v[72:75], v[108:111], 0\n\t"
"ds_read_b128 v[88:91], v58 offset:14336\n\t"
"buffer_load_dwordx4 v[232:235], v62, %[srdW], s42 offen\n\t"
"s_waitcnt lgkmcnt(3)\n\t"
"v_mfma_f32_16x16x32_f16 a[188:191], v[76:79], v[108:111], 0\n\t"
"v_mfma_f32_16x16x32_f16 a[184:187], v[76:79], v[104:107], 0\n\t"
"v_mfma_f32_16x16x32_f16 a[168:171], v[72:75], v[104:107], 0\n\t"
"v_mfma_f32_16x16x32_f16 a[164:167], v[72:75], v[100:103], 0\n\t"
"ds_read_b128 v[92:95], v58 offset:15360\n\t"
"buffer_load_dwordx4 v[236:239], v63, %[srdW], s42 offen\n\t"
"v_mfma_f32_16x16x32_f16 a[180:183], v[76:79], v[100:103], 0\n\t"
"v_mfma_f32_16x16x32_f16 a[176:179], v[76:79], v[96:99], 0\n\t"
"v_mfma_f32_16x16x32_f16 a[160:163], v[72:75], v[96:99], 0\n\t"
"s_add_u32 s89, s63, 2097152\n\t"
"s_add_u32 s43, s89, s53\n\t"
"s_waitcnt lgkmcnt(3)\n\t"
"v_mfma_f32_16x16x32_f16 a[192:195], v[80:83], v[96:99], 0\n\t"
"ds_read_b128 v[64:67], v58 offset:16384\n\t"
"buffer_load_dwordx4 v[240:243], v60, %[srdW], s43 offen\n\t"
"s_waitcnt lgkmcnt(3)\n\t"
"v_mfma_f32_16x16x32_f16 a[208:211], v[84:87], v[96:99], 0\n\t"
"v_mfma_f32_16x16x32_f16 a[212:215], v[84:87], v[100:103], 0\n\t"
"s_waitcnt vmcnt(33)\n\t"
"v_cvt_pk_f16_f32 v115, v184, v188\n\t"
"v_mfma_f32_16x16x32_f16 a[196:199], v[80:83], v[100:103], 0\n\t"
"v_cvt_pk_f16_f32 v119, v185, v189\n\t"
"v_mfma_f32_16x16x32_f16 a[200:203], v[80:83], v[104:107], 0\n\t"
"ds_read_b128 v[68:71], v58 offset:17408\n\t"
"v_cvt_pk_f16_f32 v123, v186, v190\n\t"
"buffer_load_dwordx4 v[244:247], v61, %[srdW], s43 offen\n\t"
"v_mfma_f32_16x16x32_f16 a[216:219], v[84:87], v[104:107], 0\n\t"
"v_cvt_pk_f16_f32 v127, v187, v191\n\t"
"v_mfma_f32_16x16x32_f16 a[220:223], v[84:87], v[108:111], 0\n\t"
"v_mfma_f32_16x16x32_f16 a[204:207], v[80:83], v[108:111], 0\n\t"
"s_waitcnt lgkmcnt(3)\n\t"
"v_mfma_f32_16x16x32_f16 a[236:239], v[88:91], v[108:111], 0\n\t"
"ds_read_b128 v[72:75], v58 offset:18432\n\t"
"buffer_load_dwordx4 v[248:251], v62, %[srdW], s43 offen\n\t"
"s_waitcnt lgkmcnt(3)\n\t"
"v_mfma_f32_16x16x32_f16 a[252:255], v[92:95], v[108:111], 0\n\t"
"v_mfma_f32_16x16x32_f16 a[248:251], v[92:95], v[104:107], 0\n\t"
"v_mfma_f32_16x16x32_f16 a[232:235], v[88:91], v[104:107], 0\n\t"
"v_mfma_f32_16x16x32_f16 a[228:231], v[88:91], v[100:103], 0\n\t"
"ds_read_b128 v[76:79], v58 offset:19456\n\t"
"buffer_load_dwordx4 v[252:255], v63, %[srdW], s43 offen\n\t"
"v_mfma_f32_16x16x32_f16 a[244:247], v[92:95], v[100:103], 0\n\t"
"s_sub_u32 s80, s80, 1\n\t"
"s_getpc_b64 s[68:69]\n\t"
"s_cbranch_scc1 L_hnd_%=\n\t"
"v_mfma_f32_16x16x32_f16 a[240:243], v[92:95], v[96:99], 0\n\t"
"v_mfma_f32_16x16x32_f16 a[224:227], v[88:91], v[96:99], 0\n\t"
"s_add_u32 s89, s63, 4194304\n\t"
"s_add_u32 s40, s89, s50\n\t"
"s_waitcnt lgkmcnt(3)\n\t"
"v_mfma_f32_16x16x32_f16 a[0:3], v[64:67], v[112:115], a[0:3]\n\t"
"ds_read_b128 v[80:83], v58 offset:20480\n\t"
"buffer_load_dwordx4 v[128:131], v60, %[srdW], s40 offen\n\t"
"s_waitcnt lgkmcnt(3)\n\t"
"v_mfma_f32_16x16x32_f16 a[16:19], v[68:71], v[112:115], a[16:19]\n\t"
"v_mfma_f32_16x16x32_f16 a[20:23], v[68:71], v[116:119], a[20:23]\n\t"
"s_waitcnt vmcnt(23)\n\t"
"v_cvt_pk_f16_f32 v96, v192, v196\n\t"
"v_mfma_f32_16x16x32_f16 a[4:7], v[64:67], v[116:119], a[4:7]\n\t"
"v_cvt_pk_f16_f32 v100, v193, v197\n\t"
"buffer_load_dwordx4 v[132:135], v61, %[srdW], s40 offen\n\t"
"v_mfma_f32_16x16x32_f16 a[8:11], v[64:67], v[120:123], a[8:11]\n\t"
"ds_read_b128 v[84:87], v58 offset:21504\n\t"
"v_cvt_pk_f16_f32 v104, v194, v198\n\t"
"v_mfma_f32_16x16x32_f16 a[24:27], v[68:71], v[120:123], a[24:27]\n\t"
"v_cvt_pk_f16_f32 v108, v195, v199\n\t"
"v_mfma_f32_16x16x32_f16 a[28:31], v[68:71], v[124:127], a[28:31]\n\t"
"s_add_u32 s46, s64, 65536\n\t"
"s_add_u32 s62, s46, s54\n\t"
"s_add_u32 s49, s47, s54\n\t"
"s_add_u32 m0, s49, 65536\n\t"
"v_mfma_f32_16x16x32_f16 a[12:15], v[64:67], v[124:127], a[12:15]\n\t"
"buffer_load_dwordx4 v59, s[76:79], s62 offen sc1 lds\n\t"
"s_waitcnt lgkmcnt(3)\n\t"
"v_mfma_f32_16x16x32_f16 a[44:47], v[72:75], v[124:127], a[44:47]\n\t"
"ds_read_b128 v[88:91], v58 offset:22528\n\t"
"s_waitcnt lgkmcnt(3)\n\t"
"v_mfma_f32_16x16x32_f16 a[60:63], v[76:79], v[124:127], a[60:63]\n\t"
"v_mfma_f32_16x16x32_f16 a[56:59], v[76:79], v[120:123], a[56:59]\n\t"
"v_mfma_f32_16x16x32_f16 a[40:43], v[72:75], v[120:123], a[40:43]\n\t"
"buffer_load_dwordx4 v[136:139], v62, %[srdW], s40 offen\n\t"
"v_mfma_f32_16x16x32_f16 a[36:39], v[72:75], v[116:119], a[36:39]\n\t"
"ds_read_b128 v[92:95], v58 offset:23552\n\t"
"v_mfma_f32_16x16x32_f16 a[52:55], v[76:79], v[116:119], a[52:55]\n\t"
"v_mfma_f32_16x16x32_f16 a[48:51], v[76:79], v[112:115], a[48:51]\n\t"
"v_mfma_f32_16x16x32_f16 a[32:35], v[72:75], v[112:115], a[32:35]\n\t"
"buffer_load_dwordx4 v[140:143], v63, %[srdW], s40 offen\n\t"
"s_waitcnt lgkmcnt(3)\n\t"
"v_mfma_f32_16x16x32_f16 a[64:67], v[80:83], v[112:115], a[64:67]\n\t"
"ds_read_b128 v[64:67], v58 offset:24576\n\t"
"s_waitcnt lgkmcnt(3)\n\t"
"v_mfma_f32_16x16x32_f16 a[80:83], v[84:87], v[112:115], a[80:83]\n\t"
"v_mfma_f32_16x16x32_f16 a[84:87], v[84:87], v[116:119], a[84:87]\n\t"
"s_waitcnt vmcnt(25)\n\t"
"v_cvt_pk_f16_f32 v97, v200, v204\n\t"
"s_add_u32 s62, s46, s55\n\t"
"s_add_u32 s49, s47, s55\n\t"
"s_add_u32 m0, s49, 65536\n\t"
"v_mfma_f32_16x16x32_f16 a[68:71], v[80:83], v[116:119], a[68:71]\n\t"
"v_cvt_pk_f16_f32 v101, v201, v205\n\t"
"buffer_load_dwordx4 v59, s[76:79], s62 offen sc1 lds\n\t"
"v_mfma_f32_16x16x32_f16 a[72:75], v[80:83], v[120:123], a[72:75]\n\t"
"ds_read_b128 v[68:71], v58 offset:25600\n\t"
"v_cvt_pk_f16_f32 v105, v202, v206\n\t"
"v_mfma_f32_16x16x32_f16 a[88:91], v[84:87], v[120:123], a[88:91]\n\t"
"v_cvt_pk_f16_f32 v109, v203, v207\n\t"
"v_mfma_f32_16x16x32_f16 a[92:95], v[84:87], v[124:127], a[92:95]\n\t"
"s_add_u32 s89, s63, 4194304\n\t"
"s_add_u32 s41, s89, s51\n\t"
"v_mfma_f32_16x16x32_f16 a[76:79], v[80:83], v[124:127], a[76:79]\n\t"
"buffer_load_dwordx4 v[144:147], v60, %[srdW], s41 offen\n\t"
"s_waitcnt lgkmcnt(3)\n\t"
"v_mfma_f32_16x16x32_f16 a[108:111], v[88:91], v[124:127], a[108:111]\n\t"
"ds_read_b128 v[72:75], v58 offset:26624\n\t"
"s_waitcnt lgkmcnt(3)\n\t"
"v_mfma_f32_16x16x32_f16 a[124:127], v[92:95], v[124:127], a[124:127]\n\t"
"v_mfma_f32_16x16x32_f16 a[120:123], v[92:95], v[120:123], a[120:123]\n\t"
"v_mfma_f32_16x16x32_f16 a[104:107], v[88:91], v[120:123], a[104:107]\n\t"
"buffer_load_dwordx4 v[148:151], v61, %[srdW], s41 offen\n\t"
"v_mfma_f32_16x16x32_f16 a[100:103], v[88:91], v[116:119], a[100:103]\n\t"
"ds_read_b128 v[76:79], v58 offset:27648\n\t"
"v_mfma_f32_16x16x32_f16 a[116:119], v[92:95], v[116:119], a[116:119]\n\t"
"v_mfma_f32_16x16x32_f16 a[112:115], v[92:95], v[112:115], a[112:115]\n\t"
"s_add_u32 s62, s46, s56\n\t"
"s_add_u32 s49, s47, s56\n\t"
"s_add_u32 m0, s49, 65536\n\t"
"v_mfma_f32_16x16x32_f16 a[96:99], v[88:91], v[112:115], a[96:99]\n\t"
"buffer_load_dwordx4 v59, s[76:79], s62 offen sc1 lds\n\t"
"s_waitcnt lgkmcnt(3)\n\t"
"v_mfma_f32_16x16x32_f16 a[128:131], v[64:67], v[112:115], a[128:131]\n\t"
"ds_read_b128 v[80:83], v58 offset:28672\n\t"
"s_waitcnt lgkmcnt(3)\n\t"
"v_mfma_f32_16x16x32_f16 a[144:147], v[68:71], v[112:115], a[144:147]\n\t"
"v_mfma_f32_16x16x32_f16 a[148:151], v[68:71], v[116:119], a[148:151]\n\t"
"s_waitcnt vmcnt(27)\n\t"
"v_cvt_pk_f16_f32 v98, v208, v212\n\t"
"v_mfma_f32_16x16x32_f16 a[132:135], v[64:67], v[116:119], a[132:135]\n\t"
"v_cvt_pk_f16_f32 v102, v209, v213\n\t"
"buffer_load_dwordx4 v[152:155], v62, %[srdW], s41 offen\n\t"
"v_mfma_f32_16x16x32_f16 a[136:139], v[64:67], v[120:123], a[136:139]\n\t"
"ds_read_b128 v[84:87], v58 offset:29696\n\t"
"v_cvt_pk_f16_f32 v106, v210, v214\n\t"
"v_mfma_f32_16x16x32_f16 a[152:155], v[68:71], v[120:123], a[152:155]\n\t"
"v_cvt_pk_f16_f32 v110, v211, v215\n\t"
"v_mfma_f32_16x16x32_f16 a[156:159], v[68:71], v[124:127], a[156:159]\n\t"
"v_mfma_f32_16x16x32_f16 a[140:143], v[64:67], v[124:127], a[140:143]\n\t"
"buffer_load_dwordx4 v[156:159], v63, %[srdW], s41 offen\n\t"
"s_waitcnt lgkmcnt(3)\n\t"
"v_mfma_f32_16x16x32_f16 a[172:175], v[72:75], v[124:127], a[172:175]\n\t"
"ds_read_b128 v[88:91], v58 offset:30720\n\t"
"s_waitcnt lgkmcnt(3)\n\t"
"v_mfma_f32_16x16x32_f16 a[188:191], v[76:79], v[124:127], a[188:191]\n\t"
"v_mfma_f32_16x16x32_f16 a[184:187], v[76:79], v[120:123], a[184:187]\n\t"
"s_add_u32 s62, s46, s57\n\t"
"s_add_u32 s49, s47, s57\n\t"
"s_add_u32 m0, s49, 65536\n\t"
"v_mfma_f32_16x16x32_f16 a[168:171], v[72:75], v[120:123], a[168:171]\n\t"
"buffer_load_dwordx4 v59, s[76:79], s62 offen sc1 lds\n\t"
"v_mfma_f32_16x16x32_f16 a[164:167], v[72:75], v[116:119], a[164:167]\n\t"
"ds_read_b128 v[92:95], v58 offset:31744\n\t"
"v_mfma_f32_16x16x32_f16 a[180:183], v[76:79], v[116:119], a[180:183]\n\t"
"v_mfma_f32_16x16x32_f16 a[176:179], v[76:79], v[112:115], a[176:179]\n\t"
"s_add_u32 s89, s63, 4194304\n\t"
"s_add_u32 s42, s89, s52\n\t"
"v_mfma_f32_16x16x32_f16 a[160:163], v[72:75], v[112:115], a[160:163]\n\t"
"buffer_load_dwordx4 v[160:163], v60, %[srdW], s42 offen\n\t"
"s_waitcnt lgkmcnt(3)\n\t"
"v_mfma_f32_16x16x32_f16 a[192:195], v[80:83], v[112:115], a[192:195]\n\t"
"s_waitcnt lgkmcnt(2)\n\t"
"v_mfma_f32_16x16x32_f16 a[208:211], v[84:87], v[112:115], a[208:211]\n\t"
"s_memrealtime s[90:91]\n\t"
"v_mfma_f32_16x16x32_f16 a[212:215], v[84:87], v[116:119], a[212:215]\n\t"
"s_waitcnt vmcnt(29)\n\t"
"v_cvt_pk_f16_f32 v99, v216, v220\n\t"
"v_mfma_f32_16x16x32_f16 a[196:199], v[80:83], v[116:119], a[196:199]\n\t"
"v_cvt_pk_f16_f32 v103, v217, v221\n\t"
"buffer_load_dwordx4 v[164:167], v61, %[srdW], s42 offen\n\t"
"s_waitcnt vmcnt(22) lgkmcnt(0)\n\t"
"s_sub_u32 s89, s90, s65\n\t"
"s_cmp_lt_i32 s89, 0\n\t"
"s_getpc_b64 s[68:69]\n\t"
"s_cbranch_scc1 L_pace_%=\n\t"
"s_ashr_i32 s89, s95, 3\n\t"
"s_add_u32 s65, s90, s89\n\t"
"s_sub_u32 s95, s95, %[sslope]\n\t"
"s_barrier\n\t"
"v_mfma_f32_16x16x32_f16 a[200:203], v[80:83], v[120:123], a[200:203]\n\t"
"ds_read_b128 v[64:67], v58 offset:32768\n\t"
"v_cvt_pk_f16_f32 v107, v218, v222\n\t"
"v_mfma_f32_16x16x32_f16 a[216:219], v[84:87], v[120:123], a[216:219]\n\t"
"v_cvt_pk_f16_f32 v111, v219, v223\n\t"
"v_mfma_f32_16x16x32_f16 a[220:223], v[84:87], v[124:127], a[220:223]\n\t"
"s_add_u32 s62, s46, s58\n\t"
"s_add_u32 s49, s47, s58\n\t"
"s_add_u32 m0, s49, 65536\n\t"
"v_mfma_f32_16x16x32_f16 a[204:207], v[80:83], v[124:127], a[204:207]\n\t"
"ds_read_b128 v[68:71], v58 offset:33792\n\t"
"buffer_load_dwordx4 v59, s[76:79], s62 offen sc1 lds\n\t"
"s_waitcnt lgkmcnt(3)\n\t"
"v_mfma_f32_16x16x32_f16 a[236:239], v[88:91], v[124:127], a[236:239]\n\t"
"s_waitcnt lgkmcnt(2)\n\t"
"v_mfma_f32_16x16x32_f16 a[252:255], v[92:95], v[124:127], a[252:255]\n\t"
"v_mfma_f32_16x16x32_f16 a[248:251], v[92:95], v[120:123], a[248:251]\n\t"
"ds_read_b128 v[72:75], v58 offset:34816\n\t"
"v_mfma_f32_16x16x32_f16 a[232:235], v[88:91], v[120:123], a[232:235]\n\t"
"buffer_load_dwordx4 v[168:171], v62, %[srdW], s42 offen\n\t"
"v_mfma_f32_16x16x32_f16 a[228:231], v[88:91], v[116:119], a[228:231]\n\t"
"v_mfma_f32_16x16x32_f16 a[244:247], v[92:95], v[116:119], a[244:247]\n\t"
"ds_read_b128 v[76:79], v58 offset:35840\n\t"
"s_sub_u32 s80, s80, 1\n\t"
"s_getpc_b64 s[68:69]\n\t"
"s_cbranch_scc1 L_hnd_%=\n\t"
"v_mfma_f32_16x16x32_f16 a[240:243], v[92:95], v[112:115], a[240:243]\n\t"
"v_mfma_f32_16x16x32_f16 a[224:227], v[88:91], v[112:115], a[224:227]\n\t"
"buffer_load_dwordx4 v[172:175], v63, %[srdW], s42 offen\n\t"
"s_waitcnt lgkmcnt(3)\n\t"
"v_mfma_f32_16x16x32_f16 a[0:3], v[64:67], v[96:99], a[0:3]\n\t"
"ds_read_b128 v[80:83], v58 offset:36864\n\t"
"s_waitcnt lgkmcnt(3)\n\t"
"v_mfma_f32_16x16x32_f16 a[16:19], v[68:71], v[96:99], a[16:19]\n\t"
"s_add_u32 s62, s46, s59\n\t"
"s_add_u32 s49, s47, s59\n\t"
"s_add_u32 m0, s49, 65536\n\t"
"v_mfma_f32_16x16x32_f16 a[20:23], v[68:71], v[100:103], a[20:23]\n\t"
"s_waitcnt vmcnt(23)\n\t"
"v_cvt_pk_f16_f32 v112, v224, v228\n\t"
"buffer_load_dwordx4 v59, s[76:79], s62 offen sc1 lds\n\t"
"v_mfma_f32_16x16x32_f16 a[4:7], v[64:67], v[100:103], a[4:7]\n\t"
"v_cvt_pk_f16_f32 v116, v225, v229\n\t"
"v_mfma_f32_16x16x32_f16 a[8:11], v[64:67], v[104:107], a[8:11]\n\t"
"ds_read_b128 v[84:87], v58 offset:37888\n\t"
"v_cvt_pk_f16_f32 v120, v226, v230\n\t"
"v_mfma_f32_16x16x32_f16 a[24:27], v[68:71], v[104:107], a[24:27]\n\t"
"v_cvt_pk_f16_f32 v124, v227, v231\n\t"
"s_add_u32 s89, s63, 4194304\n\t"
"s_add_u32 s43, s89, s53\n\t"
"v_mfma_f32_16x16x32_f16 a[28:31], v[68:71], v[108:111], a[28:31]\n\t"
"buffer_load_dwordx4 v[176:179], v60, %[srdW], s43 offen\n\t"
"v_mfma_f32_16x16x32_f16 a[12:15], v[64:67], v[108:111], a[12:15]\n\t"
"s_waitcnt lgkmcnt(3)\n\t"
"v_mfma_f32_16x16x32_f16 a[44:47], v[72:75], v[108:111], a[44:47]\n\t"
"ds_read_b128 v[88:91], v58 offset:38912\n\t"
"s_waitcnt lgkmcnt(3)\n\t"
"v_mfma_f32_16x16x32_f16 a[60:63], v[76:79], v[108:111], a[60:63]\n\t"
"v_mfma_f32_16x16x32_f16 a[56:59], v[76:79], v[104:107], a[56:59]\n\t"
"buffer_load_dwordx4 v[180:183], v61, %[srdW], s43 offen\n\t"
"v_mfma_f32_16x16x32_f16 a[40:43], v[72:75], v[104:107], a[40:43]\n\t"
"v_mfma_f32_16x16x32_f16 a[36:39], v[72:75], v[100:103], a[36:39]\n\t"
"ds_read_b128 v[92:95], v58 offset:39936\n\t"
"v_mfma_f32_16x16x32_f16 a[52:55], v[76:79], v[100:103], a[52:55]\n\t"
"s_add_u32 s62, s46, s60\n\t"
"s_add_u32 s49, s47, s60\n\t"
"s_add_u32 m0, s49, 65536\n\t"
"v_mfma_f32_16x16x32_f16 a[48:51], v[76:79], v[96:99], a[48:51]\n\t"
"buffer_load_dwordx4 v59, s[76:79], s62 offen sc1 lds\n\t"
"v_mfma_f32_16x16x32_f16 a[32:35], v[72:75], v[96:99], a[32:35]\n\t"
"s_waitcnt lgkmcnt(3)\n\t"
"v_mfma_f32_16x16x32_f16 a[64:67], v[80:83], v[96:99], a[64:67]\n\t"
"ds_read_b128 v[64:67], v58 offset:40960\n\t"
"s_waitcnt lgkmcnt(3)\n\t"
"v_mfma_f32_16x16x32_f16 a[80:83], v[84:87], v[96:99], a[80:83]\n\t"
"v_mfma_f32_16x16x32_f16 a[84:87], v[84:87], v[100:103], a[84:87]\n\t"
"s_waitcnt vmcnt(25)\n\t"
"v_cvt_pk_f16_f32 v113, v232, v236\n\t"
"buffer_load_dwordx4 v[184:187], v62, %[srdW], s43 offen\n\t"
"v_mfma_f32_16x16x32_f16 a[68:71], v[80:83], v[100:103], a[68:71]\n\t"
"v_cvt_pk_f16_f32 v117, v233, v237\n\t"
"v_mfma_f32_16x16x32_f16 a[72:75], v[80:83], v[104:107], a[72:75]\n\t"
"ds_read_b128 v[68:71], v58 offset:41984\n\t"
"v_cvt_pk_f16_f32 v121, v234, v238\n\t"
"v_mfma_f32_16x16x32_f16 a[88:91], v[84:87], v[104:107], a[88:91]\n\t"
"v_cvt_pk_f16_f32 v125, v235, v239\n\t"
"v_mfma_f32_16x16x32_f16 a[92:95], v[84:87], v[108:111], a[92:95]\n\t"
"buffer_load_dwordx4 v[188:191], v63, %[srdW], s43 offen\n\t"
"v_mfma_f32_16x16x32_f16 a[76:79], v[80:83], v[108:111], a[76:79]\n\t"
"s_waitcnt lgkmcnt(3)\n\t"
"v_mfma_f32_16x16x32_f16 a[108:111], v[88:91], v[108:111], a[108:111]\n\t"
"ds_read_b128 v[72:75], v58 offset:43008\n\t"
"s_waitcnt lgkmcnt(3)\n\t"
"v_mfma_f32_16x16x32_f16 a[124:127], v[92:95], v[108:111], a[124:127]\n\t"
"s_add_u32 s62, s46, s61\n\t"
"s_add_u32 s49, s47, s61\n\t"
"s_add_u32 m0, s49, 65536\n\t"
"v_mfma_f32_16x16x32_f16 a[120:123], v[92:95], v[104:107], a[120:123]\n\t"
"buffer_load_dwordx4 v59, s[76:79], s62 offen sc1 lds\n\t"
"v_mfma_f32_16x16x32_f16 a[104:107], v[88:91], v[104:107], a[104:107]\n\t"
"v_mfma_f32_16x16x32_f16 a[100:103], v[88:91], v[100:103], a[100:103]\n\t"
"ds_read_b128 v[76:79], v58 offset:44032\n\t"
"v_mfma_f32_16x16x32_f16 a[116:119], v[92:95], v[100:103], a[116:119]\n\t"
"s_add_u32 s89, s63, 6291456\n\t"
"s_add_u32 s40, s89, s50\n\t"
"v_mfma_f32_16x16x32_f16 a[112:115], v[92:95], v[96:99], a[112:115]\n\t"
"buffer_load_dwordx4 v[192:195], v60, %[srdW], s40 offen\n\t"
"v_mfma_f32_16x16x32_f16 a[96:99], v[88:91], v[96:99], a[96:99]\n\t"
"s_waitcnt lgkmcnt(3)\n\t"
"v_mfma_f32_16x16x32_f16 a[128:131], v[64:67], v[96:99], a[128:131]\n\t"
"ds_read_b128 v[80:83], v58 offset:45056\n\t"
"s_waitcnt lgkmcnt(3)\n\t"
"v_mfma_f32_16x16x32_f16 a[144:147], v[68:71], v[96:99], a[144:147]\n\t"
"v_mfma_f32_16x16x32_f16 a[148:151], v[68:71], v[100:103], a[148:151]\n\t"
"s_waitcnt vmcnt(27)\n\t"
"v_cvt_pk_f16_f32 v114, v240, v244\n\t"
"buffer_load_dwordx4 v[196:199], v61, %[srdW], s40 offen\n\t"
"v_mfma_f32_16x16x32_f16 a[132:135], v[64:67], v[100:103], a[132:135]\n\t"
"v_cvt_pk_f16_f32 v118, v241, v245\n\t"
"v_mfma_f32_16x16x32_f16 a[136:139], v[64:67], v[104:107], a[136:139]\n\t"
"ds_read_b128 v[84:87], v58 offset:46080\n\t"
"v_cvt_pk_f16_f32 v122, v242, v246\n\t"
"v_mfma_f32_16x16x32_f16 a[152:155], v[68:71], v[104:107], a[152:155]\n\t"
"v_cvt_pk_f16_f32 v126, v243, v247\n\t"
"s_add_u32 s46, s64, 98304\n\t"
"s_add_u32 s62, s46, s54\n\t"
"s_add_u32 s49, s47, s54\n\t"
"s_add_u32 m0, s49, 98304\n\t"
"v_mfma_f32_16x16x32_f16 a[156:159], v[68:71], v[108:111], a[156:159]\n\t"
"buffer_load_dwordx4 v59, s[76:79], s62 offen sc1 lds\n\t"
"v_mfma_f32_16x16x32_f16 a[140:143], v[64:67], v[108:111], a[140:143]\n\t"
"s_waitcnt lgkmcnt(3)\n\t"
"v_mfma_f32_16x16x32_f16 a[172:175], v[72:75], v[108:111], a[172:175]\n\t"
"ds_read_b128 v[88:91], v58 offset:47104\n\t"
"s_waitcnt lgkmcnt(3)\n\t"
"v_mfma_f32_16x16x32_f16 a[188:191], v[76:79], v[108:111], a[188:191]\n\t"
"v_mfma_f32_16x16x32_f16 a[184:187], v[76:79], v[104:107], a[184:187]\n\t"
"buffer_load_dwordx4 v[200:203], v62, %[srdW], s40 offen\n\t"
"v_mfma_f32_16x16x32_f16 a[168:171], v[72:75], v[104:107], a[168:171]\n\t"
"v_mfma_f32_16x16x32_f16 a[164:167], v[72:75], v[100:103], a[164:167]\n\t"
"ds_read_b128 v[92:95], v58 offset:48128\n\t"
"v_mfma_f32_16x16x32_f16 a[180:183], v[76:79], v[100:103], a[180:183]\n\t"
"v_mfma_f32_16x16x32_f16 a[176:179], v[76:79], v[96:99], a[176:179]\n\t"
"buffer_load_dwordx4 v[204:207], v63, %[srdW], s40 offen\n\t"
"v_mfma_f32_16x16x32_f16 a[160:163], v[72:75], v[96:99], a[160:163]\n\t"
"s_waitcnt lgkmcnt(3)\n\t"
"v_mfma_f32_16x16x32_f16 a[192:195], v[80:83], v[96:99], a[192:195]\n\t"
"ds_read_b128 v[64:67], v58 offset:49152\n\t"
"s_waitcnt lgkmcnt(3)\n\t"
"v_mfma_f32_16x16x32_f16 a[208:211], v[84:87], v[96:99], a[208:211]\n\t"
"s_add_u32 s62, s46, s55\n\t"
"s_add_u32 s49, s47, s55\n\t"
"s_add_u32 m0, s49, 98304\n\t"
"v_mfma_f32_16x16x32_f16 a[212:215], v[84:87], v[100:103], a[212:215]\n\t"
"s_waitcnt vmcnt(29)\n\t"
"v_cvt_pk_f16_f32 v115, v248, v252\n\t"
"buffer_load_dwordx4 v59, s[76:79], s62 offen sc1 lds\n\t"
"v_mfma_f32_16x16x32_f16 a[196:199], v[80:83], v[100:103], a[196:199]\n\t"
"v_cvt_pk_f16_f32 v119, v249, v253\n\t"
"v_mfma_f32_16x16x32_f16 a[200:203], v[80:83], v[104:107], a[200:203]\n\t"
"ds_read_b128 v[68:71], v58 offset:50176\n\t"
"v_cvt_pk_f16_f32 v123, v250, v254\n\t"
"v_mfma_f32_16x16x32_f16 a[216:219], v[84:87], v[104:107], a[216:219]\n\t"
"v_cvt_pk_f16_f32 v127, v251, v255\n\t"
"s_add_u32 s89, s63, 6291456\n\t"
"s_add_u32 s41, s89, s51\n\t"
"v_mfma_f32_16x16x32_f16 a[220:223], v[84:87], v[108:111], a[220:223]\n\t"
"buffer_load_dwordx4 v[208:211], v60, %[srdW], s41 offen\n\t"
"v_mfma_f32_16x16x32_f16 a[204:207], v[80:83], v[108:111], a[204:207]\n\t"
"s_waitcnt lgkmcnt(3)\n\t"
"v_mfma_f32_16x16x32_f16 a[236:239], v[88:91], v[108:111], a[236:239]\n\t"
"ds_read_b128 v[72:75], v58 offset:51200\n\t"
"s_waitcnt lgkmcnt(3)\n\t"
"v_mfma_f32_16x16x32_f16 a[252:255], v[92:95], v[108:111], a[252:255]\n\t"
"v_mfma_f32_16x16x32_f16 a[248:251], v[92:95], v[104:107], a[248:251]\n\t"
"buffer_load_dwordx4 v[212:215], v61, %[srdW], s41 offen\n\t"
"v_mfma_f32_16x16x32_f16 a[232:235], v[88:91], v[104:107], a[232:235]\n\t"
"v_mfma_f32_16x16x32_f16 a[228:231], v[88:91], v[100:103], a[228:231]\n\t"
"ds_read_b128 v[76:79], v58 offset:52224\n\t"
"v_mfma_f32_16x16x32_f16 a[244:247], v[92:95], v[100:103], a[244:247]\n\t"
"s_add_u32 s62, s46, s56\n\t"
"s_add_u32 s49, s47, s56\n\t"
"s_add_u32 m0, s49, 98304\n\t"
"s_sub_u32 s80, s80, 1\n\t"
"s_getpc_b64 s[68:69]\n\t"
"s_cbranch_scc1 L_hnd_%=\n\t"
"v_mfma_f32_16x16x32_f16 a[240:243], v[92:95], v[96:99], a[240:243]\n\t"
"buffer_load_dwordx4 v59, s[76:79], s62 offen sc1 lds\n\t"
"v_mfma_f32_16x16x32_f16 a[224:227], v[88:91], v[96:99], a[224:227]\n\t"
"s_waitcnt lgkmcnt(3)\n\t"
"v_mfma_f32_16x16x32_f16 a[0:3], v[64:67], v[112:115], a[0:3]\n\t"
"ds_read_b128 v[80:83], v58 offset:53248\n\t"
"s_waitcnt lgkmcnt(3)\n\t"
"v_mfma_f32_16x16x32_f16 a[16:19], v[68:71], v[112:115], a[16:19]\n\t"
"buffer_load_dwordx4 v[216:219], v62, %[srdW], s41 offen\n\t"
"v_mfma_f32_16x16x32_f16 a[20:23], v[68:71], v[116:119], a[20:23]\n\t"
"s_waitcnt vmcnt(32)\n\t"
"v_cvt_pk_f16_f32 v96, v128, v132\n\t"
"v_mfma_f32_16x16x32_f16 a[4:7], v[64:67], v[116:119], a[4:7]\n\t"
"v_cvt_pk_f16_f32 v100, v129, v133\n\t"
"v_mfma_f32_16x16x32_f16 a[8:11], v[64:67], v[120:123], a[8:11]\n\t"
"ds_read_b128 v[84:87], v58 offset:54272\n\t"
"v_cvt_pk_f16_f32 v104, v130, v134\n\t"
"v_mfma_f32_16x16x32_f16 a[24:27], v[68:71], v[120:123], a[24:27]\n\t"
"v_cvt_pk_f16_f32 v108, v131, v135\n\t"
"buffer_load_dwordx4 v[220:223], v63, %[srdW], s41 offen\n\t"
"v_mfma_f32_16x16x32_f16 a[28:31], v[68:71], v[124:127], a[28:31]\n\t"
"v_mfma_f32_16x16x32_f16 a[12:15], v[64:67], v[124:127], a[12:15]\n\t"
"s_waitcnt lgkmcnt(3)\n\t"
"v_mfma_f32_16x16x32_f16 a[44:47], v[72:75], v[124:127], a[44:47]\n\t"
"ds_read_b128 v[88:91], v58 offset:55296\n\t"
"s_add_u32 s62, s46, s57\n\t"
"s_add_u32 s49, s47, s57\n\t"
"s_add_u32 m0, s49, 98304\n\t"
"s_waitcnt lgkmcnt(3)\n\t"
"v_mfma_f32_16x16x32_f16 a[60:63], v[76:79], v[124:127], a[60:63]\n\t"
"buffer_load_dwordx4 v59, s[76:79], s62 offen sc1 lds\n\t"
"v_mfma_f32_16x16x32_f16 a[56:59], v[76:79], v[120:123], a[56:59]\n\t"
"v_mfma_f32_16x16x32_f16 a[40:43], v[72:75], v[120:123], a[40:43]\n\t"
"v_mfma_f32_16x16x32_f16 a[36:39], v[72:75], v[116:119], a[36:39]\n\t"
"ds_read_b128 v[92:95], v58 offset:56320\n\t"
"s_add_u32 s89, s63, 6291456\n\t"
"s_add_u32 s42, s89, s52\n\t"
"v_mfma_f32_16x16x32_f16 a[52:55], v[76:79], v[116:119], a[52:55]\n\t"
"buffer_load_dwordx4 v[224:227], v60, %[srdW], s42 offen\n\t"
"v_mfma_f32_16x16x32_f16 a[48:51], v[76:79], v[112:115], a[48:51]\n\t"
"v_mfma_f32_16x16x32_f16 a[32:35], v[72:75], v[112:115], a[32:35]\n\t"
"s_waitcnt lgkmcnt(3)\n\t"
"v_mfma_f32_16x16x32_f16 a[64:67], v[80:83], v[112:115], a[64:67]\n\t"
"ds_read_b128 v[64:67], v58 offset:57344\n\t"
"s_waitcnt lgkmcnt(3)\n\t"
"v_mfma_f32_16x16x32_f16 a[80:83], v[84:87], v[112:115], a[80:83]\n\t"
"buffer_load_dwordx4 v[228:231], v61, %[srdW], s42 offen\n\t"
"v_mfma_f32_16x16x32_f16 a[84:87], v[84:87], v[116:119], a[84:87]\n\t"
"s_waitcnt vmcnt(33)\n\t"
"v_cvt_pk_f16_f32 v97, v136, v140\n\t"
"v_mfma_f32_16x16x32_f16 a[68:71], v[80:83], v[116:119], a[68:71]\n\t"
"v_cvt_pk_f16_f32 v101, v137, v141\n\t"
"v_mfma_f32_16x16x32_f16 a[72:75], v[80:83], v[120:123], a[72:75]\n\t"
"ds_read_b128 v[68:71], v58 offset:58368\n\t"
"v_cvt_pk_f16_f32 v105, v138, v142\n\t"
"s_add_u32 s62, s46, s58\n\t"
"s_add_u32 s49, s47, s58\n\t"
"s_add_u32 m0, s49, 98304\n\t"
"v_mfma_f32_16x16x32_f16 a[88:91], v[84:87], v[120:123], a[88:91]\n\t"
"v_cvt_pk_f16_f32 v109, v139, v143\n\t"
"buffer_load_dwordx4 v59, s[76:79], s62 offen sc1 lds\n\t"
"v_mfma_f32_16x16x32_f16 a[92:95], v[84:87], v[124:127], a[92:95]\n\t"
"v_mfma_f32_16x16x32_f16 a[76:79], v[80:83], v[124:127], a[76:79]\n\t"
"s_waitcnt lgkmcnt(3)\n\t"
"v_mfma_f32_16x16x32_f16 a[108:111], v[88:91], v[124:127], a[108:111]\n\t"
"ds_read_b128 v[72:75], v58 offset:59392\n\t"
"s_waitcnt lgkmcnt(3)\n\t"
"v_mfma_f32_16x16x32_f16 a[124:127], v[92:95], v[124:127], a[124:127]\n\t"
"buffer_load_dwordx4 v[232:235], v62, %[srdW], s42 offen\n\t"
"v_mfma_f32_16x16x32_f16 a[120:123], v[92:95], v[120:123], a[120:123]\n\t"
"v_mfma_f32_16x16x32_f16 a[104:107], v[88:91], v[120:123], a[104:107]\n\t"
"v_mfma_f32_16x16x32_f16 a[100:103], v[88:91], v[116:119], a[100:103]\n\t"
"ds_read_b128 v[76:79], v58 offset:60416\n\t"
"v_mfma_f32_16x16x32_f16 a[116:119], v[92:95], v[116:119], a[116:119]\n\t"
"buffer_load_dwordx4 v[236:239], v63, %[srdW], s42 offen\n\t"
"v_mfma_f32_16x16x32_f16 a[112:115], v[92:95], v[112:115], a[112:115]\n\t"
"v_mfma_f32_16x16x32_f16 a[96:99], v[88:91], v[112:115], a[96:99]\n\t"
"s_waitcnt lgkmcnt(3)\n\t"
"v_mfma_f32_16x16x32_f16 a[128:131], v[64:67], v[112:115], a[128:131]\n\t"
"ds_read_b128 v[80:83], v58 offset:61440\n\t"
"s_add_u32 s62, s46, s59\n\t"
"s_add_u32 s49, s47, s59\n\t"
"s_add_u32 m0, s49, 98304\n\t"
"s_waitcnt lgkmcnt(3)\n\t"
"v_mfma_f32_16x16x32_f16 a[144:147], v[68:71], v[112:115], a[144:147]\n\t"
"buffer_load_dwordx4 v59, s[76:79], s62 offen sc1 lds\n\t"
"v_mfma_f32_16x16x32_f16 a[148:151], v[68:71], v[116:119], a[148:151]\n\t"
"s_waitcnt vmcnt(34)\n\t"
"v_cvt_pk_f16_f32 v98, v144, v148\n\t"
"v_mfma_f32_16x16x32_f16 a[132:135], v[64:67], v[116:119], a[132:135]\n\t"
"v_cvt_pk_f16_f32 v102, v145, v149\n\t"
"v_mfma_f32_16x16x32_f16 a[136:139], v[64:67], v[120:123], a[136:139]\n\t"
"ds_read_b128 v[84:87], v58 offset:62464\n\t"
"v_cvt_pk_f16_f32 v106, v146, v150\n\t"
"s_add_u32 s89, s63, 6291456\n\t"
"s_add_u32 s43, s89, s53\n\t"
"v_mfma_f32_16x16x32_f16 a[152:155], v[68:71], v[120:123], a[152:155]\n\t"
"v_cvt_pk_f16_f32 v110, v147, v151\n\t"
"buffer_load_dwordx4 v[240:243], v60, %[srdW], s43 offen\n\t"
"v_mfma_f32_16x16x32_f16 a[156:159], v[68:71], v[124:127], a[156:159]\n\t"
"v_mfma_f32_16x16x32_f16 a[140:143], v[64:67], v[124:127], a[140:143]\n\t"
"s_waitcnt lgkmcnt(3)\n\t"
"v_mfma_f32_16x16x32_f16 a[172:175], v[72:75], v[124:127], a[172:175]\n\t"
"ds_read_b128 v[88:91], v58 offset:63488\n\t"
"s_waitcnt lgkmcnt(3)\n\t"
"v_mfma_f32_16x16x32_f16 a[188:191], v[76:79], v[124:127], a[188:191]\n\t"
"buffer_load_dwordx4 v[244:247], v61, %[srdW], s43 offen\n\t"
"v_mfma_f32_16x16x32_f16 a[184:187], v[76:79], v[120:123], a[184:187]\n\t"
"v_mfma_f32_16x16x32_f16 a[168:171], v[72:75], v[120:123], a[168:171]\n\t"
"v_mfma_f32_16x16x32_f16 a[164:167], v[72:75], v[116:119], a[164:167]\n\t"
"ds_read_b128 v[92:95], v58 offset:64512\n\t"
"s_add_u32 s62, s46, s60\n\t"
"s_add_u32 s49, s47, s60\n\t"
"s_add_u32 m0, s49, 98304\n\t"
"v_mfma_f32_16x16x32_f16 a[180:183], v[76:79], v[116:119], a[180:183]\n\t"
"buffer_load_dwordx4 v59, s[76:79], s62 offen sc1 lds\n\t"
"v_mfma_f32_16x16x32_f16 a[176:179], v[76:79], v[112:115], a[176:179]\n\t"
"v_mfma_f32_16x16x32_f16 a[160:163], v[72:75], v[112:115], a[160:163]\n\t"
"s_waitcnt lgkmcnt(3)\n\t"
"v_mfma_f32_16x16x32_f16 a[192:195], v[80:83], v[112:115], a[192:195]\n\t"
"s_waitcnt lgkmcnt(2)\n\t"
"v_mfma_f32_16x16x32_f16 a[208:211], v[84:87], v[112:115], a[208:211]\n\t"
"s_memrealtime s[90:91]\n\t"
"buffer_load_dwordx4 v[248:251], v62, %[srdW], s43 offen\n\t"
"v_mfma_f32_16x16x32_f16 a[212:215], v[84:87], v[116:119], a[212:215]\n\t"
"s_waitcnt vmcnt(35)\n\t"
"v_cvt_pk_f16_f32 v99, v152, v156\n\t"
"v_mfma_f32_16x16x32_f16 a[196:199], v[80:83], v[116:119], a[196:199]\n\t"
"v_cvt_pk_f16_f32 v103, v153, v157\n\t"
"s_waitcnt vmcnt(22) lgkmcnt(0)\n\t"
"s_sub_u32 s89, s90, s65\n\t"
"s_cmp_lt_i32 s89, 0\n\t"
"s_getpc_b64 s[68:69]\n\t"
"s_cbranch_scc1 L_pace_%=\n\t"
"s_ashr_i32 s89, s95, 3\n\t"
"s_add_u32 s65, s90, s89\n\t"
"s_sub_u32 s95, s95, %[sslope]\n\t"
"s_barrier\n\t"
"v_mfma_f32_16x16x32_f16 a[200:203], v[80:83], v[120:123], a[200:203]\n\t"
"ds_read_b128 v[64:67], v57 offset:0\n\t"
"v_cvt_pk_f16_f32 v107, v154, v158\n\t"
"v_mfma_f32_16x16x32_f16 a[216:219], v[84:87], v[120:123], a[216:219]\n\t"
"v_cvt_pk_f16_f32 v111, v155, v159\n\t"
"buffer_load_dwordx4 v[252:255], v63, %[srdW], s43 offen\n\t"
"v_mfma_f32_16x16x32_f16 a[220:223], v[84:87], v[124:127], a[220:223]\n\t"
"v_mfma_f32_16x16x32_f16 a[204:207], v[80:83], v[124:127], a[204:207]\n\t"
"ds_read_b128 v[68:71], v57 offset:1024\n\t"
"s_waitcnt lgkmcnt(3)\n\t"
"v_mfma_f32_16x16x32_f16 a[236:239], v[88:91], v[124:127], a[236:239]\n\t"
"s_add_u32 s62, s46, s61\n\t"
"s_add_u32 s49, s47, s61\n\t"
"s_add_u32 m0, s49, 98304\n\t"
"s_waitcnt lgkmcnt(2)\n\t"
"v_mfma_f32_16x16x32_f16 a[252:255], v[92:95], v[124:127], a[252:255]\n\t"
"buffer_load_dwordx4 v59, s[76:79], s62 offen sc1 lds\n\t"
"v_mfma_f32_16x16x32_f16 a[248:251], v[92:95], v[120:123], a[248:251]\n\t"
"ds_read_b128 v[72:75], v57 offset:2048\n\t"
"v_mfma_f32_16x16x32_f16 a[232:235], v[88:91], v[120:123], a[232:235]\n\t"
"v_mfma_f32_16x16x32_f16 a[228:231], v[88:91], v[116:119], a[228:231]\n\t"
"s_add_u32 s89, s63, 8388608\n\t"
"s_add_u32 s40, s89, s50\n\t"
"v_mfma_f32_16x16x32_f16 a[244:247], v[92:95], v[116:119], a[244:247]\n\t"
"ds_read_b128 v[76:79], v57 offset:3072\n\t"
"buffer_load_dwordx4 v[128:131], v60, %[srdW], s40 offen\n\t"
"s_sub_u32 s80, s80, 1\n\t"
"s_getpc_b64 s[68:69]\n\t"
"s_cbranch_scc1 L_hnd_%=\n\t"
"v_mfma_f32_16x16x32_f16 a[240:243], v[92:95], v[112:115], a[240:243]\n\t"
"v_mfma_f32_16x16x32_f16 a[224:227], v[88:91], v[112:115], a[224:227]\n\t"
"s_waitcnt lgkmcnt(3)\n\t"
"v_mfma_f32_16x16x32_f16 a[0:3], v[64:67], v[96:99], a[0:3]\n\t"
"ds_read_b128 v[80:83], v57 offset:4096\n\t"
"buffer_load_dwordx4 v[132:135], v61, %[srdW], s40 offen\n\t"
"s_waitcnt lgkmcnt(3)\n\t"
"v_mfma_f32_16x16x32_f16 a[16:19], v[68:71], v[96:99], a[16:19]\n\t"
"v_mfma_f32_16x16x32_f16 a[20:23], v[68:71], v[100:103], a[20:23]\n\t"
"s_waitcnt vmcnt(36)\n\t"
"v_cvt_pk_f16_f32 v112, v160, v164\n\t"
"v_mfma_f32_16x16x32_f16 a[4:7], v[64:67], v[100:103], a[4:7]\n\t"
"v_cvt_pk_f16_f32 v116, v161, v165\n\t"
"s_add_u32 s46, s64, 131072\n\t"
"s_add_u32 s62, s46, s54\n\t"
"s_add_u32 s49, s47, s54\n\t"
"s_add_u32 m0, s49, 0\n\t"
"v_mfma_f32_16x16x32_f16 a[8:11], v[64:67], v[104:107], a[8:11]\n\t"
"ds_read_b128 v[84:87], v57 offset:5120\n\t"
"v_cvt_pk_f16_f32 v120, v162, v166\n\t"
"buffer_load_dwordx4 v59, s[76:79], s62 offen sc1 lds\n\t"
"v_mfma_f32_16x16x32_f16 a[24:27], v[68:71], v[104:107], a[24:27]\n\t"
"v_cvt_pk_f16_f32 v124, v163, v167\n\t"
"v_mfma_f32_16x16x32_f16 a[28:31], v[68:71], v[108:111], a[28:31]\n\t"
"v_mfma_f32_16x16x32_f16 a[12:15], v[64:67], v[108:111], a[12:15]\n\t"
"s_waitcnt lgkmcnt(3)\n\t"
"v_mfma_f32_16x16x32_f16 a[44:47], v[72:75], v[108:111], a[44:47]\n\t"
"ds_read_b128 v[88:91], v57 offset:6144\n\t"
"buffer_load_dwordx4 v[136:139], v62, %[srdW], s40 offen\n\t"
"s_waitcnt lgkmcnt(3)\n\t"
"v_mfma_f32_16x16x32_f16 a[60:63], v[76:79], v[108:111], a[60:63]\n\t"
"v_mfma_f32_16x16x32_f16 a[56:59], v[76:79], v[104:107], a[56:59]\n\t"
"v_mfma_f32_16x16x32_f16 a[40:43], v[72:75], v[104:107], a[40:43]\n\t"
"v_mfma_f32_16x16x32_f16 a[36:39], v[72:75], v[100:103], a[36:39]\n\t"
"ds_read_b128 v[92:95], v57 offset:7168\n\t"
"buffer_load_dwordx4 v[140:143], v63, %[srdW], s40 offen\n\t"
"v_mfma_f32_16x16x32_f16 a[52:55], v[76:79], v[100:103], a[52:55]\n\t"
"v_mfma_f32_16x16x32_f16 a[48:51], v[76:79], v[96:99], a[48:51]\n\t"
"v_mfma_f32_16x16x32_f16 a[32:35], v[72:75], v[96:99], a[32:35]\n\t"
"s_add_u32 s62, s46, s55\n\t"
"s_add_u32 s49, s47, s55\n\t"
"s_add_u32 m0, s49, 0\n\t"
"s_waitcnt lgkmcnt(3)\n\t"
"v_mfma_f32_16x16x32_f16 a[64:67], v[80:83], v[96:99], a[64:67]\n\t"
"ds_read_b128 v[64:67], v57 offset:8192\n\t"
"buffer_load_dwordx4 v59, s[76:79], s62 offen sc1 lds\n\t"
"s_waitcnt lgkmcnt(3)\n\t"
"v_mfma_f32_16x16x32_f16 a[80:83], v[84:87], v[96:99], a[80:83]\n\t"
"v_mfma_f32_16x16x32_f16 a[84:87], v[84:87], v[100:103], a[84:87]\n\t"
"s_waitcnt vmcnt(37)\n\t"
"v_cvt_pk_f16_f32 v113, v168, v172\n\t"
"v_mfma_f32_16x16x32_f16 a[68:71], v[80:83], v[100:103], a[68:71]\n\t"
"v_cvt_pk_f16_f32 v117, v169, v173\n\t"
"s_add_u32 s89, s63, 8388608\n\t"
"s_add_u32 s41, s89, s51\n\t"
"v_mfma_f32_16x16x32_f16 a[72:75], v[80:83], v[104:107], a[72:75]\n\t"
"ds_read_b128 v[68:71], v57 offset:9216\n\t"
"v_cvt_pk_f16_f32 v121, v170, v174\n\t"
"buffer_load_dwordx4 v[144:147], v60, %[srdW], s41 offen\n\t"
"v_mfma_f32_16x16x32_f16 a[88:91], v[84:87], v[104:107], a[88:91]\n\t"
"v_cvt_pk_f16_f32 v125, v171, v175\n\t"
"v_mfma_f32_16x16x32_f16 a[92:95], v[84:87], v[108:111], a[92:95]\n\t"
"v_mfma_f32_16x16x32_f16 a[76:79], v[80:83], v[108:111], a[76:79]\n\t"
"s_waitcnt lgkmcnt(3)\n\t"
"v_mfma_f32_16x16x32_f16 a[108:111], v[88:91], v[108:111], a[108:111]\n\t"
"ds_read_b128 v[72:75], v57 offset:10240\n\t"
"buffer_load_dwordx4 v[148:151], v61, %[srdW], s41 offen\n\t"
"s_waitcnt lgkmcnt(3)\n\t"
"v_mfma_f32_16x16x32_f16 a[124:127], v[92:95], v[108:111], a[124:127]\n\t"
"v_mfma_f32_16x16x32_f16 a[120:123], v[92:95], v[104:107], a[120:123]\n\t"
"v_mfma_f32_16x16x32_f16 a[104:107], v[88:91], v[104:107], a[104:107]\n\t"
"s_add_u32 s62, s46, s56\n\t"
"s_add_u32 s49, s47, s56\n\t"
"s_add_u32 m0, s49, 0\n\t"
"v_mfma_f32_16x16x32_f16 a[100:103], v[88:91], v[100:103], a[100:103]\n\t"
"ds_read_b128 v[76:79], v57 offset:11264\n\t"
"buffer_load_dwordx4 v59, s[76:79], s62 offen sc1 lds\n\t"
"v_mfma_f32_16x16x32_f16 a[116:119], v[92:95], v[100:103], a[116:119]\n\t"
"v_mfma_f32_16x16x32_f16 a[112:115], v[92:95], v[96:99], a[112:115]\n\t"
"v_mfma_f32_16x16x32_f16 a[96:99], v[88:91], v[96:99], a[96:99]\n\t"
"s_waitcnt lgkmcnt(3)\n\t"
"v_mfma_f32_16x16x32_f16 a[128:131], v[64:67], v[96:99], a[128:131]\n\t"
"ds_read_b128 v[80:83], v57 offset:12288\n\t"
"buffer_load_dwordx4 v[152:155], v62, %[srdW], s41 offen\n\t"
"s_waitcnt lgkmcnt(3)\n\t"
"v_mfma_f32_16x16x32_f16 a[144:147], v[68:71], v[96:99], a[144:147]\n\t"
"v_mfma_f32_16x16x32_f16 a[148:151], v[68:71], v[100:103], a[148:151]\n\t"
"s_waitcnt vmcnt(38)\n\t"
"v_cvt_pk_f16_f32 v114, v176, v180\n\t"
"v_mfma_f32_16x16x32_f16 a[132:135], v[64:67], v[100:103], a[132:135]\n\t"
"v_cvt_pk_f16_f32 v118, v177, v181\n\t"
"v_mfma_f32_16x16x32_f16 a[136:139], v[64:67], v[104:107], a[136:139]\n\t"
"ds_read_b128 v[84:87], v57 offset:13312\n\t"
"v_cvt_pk_f16_f32 v122, v178, v182\n\t"
"buffer_load_dwordx4 v[156:159], v63, %[srdW], s41 offen\n\t"
"v_mfma_f32_16x16x32_f16 a[152:155], v[68:71], v[104:107], a[152:155]\n\t"
"v_cvt_pk_f16_f32 v126, v179, v183\n\t"
"v_mfma_f32_16x16x32_f16 a[156:159], v[68:71], v[108:111], a[156:159]\n\t"
"v_mfma_f32_16x16x32_f16 a[140:143], v[64:67], v[108:111], a[140:143]\n\t"
"s_add_u32 s62, s46, s57\n\t"
"s_add_u32 s49, s47, s57\n\t"
"s_add_u32 m0, s49, 0\n\t"
"s_waitcnt lgkmcnt(3)\n\t"
"v_mfma_f32_16x16x32_f16 a[172:175], v[72:75], v[108:111], a[172:175]\n\t"
"ds_read_b128 v[88:91], v57 offset:14336\n\t"
"buffer_load_dwordx4 v59, s[76:79], s62 offen sc1 lds\n\t"
"s_waitcnt lgkmcnt(3)\n\t"
"v_mfma_f32_16x16x32_f16 a[188:191], v[76:79], v[108:111], a[188:191]\n\t"
"v_mfma_f32_16x16x32_f16 a[184:187], v[76:79], v[104:107], a[184:187]\n\t"
"v_mfma_f32_16x16x32_f16 a[168:171], v[72:75], v[104:107], a[168:171]\n\t"
"s_add_u32 s89, s63, 8388608\n\t"
"s_add_u32 s42, s89, s52\n\t"
"v_mfma_f32_16x16x32_f16 a[164:167], v[72:75], v[100:103], a[164:167]\n\t"
"ds_read_b128 v[92:95], v57 offset:15360\n\t"
"buffer_load_dwordx4 v[160:163], v60, %[srdW], s42 offen\n\t"
"v_mfma_f32_16x16x32_f16 a[180:183], v[76:79], v[100:103], a[180:183]\n\t"
"v_mfma_f32_16x16x32_f16 a[176:179], v[76:79], v[96:99], a[176:179]\n\t"
"v_mfma_f32_16x16x32_f16 a[160:163], v[72:75], v[96:99], a[160:163]\n\t"
"s_waitcnt lgkmcnt(3)\n\t"
"v_mfma_f32_16x16x32_f16 a[192:195], v[80:83], v[96:99], a[192:195]\n\t"
"ds_read_b128 v[64:67], v57 offset:16384\n\t"
"buffer_load_dwordx4 v[164:167], v61, %[srdW], s42 offen\n\t"
"s_waitcnt lgkmcnt(3)\n\t"
"v_mfma_f32_16x16x32_f16 a[208:211], v[84:87], v[96:99], a[208:211]\n\t"
"v_mfma_f32_16x16x32_f16 a[212:215], v[84:87], v[100:103], a[212:215]\n\t"
"s_waitcnt vmcnt(39)\n\t"
"v_cvt_pk_f16_f32 v115, v184, v188\n\t"
"v_mfma_f32_16x16x32_f16 a[196:199], v[80:83], v[100:103], a[196:199]\n\t"
"v_cvt_pk_f16_f32 v119, v185, v189\n\t"
"s_add_u32 s62, s46, s58\n\t"
"s_add_u32 s49, s47, s58\n\t"
"s_add_u32 m0, s49, 0\n\t"
"v_mfma_f32_16x16x32_f16 a[200:203], v[80:83], v[104:107], a[200:203]\n\t"
"ds_read_b128 v[68:71], v57 offset:17408\n\t"
"v_cvt_pk_f16_f32 v123, v186, v190\n\t"
"buffer_load_dwordx4 v59, s[76:79], s62 offen sc1 lds\n\t"
"v_mfma_f32_16x16x32_f16 a[216:219], v[84:87], v[104:107], a[216:219]\n\t"
"v_cvt_pk_f16_f32 v127, v187, v191\n\t"
"v_mfma_f32_16x16x32_f16 a[220:223], v[84:87], v[108:111], a[220:223]\n\t"
"v_mfma_f32_16x16x32_f16 a[204:207], v[80:83], v[108:111], a[204:207]\n\t"
"s_waitcnt lgkmcnt(3)\n\t"
"v_mfma_f32_16x16x32_f16 a[236:239], v[88:91], v[108:111], a[236:239]\n\t"
"ds_read_b128 v[72:75], v57 offset:18432\n\t"
"buffer_load_dwordx4 v[168:171], v62, %[srdW], s42 offen\n\t"
"s_waitcnt lgkmcnt(3)\n\t"
"v_mfma_f32_16x16x32_f16 a[252:255], v[92:95], v[108:111], a[252:255]\n\t"
"v_mfma_f32_16x16x32_f16 a[248:251], v[92:95], v[104:107], a[248:251]\n\t"
"v_mfma_f32_16x16x32_f16 a[232:235], v[88:91], v[104:107], a[232:235]\n\t"
"v_mfma_f32_16x16x32_f16 a[228:231], v[88:91], v[100:103], a[228:231]\n\t"
"ds_read_b128 v[76:79], v57 offset:19456\n\t"
"buffer_load_dwordx4 v[172:175], v63, %[srdW], s42 offen\n\t"
"v_mfma_f32_16x16x32_f16 a[244:247], v[92:95], v[100:103], a[244:247]\n\t"
"s_sub_u32 s80, s80, 1\n\t"
"s_getpc_b64 s[68:69]\n\t"
"s_cbranch_scc1 L_hnd_%=\n\t"
"v_mfma_f32_16x16x32_f16 a[240:243], v[92:95], v[96:99], a[240:243]\n\t"
"s_add_u32 s62, s46, s59\n\t"
"s_add_u32 s49, s47, s59\n\t"
"s_add_u32 m0, s49, 0\n\t"
"v_mfma_f32_16x16x32_f16 a[224:227], v[88:91], v[96:99], a[224:227]\n\t"
"buffer_load_dwordx4 v59, s[76:79], s62 offen sc1 lds\n\t"
"s_waitcnt lgkmcnt(3)\n\t"
"v_mfma_f32_16x16x32_f16 a[0:3], v[64:67], v[112:115], a[0:3]\n\t"
"ds_read_b128 v[80:83], v57 offset:20480\n\t"
"s_waitcnt lgkmcnt(3)\n\t"
"v_mfma_f32_16x16x32_f16 a[16:19], v[68:71], v[112:115], a[16:19]\n\t"
"v_mfma_f32_16x16x32_f16 a[20:23], v[68:71], v[116:119], a[20:23]\n\t"
"s_waitcnt vmcnt(40)\n\t"
"v_cvt_pk_f16_f32 v96, v192, v196\n\t"
"s_add_u32 s89, s63, 8388608\n\t"
"s_add_u32 s43, s89, s53\n\t"
"v_mfma_f32_16x16x32_f16 a[4:7], v[64:67], v[116:119], a[4:7]\n\t"
"v_cvt_pk_f16_f32 v100, v193, v197\n\t"
"buffer_load_dwordx4 v[176:179], v60, %[srdW], s43 offen\n\t"
"v_mfma_f32_16x16x32_f16 a[8:11], v[64:67], v[120:123], a[8:11]\n\t"
"ds_read_b128 v[84:87], v57 offset:21504\n\t"
"v_cvt_pk_f16_f32 v104, v194, v198\n\t"
"v_mfma_f32_16x16x32_f16 a[24:27], v[68:71], v[120:123], a[24:27]\n\t"
"v_cvt_pk_f16_f32 v108, v195, v199\n\t"
"v_mfma_f32_16x16x32_f16 a[28:31], v[68:71], v[124:127], a[28:31]\n\t"
"v_mfma_f32_16x16x32_f16 a[12:15], v[64:67], v[124:127], a[12:15]\n\t"
"buffer_load_dwordx4 v[180:183], v61, %[srdW], s43 offen\n\t"
"s_waitcnt lgkmcnt(3)\n\t"
"v_mfma_f32_16x16x32_f16 a[44:47], v[72:75], v[124:127], a[44:47]\n\t"
"ds_read_b128 v[88:91], v57 offset:22528\n\t"
"s_waitcnt lgkmcnt(3)\n\t"
"v_mfma_f32_16x16x32_f16 a[60:63], v[76:79], v[124:127], a[60:63]\n\t"
"v_mfma_f32_16x16x32_f16 a[56:59], v[76:79], v[120:123], a[56:59]\n\t"
"s_add_u32 s62, s46, s60\n\t"
"s_add_u32 s49, s47, s60\n\t"
"s_add_u32 m0, s49, 0\n\t"
"v_mfma_f32_16x16x32_f16 a[40:43], v[72:75], v[120:123], a[40:43]\n\t"
"buffer_load_dwordx4 v59, s[76:79], s62 offen sc1 lds\n\t"
"v_mfma_f32_16x16x32_f16 a[36:39], v[72:75], v[116:119], a[36:39]\n\t"
"ds_read_b128 v[92:95], v57 offset:23552\n\t"
"v_mfma_f32_16x16x32_f16 a[52:55], v[76:79], v[116:119], a[52:55]\n\t"
"v_mfma_f32_16x16x32_f16 a[48:51], v[76:79], v[112:115], a[48:51]\n\t"
"v_mfma_f32_16x16x32_f16 a[32:35], v[72:75], v[112:115], a[32:35]\n\t"
"buffer_load_dwordx4 v[184:187], v62, %[srdW], s43 offen\n\t"
"s_waitcnt lgkmcnt(3)\n\t"
"v_mfma_f32_16x16x32_f16 a[64:67], v[80:83], v[112:115], a[64:67]\n\t"
"ds_read_b128 v[64:67], v57 offset:24576\n\t"
"s_waitcnt lgkmcnt(3)\n\t"
"v_mfma_f32_16x16x32_f16 a[80:83], v[84:87], v[112:115], a[80:83]\n\t"
"v_mfma_f32_16x16x32_f16 a[84:87], v[84:87], v[116:119], a[84:87]\n\t"
"s_waitcnt vmcnt(41)\n\t"
"v_cvt_pk_f16_f32 v97, v200, v204\n\t"
"v_mfma_f32_16x16x32_f16 a[68:71], v[80:83], v[116:119], a[68:71]\n\t"
"v_cvt_pk_f16_f32 v101, v201, v205\n\t"
"buffer_load_dwordx4 v[188:191], v63, %[srdW], s43 offen\n\t"
"v_mfma_f32_16x16x32_f16 a[72:75], v[80:83], v[120:123], a[72:75]\n\t"
"ds_read_b128 v[68:71], v57 offset:25600\n\t"
"v_cvt_pk_f16_f32 v105, v202, v206\n\t"
"v_mfma_f32_16x16x32_f16 a[88:91], v[84:87], v[120:123], a[88:91]\n\t"
"v_cvt_pk_f16_f32 v109, v203, v207\n\t"
"v_mfma_f32_16x16x32_f16 a[92:95], v[84:87], v[124:127], a[92:95]\n\t"
"s_add_u32 s62, s46, s61\n\t"
"s_add_u32 s49, s47, s61\n\t"
"s_add_u32 m0, s49, 0\n\t"
"v_mfma_f32_16x16x32_f16 a[76:79], v[80:83], v[124:127], a[76:79]\n\t"
"buffer_load_dwordx4 v59, s[76:79], s62 offen sc1 lds\n\t"
"s_waitcnt lgkmcnt(3)\n\t"
"v_mfma_f32_16x16x32_f16 a[108:111], v[88:91], v[124:127], a[108:111]\n\t"
"ds_read_b128 v[72:75], v57 offset:26624\n\t"
"s_waitcnt lgkmcnt(3)\n\t"
"v_mfma_f32_16x16x32_f16 a[124:127], v[92:95], v[124:127], a[124:127]\n\t"
"v_mfma_f32_16x16x32_f16 a[120:123], v[92:95], v[120:123], a[120:123]\n\t"
"s_add_u32 s89, s63, 10485760\n\t"
"s_add_u32 s40, s89, s50\n\t"
"v_mfma_f32_16x16x32_f16 a[104:107], v[88:91], v[120:123], a[104:107]\n\t"
"buffer_load_dwordx4 v[192:195], v60, %[srdW], s40 offen\n\t"
"v_mfma_f32_16x16x32_f16 a[100:103], v[88:91], v[116:119], a[100:103]\n\t"
"ds_read_b128 v[76:79], v57 offset:27648\n\t"
"v_mfma_f32_16x16x32_f16 a[116:119], v[92:95], v[116:119], a[116:119]\n\t"
"v_mfma_f32_16x16x32_f16 a[112:115], v[92:95], v[112:115], a[112:115]\n\t"
"v_mfma_f32_16x16x32_f16 a[96:99], v[88:91], v[112:115], a[96:99]\n\t"
"buffer_load_dwordx4 v[196:199], v61, %[srdW], s40 offen\n\t"
"s_waitcnt lgkmcnt(3)\n\t"
"v_mfma_f32_16x16x32_f16 a[128:131], v[64:67], v[112:115], a[128:131]\n\t"
"ds_read_b128 v[80:83], v57 offset:28672\n\t"
"s_waitcnt lgkmcnt(3)\n\t"
"v_mfma_f32_16x16x32_f16 a[144:147], v[68:71], v[112:115], a[144:147]\n\t"
"v_mfma_f32_16x16x32_f16 a[148:151], v[68:71], v[116:119], a[148:151]\n\t"
"s_waitcnt vmcnt(42)\n\t"
"v_cvt_pk_f16_f32 v98, v208, v212\n\t"
"s_add_u32 s46, s64, 163840\n\t"
"s_add_u32 s62, s46, s54\n\t"
"s_add_u32 s49, s47, s54\n\t"
"s_add_u32 m0, s49, 32768\n\t"
"v_mfma_f32_16x16x32_f16 a[132:135], v[64:67], v[116:119], a[132:135]\n\t"
"v_cvt_pk_f16_f32 v102, v209, v213\n\t"
"buffer_load_dwordx4 v59, s[76:79], s62 offen sc1 lds\n\t"
"v_mfma_f32_16x16x32_f16 a[136:139], v[64:67], v[120:123], a[136:139]\n\t"
"ds_read_b128 v[84:87], v57 offset:29696\n\t"
"v_cvt_pk_f16_f32 v106, v210, v214\n\t"
"v_mfma_f32_16x16x32_f16 a[152:155], v[68:71], v[120:123], a[152:155]\n\t"
"v_cvt_pk_f16_f32 v110, v211, v215\n\t"
"v_mfma_f32_16x16x32_f16 a[156:159], v[68:71], v[124:127], a[156:159]\n\t"
"v_mfma_f32_16x16x32_f16 a[140:143], v[64:67], v[124:127], a[140:143]\n\t"
"buffer_load_dwordx4 v[200:203], v62, %[srdW], s40 offen\n\t"
"s_waitcnt lgkmcnt(3)\n\t"
"v_mfma_f32_16x16x32_f16 a[172:175], v[72:75], v[124:127], a[172:175]\n\t"
"ds_read_b128 v[88:91], v57 offset:30720\n\t"
"s_waitcnt lgkmcnt(3)\n\t"
"v_mfma_f32_16x16x32_f16 a[188:191], v[76:79], v[124:127], a[188:191]\n\t"
"v_mfma_f32_16x16x32_f16 a[184:187], v[76:79], v[120:123], a[184:187]\n\t"
"v_mfma_f32_16x16x32_f16 a[168:171], v[72:75], v[120:123], a[168:171]\n\t"
"buffer_load_dwordx4 v[204:207], v63, %[srdW], s40 offen\n\t"
"v_mfma_f32_16x16x32_f16 a[164:167], v[72:75], v[116:119], a[164:167]\n\t"
"ds_read_b128 v[92:95], v57 offset:31744\n\t"
"v_mfma_f32_16x16x32_f16 a[180:183], v[76:79], v[116:119], a[180:183]\n\t"
"v_mfma_f32_16x16x32_f16 a[176:179], v[76:79], v[112:115], a[176:179]\n\t"
"s_add_u32 s62, s46, s55\n\t"
"s_add_u32 s49, s47, s55\n\t"
"s_add_u32 m0, s49, 32768\n\t"
"v_mfma_f32_16x16x32_f16 a[160:163], v[72:75], v[112:115], a[160:163]\n\t"
"buffer_load_dwordx4 v59, s[76:79], s62 offen sc1 lds\n\t"
"s_waitcnt lgkmcnt(3)\n\t"
"v_mfma_f32_16x16x32_f16 a[192:195], v[80:83], v[112:115], a[192:195]\n\t"
"s_waitcnt lgkmcnt(2)\n\t"
"v_mfma_f32_16x16x32_f16 a[208:211], v[84:87], v[112:115], a[208:211]\n\t"
"s_memrealtime s[90:91]\n\t"
"v_mfma_f32_16x16x32_f16 a[212:215], v[84:87], v[116:119], a[212:215]\n\t"
"s_waitcnt vmcnt(43)\n\t"
"v_cvt_pk_f16_f32 v99, v216, v220\n\t"
"s_add_u32 s89, s63, 10485760\n\t"
"s_add_u32 s41, s89, s51\n\t"
"v_mfma_f32_16x16x32_f16 a[196:199], v[80:83], v[116:119], a[196:199]\n\t"
"v_cvt_pk_f16_f32 v103, v217, v221\n\t"
"buffer_load_dwordx4 v[208:211], v60, %[srdW], s41 offen\n\t"
"s_waitcnt vmcnt(31) lgkmcnt(0)\n\t"
"s_sub_u32 s89, s90, s65\n\t"
"s_cmp_lt_i32 s89, 0\n\t"
"s_getpc_b64 s[68:69]\n\t"
"s_cbranch_scc1 L_pace_%=\n\t"
"s_ashr_i32 s89, s95, 3\n\t"
"s_add_u32 s65, s90, s89\n\t"
"s_sub_u32 s95, s95, %[sslope]\n\t"
"s_barrier\n\t"
"v_mfma_f32_16x16x32_f16 a[200:203], v[80:83], v[120:123], a[200:203]\n\t"
"ds_read_b128 v[64:67], v57 offset:32768\n\t"
"v_cvt_pk_f16_f32 v107, v218, v222\n\t"
"v_mfma_f32_16x16x32_f16 a[216:219], v[84:87], v[120:123], a[216:219]\n\t"
"v_cvt_pk_f16_f32 v111, v219, v223\n\t"
"v_mfma_f32_16x16x32_f16 a[220:223], v[84:87], v[124:127], a[220:223]\n\t"
"v_mfma_f32_16x16x32_f16 a[204:207], v[80:83], v[124:127], a[204:207]\n\t"
"ds_read_b128 v[68:71], v57 offset:33792\n\t"
"buffer_load_dwordx4 v[212:215], v61, %[srdW], s41 offen\n\t"
"s_waitcnt lgkmcnt(3)\n\t"
"v_mfma_f32_16x16x32_f16 a[236:239], v[88:91], v[124:127], a[236:239]\n\t"
"s_waitcnt lgkmcnt(2)\n\t"
"v_mfma_f32_16x16x32_f16 a[252:255], v[92:95], v[124:127], a[252:255]\n\t"
"v_mfma_f32_16x16x32_f16 a[248:251], v[92:95], v[120:123], a[248:251]\n\t"
"ds_read_b128 v[72:75], v57 offset:34816\n\t"
"s_add_u32 s62, s46, s56\n\t"
"s_add_u32 s49, s47, s56\n\t"
"s_add_u32 m0, s49, 32768\n\t"
"v_mfma_f32_16x16x32_f16 a[232:235], v[88:91], v[120:123], a[232:235]\n\t"
"buffer_load_dwordx4 v59, s[76:79], s62 offen sc1 lds\n\t"
"v_mfma_f32_16x16x32_f16 a[228:231], v[88:91], v[116:119], a[228:231]\n\t"
"v_mfma_f32_16x16x32_f16 a[244:247], v[92:95], v[116:119], a[244:247]\n\t"
"ds_read_b128 v[76:79], v57 offset:35840\n\t"
"s_sub_u32 s80, s80, 1\n\t"
"s_getpc_b64 s[68:69]\n\t"
"s_cbranch_scc1 L_hnd_%=\n\t"
"v_mfma_f32_16x16x32_f16 a[240:243], v[92:95], v[112:115], a[240:243]\n\t"
"buffer_load_dwordx4 v[216:219], v62, %[srdW], s41 offen\n\t"
"v_mfma_f32_16x16x32_f16 a[224:227], v[88:91], v[112:115], a[224:227]\n\t"
"s_waitcnt lgkmcnt(3)\n\t"
"v_mfma_f32_16x16x32_f16 a[0:3], v[64:67], v[96:99], a[0:3]\n\t"
"ds_read_b128 v[80:83], v57 offset:36864\n\t"
"s_waitcnt lgkmcnt(3)\n\t"
"v_mfma_f32_16x16x32_f16 a[16:19], v[68:71], v[96:99], a[16:19]\n\t"
"v_mfma_f32_16x16x32_f16 a[20:23], v[68:71], v[100:103], a[20:23]\n\t"
"s_waitcnt vmcnt(44)\n\t"
"v_cvt_pk_f16_f32 v112, v224, v228\n\t"
"buffer_load_dwordx4 v[220:223], v63, %[srdW], s41 offen\n\t"
"v_mfma_f32_16x16x32_f16 a[4:7], v[64:67], v[100:103], a[4:7]\n\t"
"v_cvt_pk_f16_f32 v116, v225, v229\n\t"
"v_mfma_f32_16x16x32_f16 a[8:11], v[64:67], v[104:107], a[8:11]\n\t"
"ds_read_b128 v[84:87], v57 offset:37888\n\t"
"v_cvt_pk_f16_f32 v120, v226, v230\n\t"
"v_mfma_f32_16x16x32_f16 a[24:27], v[68:71], v[104:107], a[24:27]\n\t"
"v_cvt_pk_f16_f32 v124, v227, v231\n\t"
"s_add_u32 s62, s46, s57\n\t"
"s_add_u32 s49, s47, s57\n\t"
"s_add_u32 m0, s49, 32768\n\t"
"v_mfma_f32_16x16x32_f16 a[28:31], v[68:71], v[108:111], a[28:31]\n\t"
"buffer_load_dwordx4 v59, s[76:79], s62 offen sc1 lds\n\t"
"v_mfma_f32_16x16x32_f16 a[12:15], v[64:67], v[108:111], a[12:15]\n\t"
"buffer_load_dword v43, off, %[srdC], s70 sc1\n\t"
"s_waitcnt lgkmcnt(3)\n\t"
"v_mfma_f32_16x16x32_f16 a[44:47], v[72:75], v[108:111], a[44:47]\n\t"
"ds_read_b128 v[88:91], v57 offset:38912\n\t"
"s_waitcnt lgkmcnt(3)\n\t"
"v_mfma_f32_16x16x32_f16 a[60:63], v[76:79], v[108:111], a[60:63]\n\t"
"s_add_u32 s89, s63, 10485760\n\t"
"s_add_u32 s42, s89, s52\n\t"
"v_mfma_f32_16x16x32_f16 a[56:59], v[76:79], v[104:107], a[56:59]\n\t"
"buffer_load_dwordx4 v[224:227], v60, %[srdW], s42 offen\n\t"
"v_mfma_f32_16x16x32_f16 a[40:43], v[72:75], v[104:107], a[40:43]\n\t"
"v_mfma_f32_16x16x32_f16 a[36:39], v[72:75], v[100:103], a[36:39]\n\t"
"ds_read_b128 v[92:95], v57 offset:39936\n\t"
"v_mfma_f32_16x16x32_f16 a[52:55], v[76:79], v[100:103], a[52:55]\n\t"
"v_mfma_f32_16x16x32_f16 a[48:51], v[76:79], v[96:99], a[48:51]\n\t"
"buffer_load_dwordx4 v[228:231], v61, %[srdW], s42 offen\n\t"
"v_mfma_f32_16x16x32_f16 a[32:35], v[72:75], v[96:99], a[32:35]\n\t"
"s_waitcnt lgkmcnt(3)\n\t"
"v_mfma_f32_16x16x32_f16 a[64:67], v[80:83], v[96:99], a[64:67]\n\t"
"ds_read_b128 v[64:67], v57 offset:40960\n\t"
"s_waitcnt lgkmcnt(3)\n\t"
"v_mfma_f32_16x16x32_f16 a[80:83], v[84:87], v[96:99], a[80:83]\n\t"
"s_add_u32 s62, s46, s58\n\t"
"s_add_u32 s49, s47, s58\n\t"
"s_add_u32 m0, s49, 32768\n\t"
"v_mfma_f32_16x16x32_f16 a[84:87], v[84:87], v[100:103], a[84:87]\n\t"
"s_waitcnt vmcnt(46)\n\t"
"v_cvt_pk_f16_f32 v113, v232, v236\n\t"
"buffer_load_dwordx4 v59, s[76:79], s62 offen sc1 lds\n\t"
"v_mfma_f32_16x16x32_f16 a[68:71], v[80:83], v[100:103], a[68:71]\n\t"
"v_cvt_pk_f16_f32 v117, v233, v237\n\t"
"v_mfma_f32_16x16x32_f16 a[72:75], v[80:83], v[104:107], a[72:75]\n\t"
"ds_read_b128 v[68:71], v57 offset:41984\n\t"
"v_cvt_pk_f16_f32 v121, v234, v238\n\t"
"v_mfma_f32_16x16x32_f16 a[88:91], v[84:87], v[104:107], a[88:91]\n\t"
"v_cvt_pk_f16_f32 v125, v235, v239\n\t"
"v_mfma_f32_16x16x32_f16 a[92:95], v[84:87], v[108:111], a[92:95]\n\t"
"buffer_load_dwordx4 v[232:235], v62, %[srdW], s42 offen\n\t"
"v_mfma_f32_16x16x32_f16 a[76:79], v[80:83], v[108:111], a[76:79]\n\t"
"s_waitcnt lgkmcnt(3)\n\t"
"v_mfma_f32_16x16x32_f16 a[108:111], v[88:91], v[108:111], a[108:111]\n\t"
"ds_read_b128 v[72:75], v57 offset:43008\n\t"
"s_waitcnt lgkmcnt(3)\n\t"
"v_mfma_f32_16x16x32_f16 a[124:127], v[92:95], v[108:111], a[124:127]\n\t"
"v_mfma_f32_16x16x32_f16 a[120:123], v[92:95], v[104:107], a[120:123]\n\t"
"buffer_load_dwordx4 v[236:239], v63, %[srdW], s42 offen\n\t"
"v_mfma_f32_16x16x32_f16 a[104:107], v[88:91], v[104:107], a[104:107]\n\t"
"v_mfma_f32_16x16x32_f16 a[100:103], v[88:91], v[100:103], a[100:103]\n\t"
"ds_read_b128 v[76:79], v57 offset:44032\n\t"
"v_mfma_f32_16x16x32_f16 a[116:119], v[92:95], v[100:103], a[116:119]\n\t"
"s_add_u32 s62, s46, s59\n\t"
"s_add_u32 s49, s47, s59\n\t"
"s_add_u32 m0, s49, 32768\n\t"
"v_mfma_f32_16x16x32_f16 a[112:115], v[92:95], v[96:99], a[112:115]\n\t"
"buffer_load_dwordx4 v59, s[76:79], s62 offen sc1 lds\n\t"
"v_mfma_f32_16x16x32_f16 a[96:99], v[88:91], v[96:99], a[96:99]\n\t"
"s_waitcnt lgkmcnt(3)\n\t"
"v_mfma_f32_16x16x32_f16 a[128:131], v[64:67], v[96:99], a[128:131]\n\t"
"ds_read_b128 v[80:83], v57 offset:45056\n\t"
"s_waitcnt lgkmcnt(3)\n\t"
"v_mfma_f32_16x16x32_f16 a[144:147], v[68:71], v[96:99], a[144:147]\n\t"
"v_mfma_f32_16x16x32_f16 a[148:151], v[68:71], v[100:103], a[148:151]\n\t"
"s_waitcnt vmcnt(47)\n\t"
"v_cvt_pk_f16_f32 v114, v240, v244\n\t"
"v_mfma_f32_16x16x32_f16 a[132:135], v[64:67], v[100:103], a[132:135]\n\t"
"v_cvt_pk_f16_f32 v118, v241, v245\n\t"
"v_mfma_f32_16x16x32_f16 a[136:139], v[64:67], v[104:107], a[136:139]\n\t"
"ds_read_b128 v[84:87], v57 offset:46080\n\t"
"v_cvt_pk_f16_f32 v122, v242, v246\n\t"
"v_mfma_f32_16x16x32_f16 a[152:155], v[68:71], v[104:107], a[152:155]\n\t"
"v_cvt_pk_f16_f32 v126, v243, v247\n\t"
"s_add_u32 s89, s63, 10485760\n\t"
"s_add_u32 s43, s89, s53\n\t"
"v_mfma_f32_16x16x32_f16 a[156:159], v[68:71], v[108:111], a[156:159]\n\t"
"buffer_load_dwordx4 v[240:243], v60, %[srdW], s43 offen\n\t"
"v_mfma_f32_16x16x32_f16 a[140:143], v[64:67], v[108:111], a[140:143]\n\t"
"s_waitcnt lgkmcnt(3)\n\t"
"v_mfma_f32_16x16x32_f16 a[172:175], v[72:75], v[108:111], a[172:175]\n\t"
"ds_read_b128 v[88:91], v57 offset:47104\n\t"
"buffer_load_dwordx4 v[244:247], v61, %[srdW], s43 offen\n\t"
"s_waitcnt lgkmcnt(3)\n\t"
"v_mfma_f32_16x16x32_f16 a[188:191], v[76:79], v[108:111], a[188:191]\n\t"
"s_add_u32 s62, s46, s60\n\t"
"s_add_u32 s49, s47, s60\n\t"
"s_add_u32 m0, s49, 32768\n\t"
"v_mfma_f32_16x16x32_f16 a[184:187], v[76:79], v[104:107], a[184:187]\n\t"
"buffer_load_dwordx4 v59, s[76:79], s62 offen sc1 lds\n\t"
"v_mfma_f32_16x16x32_f16 a[168:171], v[72:75], v[104:107], a[168:171]\n\t"
"v_mfma_f32_16x16x32_f16 a[164:167], v[72:75], v[100:103], a[164:167]\n\t"
"ds_read_b128 v[92:95], v57 offset:48128\n\t"
"v_mfma_f32_16x16x32_f16 a[180:183], v[76:79], v[100:103], a[180:183]\n\t"
"v_mfma_f32_16x16x32_f16 a[176:179], v[76:79], v[96:99], a[176:179]\n\t"
"v_mfma_f32_16x16x32_f16 a[160:163], v[72:75], v[96:99], a[160:163]\n\t"
"s_waitcnt lgkmcnt(3)\n\t"
"v_mfma_f32_16x16x32_f16 a[192:195], v[80:83], v[96:99], a[192:195]\n\t"
"ds_read_b128 v[64:67], v57 offset:49152\n\t"
"s_waitcnt lgkmcnt(3)\n\t"
"v_mfma_f32_16x16x32_f16 a[208:211], v[84:87], v[96:99], a[208:211]\n\t"
"v_mfma_f32_16x16x32_f16 a[212:215], v[84:87], v[100:103], a[212:215]\n\t"
"s_waitcnt vmcnt(47)\n\t"
"v_cvt_pk_f16_f32 v115, v248, v252\n\t"
"v_mfma_f32_16x16x32_f16 a[196:199], v[80:83], v[100:103], a[196:199]\n\t"
"v_cvt_pk_f16_f32 v119, v249, v253\n\t"
"v_mfma_f32_16x16x32_f16 a[200:203], v[80:83], v[104:107], a[200:203]\n\t"
"ds_read_b128 v[68:71], v57 offset:50176\n\t"
"v_cvt_pk_f16_f32 v123, v250, v254\n\t"
"v_mfma_f32_16x16x32_f16 a[216:219], v[84:87], v[104:107], a[216:219]\n\t"
"v_cvt_pk_f16_f32 v127, v251, v255\n\t"
"v_mfma_f32_16x16x32_f16 a[220:223], v[84:87], v[108:111], a[220:223]\n\t"
"buffer_load_dwordx4 v[248:251], v62, %[srdW], s43 offen\n\t"
"v_mfma_f32_16x16x32_f16 a[204:207], v[80:83], v[108:111], a[204:207]\n\t"
"s_waitcnt lgkmcnt(3)\n\t"
"v_mfma_f32_16x16x32_f16 a[236:239], v[88:91], v[108:111], a[236:239]\n\t"
"ds_read_b128 v[72:75], v57 offset:51200\n\t"
"buffer_load_dwordx4 v[252:255], v63, %[srdW], s43 offen\n\t"
"s_waitcnt lgkmcnt(3)\n\t"
"v_mfma_f32_16x16x32_f16 a[252:255], v[92:95], v[108:111], a[252:255]\n\t"
"s_add_u32 s62, s46, s61\n\t"
"s_add_u32 s49, s47, s61\n\t"
"s_add_u32 m0, s49, 32768\n\t"
"v_mfma_f32_16x16x32_f16 a[248:251], v[92:95], v[104:107], a[248:251]\n\t"
"buffer_load_dwordx4 v59, s[76:79], s62 offen sc1 lds\n\t"
"v_mfma_f32_16x16x32_f16 a[232:235], v[88:91], v[104:107], a[232:235]\n\t"
"v_mfma_f32_16x16x32_f16 a[228:231], v[88:91], v[100:103], a[228:231]\n\t"
"ds_read_b128 v[76:79], v57 offset:52224\n\t"
"v_mfma_f32_16x16x32_f16 a[244:247], v[92:95], v[100:103], a[244:247]\n\t"
"s_sub_u32 s80, s80, 1\n\t"
"s_getpc_b64 s[68:69]\n\t"
"s_cbranch_scc1 L_hnd_%=\n\t"
"v_mfma_f32_16x16x32_f16 a[240:243], v[92:95], v[96:99], a[240:243]\n\t"
"v_mfma_f32_16x16x32_f16 a[224:227], v[88:91], v[96:99], a[224:227]\n\t"
"s_waitcnt lgkmcnt(3)\n\t"
"v_mfma_f32_16x16x32_f16 a[0:3], v[64:67], v[112:115], a[0:3]\n\t"
"ds_read_b128 v[80:83], v57 offset:53248\n\t"
"s_waitcnt lgkmcnt(3)\n\t"
"v_mfma_f32_16x16x32_f16 a[16:19], v[68:71], v[112:115], a[16:19]\n\t"
"v_mfma_f32_16x16x32_f16 a[20:23], v[68:71], v[116:119], a[20:23]\n\t"
"s_waitcnt vmcnt(47)\n\t"
"v_cvt_pk_f16_f32 v96, v128, v132\n\t"
"v_mfma_f32_16x16x32_f16 a[4:7], v[64:67], v[116:119], a[4:7]\n\t"
"v_cvt_pk_f16_f32 v100, v129, v133\n\t"
"v_mfma_f32_16x16x32_f16 a[8:11], v[64:67], v[120:123], a[8:11]\n\t"
"ds_read_b128 v[84:87], v57 offset:54272\n\t"
"v_cvt_pk_f16_f32 v104, v130, v134\n\t"
"v_mfma_f32_16x16x32_f16 a[24:27], v[68:71], v[120:123], a[24:27]\n\t"
"v_cvt_pk_f16_f32 v108, v131, v135\n\t"
"s_add_u32 s89, s63, 12582912\n\t"
"s_add_u32 s40, s89, s50\n\t"
"v_mfma_f32_16x16x32_f16 a[28:31], v[68:71], v[124:127], a[28:31]\n\t"
"buffer_load_dwordx4 v[128:131], v60, %[srdW], s40 offen\n\t"
"v_mfma_f32_16x16x32_f16 a[12:15], v[64:67], v[124:127], a[12:15]\n\t"
"s_add_u32 s46, s64, 196608\n\t"
"s_add_u32 s62, s46, s54\n\t"
"s_add_u32 s49, s47, s54\n\t"
"s_add_u32 m0, s49, 65536\n\t"
"s_waitcnt lgkmcnt(3)\n\t"
"v_mfma_f32_16x16x32_f16 a[44:47], v[72:75], v[124:127], a[44:47]\n\t"
"ds_read_b128 v[88:91], v57 offset:55296\n\t"
"buffer_load_dwordx4 v59, s[76:79], s62 offen sc1 lds\n\t"
"s_waitcnt lgkmcnt(3)\n\t"
"v_mfma_f32_16x16x32_f16 a[60:63], v[76:79], v[124:127], a[60:63]\n\t"
"v_mfma_f32_16x16x32_f16 a[56:59], v[76:79], v[120:123], a[56:59]\n\t"
"buffer_load_dwordx4 v[132:135], v61, %[srdW], s40 offen\n\t"
"v_mfma_f32_16x16x32_f16 a[40:43], v[72:75], v[120:123], a[40:43]\n\t"
"s_add_u32 s62, s46, s55\n\t"
"s_add_u32 s49, s47, s55\n\t"
"s_add_u32 m0, s49, 65536\n\t"
"v_mfma_f32_16x16x32_f16 a[36:39], v[72:75], v[116:119], a[36:39]\n\t"
"ds_read_b128 v[92:95], v57 offset:56320\n\t"
"buffer_load_dwordx4 v59, s[76:79], s62 offen sc1 lds\n\t"
"v_mfma_f32_16x16x32_f16 a[52:55], v[76:79], v[116:119], a[52:55]\n\t"
"v_mfma_f32_16x16x32_f16 a[48:51], v[76:79], v[112:115], a[48:51]\n\t"
"v_mfma_f32_16x16x32_f16 a[32:35], v[72:75], v[112:115], a[32:35]\n\t"
"s_waitcnt lgkmcnt(3)\n\t"
"v_mfma_f32_16x16x32_f16 a[64:67], v[80:83], v[112:115], a[64:67]\n\t"
"ds_read_b128 v[64:67], v57 offset:57344\n\t"
"s_waitcnt lgkmcnt(3)\n\t"
"v_mfma_f32_16x16x32_f16 a[80:83], v[84:87], v[112:115], a[80:83]\n\t"
"v_mfma_f32_16x16x32_f16 a[84:87], v[84:87], v[116:119], a[84:87]\n\t"
"s_waitcnt vmcnt(48)\n\t"
"v_cvt_pk_f16_f32 v97, v136, v140\n\t"
"v_mfma_f32_16x16x32_f16 a[68:71], v[80:83], v[116:119], a[68:71]\n\t"
"v_cvt_pk_f16_f32 v101, v137, v141\n\t"
"v_mfma_f32_16x16x32_f16 a[72:75], v[80:83], v[120:123], a[72:75]\n\t"
"ds_read_b128 v[68:71], v57 offset:58368\n\t"
"v_cvt_pk_f16_f32 v105, v138, v142\n\t"
"v_mfma_f32_16x16x32_f16 a[88:91], v[84:87], v[120:123], a[88:91]\n\t"
"v_cvt_pk_f16_f32 v109, v139, v143\n\t"
"v_mfma_f32_16x16x32_f16 a[92:95], v[84:87], v[124:127], a[92:95]\n\t"
"buffer_load_dwordx4 v[136:139], v62, %[srdW], s40 offen\n\t"
"v_mfma_f32_16x16x32_f16 a[76:79], v[80:83], v[124:127], a[76:79]\n\t"
"s_add_u32 s62, s46, s56\n\t"
"s_add_u32 s49, s47, s56\n\t"
"s_add_u32 m0, s49, 65536\n\t"
"s_waitcnt lgkmcnt(3)\n\t"
"v_mfma_f32_16x16x32_f16 a[108:111], v[88:91], v[124:127], a[108:111]\n\t"
"ds_read_b128 v[72:75], v57 offset:59392\n\t"
"buffer_load_dwordx4 v59, s[76:79], s62 offen sc1 lds\n\t"
"s_waitcnt lgkmcnt(3)\n\t"
"v_mfma_f32_16x16x32_f16 a[124:127], v[92:95], v[124:127], a[124:127]\n\t"
"v_mfma_f32_16x16x32_f16 a[120:123], v[92:95], v[120:123], a[120:123]\n\t"
"buffer_load_dwordx4 v[140:143], v63, %[srdW], s40 offen\n\t"
"v_mfma_f32_16x16x32_f16 a[104:107], v[88:91], v[120:123], a[104:107]\n\t"
"s_add_u32 s62, s46, s57\n\t"
"s_add_u32 s49, s47, s57\n\t"
"s_add_u32 m0, s49, 65536\n\t"
"v_mfma_f32_16x16x32_f16 a[100:103], v[88:91], v[116:119], a[100:103]\n\t"
"ds_read_b128 v[76:79], v57 offset:60416\n\t"
"buffer_load_dwordx4 v59, s[76:79], s62 offen sc1 lds\n\t"
"v_mfma_f32_16x16x32_f16 a[116:119], v[92:95], v[116:119], a[116:119]\n\t"
"v_mfma_f32_16x16x32_f16 a[112:115], v[92:95], v[112:115], a[112:115]\n\t"
"v_mfma_f32_16x16x32_f16 a[96:99], v[88:91], v[112:115], a[96:99]\n\t"
"s_waitcnt lgkmcnt(3)\n\t"
"v_mfma_f32_16x16x32_f16 a[128:131], v[64:67], v[112:115], a[128:131]\n\t"
"ds_read_b128 v[80:83], v57 offset:61440\n\t"
"s_waitcnt lgkmcnt(3)\n\t"
"v_mfma_f32_16x16x32_f16 a[144:147], v[68:71], v[112:115], a[144:147]\n\t"
"v_mfma_f32_16x16x32_f16 a[148:151], v[68:71], v[116:119], a[148:151]\n\t"
"s_waitcnt vmcnt(49)\n\t"
"v_cvt_pk_f16_f32 v98, v144, v148\n\t"
"v_mfma_f32_16x16x32_f16 a[132:135], v[64:67], v[116:119], a[132:135]\n\t"
"v_cvt_pk_f16_f32 v102, v145, v149\n\t"
"v_mfma_f32_16x16x32_f16 a[136:139], v[64:67], v[120:123], a[136:139]\n\t"
"ds_read_b128 v[84:87], v57 offset:62464\n\t"
"v_cvt_pk_f16_f32 v106, v146, v150\n\t"
"v_mfma_f32_16x16x32_f16 a[152:155], v[68:71], v[120:123], a[152:155]\n\t"
"v_cvt_pk_f16_f32 v110, v147, v151\n\t"
"s_add_u32 s89, s63, 12582912\n\t"
"s_add_u32 s41, s89, s51\n\t"
"v_mfma_f32_16x16x32_f16 a[156:159], v[68:71], v[124:127], a[156:159]\n\t"
"buffer_load_dwordx4 v[144:147], v60, %[srdW], s41 offen\n\t"
"v_mfma_f32_16x16x32_f16 a[140:143], v[64:67], v[124:127], a[140:143]\n\t"
"s_add_u32 s62, s46, s58\n\t"
"s_add_u32 s49, s47, s58\n\t"
"s_add_u32 m0, s49, 65536\n\t"
"s_waitcnt lgkmcnt(3)\n\t"
"v_mfma_f32_16x16x32_f16 a[172:175], v[72:75], v[124:127], a[172:175]\n\t"
"ds_read_b128 v[88:91], v57 offset:63488\n\t"
"buffer_load_dwordx4 v59, s[76:79], s62 offen sc1 lds\n\t"
"s_waitcnt lgkmcnt(3)\n\t"
"v_mfma_f32_16x16x32_f16 a[188:191], v[76:79], v[124:127], a[188:191]\n\t"
"v_mfma_f32_16x16x32_f16 a[184:187], v[76:79], v[120:123], a[184:187]\n\t"
"buffer_load_dwordx4 v[148:151], v61, %[srdW], s41 offen\n\t"
"v_mfma_f32_16x16x32_f16 a[168:171], v[72:75], v[120:123], a[168:171]\n\t"
"s_add_u32 s62, s46, s59\n\t"
"s_add_u32 s49, s47, s59\n\t"
"s_add_u32 m0, s49, 65536\n\t"
"v_mfma_f32_16x16x32_f16 a[164:167], v[72:75], v[116:119], a[164:167]\n\t"
"ds_read_b128 v[92:95], v57 offset:64512\n\t"
"buffer_load_dwordx4 v59, s[76:79], s62 offen sc1 lds\n\t"
"v_mfma_f32_16x16x32_f16 a[180:183], v[76:79], v[116:119], a[180:183]\n\t"
"v_mfma_f32_16x16x32_f16 a[176:179], v[76:79], v[112:115], a[176:179]\n\t"
"v_mfma_f32_16x16x32_f16 a[160:163], v[72:75], v[112:115], a[160:163]\n\t"
"s_waitcnt lgkmcnt(3)\n\t"
"v_mfma_f32_16x16x32_f16 a[192:195], v[80:83], v[112:115], a[192:195]\n\t"
"s_waitcnt lgkmcnt(2)\n\t"
"v_mfma_f32_16x16x32_f16 a[208:211], v[84:87], v[112:115], a[208:211]\n\t"
"s_memrealtime s[90:91]\n\t"
"v_mfma_f32_16x16x32_f16 a[212:215], v[84:87], v[116:119], a[212:215]\n\t"
"s_waitcnt vmcnt(50)\n\t"
"v_cvt_pk_f16_f32 v99, v152, v156\n\t"
"v_mfma_f32_16x16x32_f16 a[196:199], v[80:83], v[116:119], a[196:199]\n\t"
"v_cvt_pk_f16_f32 v103, v153, v157\n\t"
"s_waitcnt vmcnt(37) lgkmcnt(0)\n\t"
"s_sub_u32 s89, s90, s65\n\t"
"s_cmp_lt_i32 s89, 0\n\t"
"s_getpc_b64 s[68:69]\n\t"
"s_cbranch_scc1 L_pace_%=\n\t"
"s_ashr_i32 s89, s95, 3\n\t"
"s_add_u32 s65, s90, s89\n\t"
"s_sub_u32 s95, s95, %[sslope]\n\t"
"s_barrier\n\t"
"v_mfma_f32_16x16x32_f16 a[200:203], v[80:83], v[120:123], a[200:203]\n\t"
"ds_read_b128 v[64:67], v58 offset:0\n\t"
"v_cvt_pk_f16_f32 v107, v154, v158\n\t"
"v_mfma_f32_16x16x32_f16 a[216:219], v[84:87], v[120:123], a[216:219]\n\t"
"ds_read_b64 v[54:55], v53\n\t"
"v_add_u32 v53, 8, v53\n\t"
"v_cvt_pk_f16_f32 v111, v155, v159\n\t"
"v_mfma_f32_16x16x32_f16 a[220:223], v[84:87], v[124:127], a[220:223]\n\t"
"buffer_load_dwordx4 v[152:155], v62, %[srdW], s41 offen\n\t"
"v_mfma_f32_16x16x32_f16 a[204:207], v[80:83], v[124:127], a[204:207]\n\t"
"ds_read_b128 v[68:71], v58 offset:1024\n\t"
"s_add_u32 s62, s46, s60\n\t"
"s_add_u32 s49, s47, s60\n\t"
"s_add_u32 m0, s49, 65536\n\t"
"s_waitcnt lgkmcnt(4)\n\t"
"v_mfma_f32_16x16x32_f16 a[236:239], v[88:91], v[124:127], a[236:239]\n\t"
"buffer_load_dwordx4 v59, s[76:79], s62 offen sc1 lds\n\t"
"s_waitcnt lgkmcnt(3)\n\t"
"v_mfma_f32_16x16x32_f16 a[252:255], v[92:95], v[124:127], a[252:255]\n\t"
"v_mfma_f32_16x16x32_f16 a[248:251], v[92:95], v[120:123], a[248:251]\n\t"
"ds_read_b128 v[72:75], v58 offset:2048\n\t"
"buffer_load_dwordx4 v[156:159], v63, %[srdW], s41 offen\n\t"
"v_mfma_f32_16x16x32_f16 a[232:235], v[88:91], v[120:123], a[232:235]\n\t"
"s_add_u32 s62, s46, s61\n\t"
"s_add_u32 s49, s47, s61\n\t"
"s_add_u32 m0, s49, 65536\n\t"
"v_mfma_f32_16x16x32_f16 a[228:231], v[88:91], v[116:119], a[228:231]\n\t"
"buffer_load_dwordx4 v59, s[76:79], s62 offen sc1 lds\n\t"
"v_mfma_f32_16x16x32_f16 a[244:247], v[92:95], v[116:119], a[244:247]\n\t"
"ds_read_b128 v[76:79], v58 offset:3072\n\t"
"s_sub_u32 s80, s80, 1\n\t"
"s_getpc_b64 s[68:69]\n\t"
"s_cbranch_scc1 L_hnd_%=\n\t"
"v_mfma_f32_16x16x32_f16 a[240:243], v[92:95], v[112:115], a[240:243]\n\t"
"s_add_u32 s89, s63, 12582912\n\t"
"s_add_u32 s42, s89, s52\n\t"
"v_mfma_f32_16x16x32_f16 a[224:227], v[88:91], v[112:115], a[224:227]\n\t"
"s_add_u32 s89, s63, 12582912\n\t"
"s_add_u32 s43, s89, s53\n\t"
"s_sub_u32 s48, s48, 1\n\t"
"L_loop_%=:\n\t"
"s_waitcnt lgkmcnt(4)\n\t"
"v_mfma_f32_16x16x32_f16 a[0:3], v[64:67], v[96:99], a[0:3]\n\t"
"ds_read_b128 v[80:83], v58 offset:4096\n\t"
"s_waitcnt vmcnt(23)\n\t"
"v_readfirstlane_b32 s89, v43\n\t"
"s_cmp_ge_u32 s89, s71\n\t"
"s_cbranch_scc0 L_slow0_%=\n\t"
"L_sback0_%=:\n\t"
"s_waitcnt lgkmcnt(3)\n\t"
"v_mfma_f32_16x16x32_f16 a[16:19], v[68:71], v[96:99], a[16:19]\n\t"
"s_waitcnt lgkmcnt(4)\n\t"
"v_readfirstlane_b32 s49, v54\n\t"
"v_readfirstlane_b32 s70, v55\n\t"
"s_lshl_b32 s46, s49, 14\n\t"
"s_lshl_b32 s49, s49, 20\n\t"
"v_mfma_f32_16x16x32_f16 a[20:23], v[68:71], v[100:103], a[20:23]\n\t"
"s_add_u32 s40, s49, s50\n\t"
"s_add_u32 s41, s49, s51\n\t"
"s_waitcnt vmcnt(46)\n\t"
"v_cvt_pk_f16_f32 v112, v160, v164\n\t"
"v_mfma_f32_16x16x32_f16 a[4:7], v[64:67], v[100:103], a[4:7]\n\t"
"s_add_u32 s44, s49, s52\n\t"
"s_add_u32 s45, s49, s53\n\t"
"v_cvt_pk_f16_f32 v116, v161, v165\n\t"
"v_mfma_f32_16x16x32_f16 a[8:11], v[64:67], v[104:107], a[8:11]\n\t"
"ds_read_b128 v[84:87], v58 offset:5120\n\t"
"v_cvt_pk_f16_f32 v120, v162, v166\n\t"
"v_mfma_f32_16x16x32_f16 a[24:27], v[68:71], v[104:107], a[24:27]\n\t"
"v_cvt_pk_f16_f32 v124, v163, v167\n\t"
"v_mfma_f32_16x16x32_f16 a[28:31], v[68:71], v[108:111], a[28:31]\n\t"
"buffer_load_dwordx4 v[160:163], v60, %[srdW], s42 offen\n\t"
"v_mfma_f32_16x16x32_f16 a[12:15], v[64:67], v[108:111], a[12:15]\n\t"
"buffer_load_dword v43, off, %[srdC], s70 sc1\n\t"
"s_waitcnt lgkmcnt(3)\n\t"
"v_mfma_f32_16x16x32_f16 a[44:47], v[72:75], v[108:111], a[44:47]\n\t"
"ds_read_b128 v[88:91], v58 offset:6144\n\t"
"s_add_u32 s62, s46, s54\n\t"
"s_add_u32 s49, s47, s54\n\t"
"s_add_u32 m0, s49, 98304\n\t"
"s_waitcnt lgkmcnt(3)\n\t"
"v_mfma_f32_16x16x32_f16 a[60:63], v[76:79], v[108:111], a[60:63]\n\t"
"buffer_load_dwordx4 v59, s[76:79], s62 offen sc1 lds\n\t"
"v_mfma_f32_16x16x32_f16 a[56:59], v[76:79], v[104:107], a[56:59]\n\t"
"v_mfma_f32_16x16x32_f16 a[40:43], v[72:75], v[104:107], a[40:43]\n\t"
"v_mfma_f32_16x16x32_f16 a[36:39], v[72:75], v[100:103], a[36:39]\n\t"
"ds_read_b128 v[92:95], v58 offset:7168\n\t"
"buffer_load_dwordx4 v[164:167], v61, %[srdW], s42 offen\n\t"
"v_mfma_f32_16x16x32_f16 a[52:55], v[76:79], v[100:103], a[52:55]\n\t"
"v_mfma_f32_16x16x32_f16 a[48:51], v[76:79], v[96:99], a[48:51]\n\t"
"v_mfma_f32_16x16x32_f16 a[32:35], v[72:75], v[96:99], a[32:35]\n\t"
"s_waitcnt lgkmcnt(3)\n\t"
"v_mfma_f32_16x16x32_f16 a[64:67], v[80:83], v[96:99], a[64:67]\n\t"
"ds_read_b128 v[64:67], v58 offset:8192\n\t"
"s_waitcnt lgkmcnt(3)\n\t"
"v_mfma_f32_16x16x32_f16 a[80:83], v[84:87], v[96:99], a[80:83]\n\t"
"v_mfma_f32_16x16x32_f16 a[84:87], v[84:87], v[100:103], a[84:87]\n\t"
"s_waitcnt vmcnt(47)\n\t"
"v_cvt_pk_f16_f32 v113, v168, v172\n\t"
"v_mfma_f32_16x16x32_f16 a[68:71], v[80:83], v[100:103], a[68:71]\n\t"
"v_cvt_pk_f16_f32 v117, v169, v173\n\t"
"v_mfma_f32_16x16x32_f16 a[72:75], v[80:83], v[104:107], a[72:75]\n\t"
"ds_read_b128 v[68:71], v58 offset:9216\n\t"
"v_cvt_pk_f16_f32 v121, v170, v174\n\t"
"v_mfma_f32_16x16x32_f16 a[88:91], v[84:87], v[104:107], a[88:91]\n\t"
"v_cvt_pk_f16_f32 v125, v171, v175\n\t"
"v_mfma_f32_16x16x32_f16 a[92:95], v[84:87], v[108:111], a[92:95]\n\t"
"buffer_load_dwordx4 v[168:171], v62, %[srdW], s42 offen\n\t"
"v_mfma_f32_16x16x32_f16 a[76:79], v[80:83], v[108:111], a[76:79]\n\t"
"s_waitcnt lgkmcnt(3)\n\t"
"v_mfma_f32_16x16x32_f16 a[108:111], v[88:91], v[108:111], a[108:111]\n\t"
"ds_read_b128 v[72:75], v58 offset:10240\n\t"
"s_add_u32 s62, s46, s55\n\t"
"s_add_u32 s49, s47, s55\n\t"
"s_add_u32 m0, s49, 98304\n\t"
"s_waitcnt lgkmcnt(3)\n\t"
"v_mfma_f32_16x16x32_f16 a[124:127], v[92:95], v[108:111], a[124:127]\n\t"
"buffer_load_dwordx4 v59, s[76:79], s62 offen sc1 lds\n\t"
"v_mfma_f32_16x16x32_f16 a[120:123], v[92:95], v[104:107], a[120:123]\n\t"
"v_mfma_f32_16x16x32_f16 a[104:107], v[88:91], v[104:107], a[104:107]\n\t"
"v_mfma_f32_16x16x32_f16 a[100:103], v[88:91], v[100:103], a[100:103]\n\t"
"ds_read_b128 v[76:79], v58 offset:11264\n\t"
"buffer_load_dwordx4 v[172:175], v63, %[srdW], s42 offen\n\t"
"v_mfma_f32_16x16x32_f16 a[116:119], v[92:95], v[100:103], a[116:119]\n\t"
"v_mfma_f32_16x16x32_f16 a[112:115], v[92:95], v[96:99], a[112:115]\n\t"
"v_mfma_f32_16x16x32_f16 a[96:99], v[88:91], v[96:99], a[96:99]\n\t"
"s_waitcnt lgkmcnt(3)\n\t"
"v_mfma_f32_16x16x32_f16 a[128:131], v[64:67], v[96:99], a[128:131]\n\t"
"ds_read_b128 v[80:83], v58 offset:12288\n\t"
"s_waitcnt lgkmcnt(3)\n\t"
"v_mfma_f32_16x16x32_f16 a[144:147], v[68:71], v[96:99], a[144:147]\n\t"
"v_mfma_f32_16x16x32_f16 a[148:151], v[68:71], v[100:103], a[148:151]\n\t"
"s_waitcnt vmcnt(47)\n\t"
"v_cvt_pk_f16_f32 v114, v176, v180\n\t"
"v_mfma_f32_16x16x32_f16 a[132:135], v[64:67], v[100:103], a[132:135]\n\t"
"v_cvt_pk_f16_f32 v118, v177, v181\n\t"
"v_mfma_f32_16x16x32_f16 a[136:139], v[64:67], v[104:107], a[136:139]\n\t"
"ds_read_b128 v[84:87], v58 offset:13312\n\t"
"v_cvt_pk_f16_f32 v122, v178, v182\n\t"
"v_mfma_f32_16x16x32_f16 a[152:155], v[68:71], v[104:107], a[152:155]\n\t"
"v_cvt_pk_f16_f32 v126, v179, v183\n\t"
"v_mfma_f32_16x16x32_f16 a[156:159], v[68:71], v[108:111], a[156:159]\n\t"
"buffer_load_dwordx4 v[176:179], v60, %[srdW], s43 offen\n\t"
"v_mfma_f32_16x16x32_f16 a[140:143], v[64:67], v[108:111], a[140:143]\n\t"
"s_waitcnt lgkmcnt(3)\n\t"
"v_mfma_f32_16x16x32_f16 a[172:175], v[72:75], v[108:111], a[172:175]\n\t"
"ds_read_b128 v[88:91], v58 offset:14336\n\t"
"s_add_u32 s62, s46, s56\n\t"
"s_add_u32 s49, s47, s56\n\t"
"s_add_u32 m0, s49, 98304\n\t"
"s_waitcnt lgkmcnt(3)\n\t"
"v_mfma_f32_16x16x32_f16 a[188:191], v[76:79], v[108:111], a[188:191]\n\t"
"buffer_load_dwordx4 v59, s[76:79], s62 offen sc1 lds\n\t"
"v_mfma_f32_16x16x32_f16 a[184:187], v[76:79], v[104:107], a[184:187]\n\t"
"v_mfma_f32_16x16x32_f16 a[168:171], v[72:75], v[104:107], a[168:171]\n\t"
"v_mfma_f32_16x16x32_f16 a[164:167], v[72:75], v[100:103], a[164:167]\n\t"
"ds_read_b128 v[92:95], v58 offset:15360\n\t"
"buffer_load_dwordx4 v[180:183], v61, %[srdW], s43 offen\n\t"
"v_mfma_f32_16x16x32_f16 a[180:183], v[76:79], v[100:103], a[180:183]\n\t"
"v_mfma_f32_16x16x32_f16 a[176:179], v[76:79], v[96:99], a[176:179]\n\t"
"v_mfma_f32_16x16x32_f16 a[160:163], v[72:75], v[96:99], a[160:163]\n\t"
"s_waitcnt lgkmcnt(3)\n\t"
"v_mfma_f32_16x16x32_f16 a[192:195], v[80:83], v[96:99], a[192:195]\n\t"
"ds_read_b128 v[64:67], v58 offset:16384\n\t"
"s_waitcnt lgkmcnt(3)\n\t"
"v_mfma_f32_16x16x32_f16 a[208:211], v[84:87], v[96:99], a[208:211]\n\t"
"v_mfma_f32_16x16x32_f16 a[212:215], v[84:87], v[100:103], a[212:215]\n\t"
"s_waitcnt vmcnt(47)\n\t"
"v_cvt_pk_f16_f32 v115, v184, v188\n\t"
"v_mfma_f32_16x16x32_f16 a[196:199], v[80:83], v[100:103], a[196:199]\n\t"
"v_cvt_pk_f16_f32 v119, v185, v189\n\t"
"v_mfma_f32_16x16x32_f16 a[200:203], v[80:83], v[104:107], a[200:203]\n\t"
"ds_read_b128 v[68:71], v58 offset:17408\n\t"
"v_cvt_pk_f16_f32 v123, v186, v190\n\t"
"v_mfma_f32_16x16x32_f16 a[216:219], v[84:87], v[104:107], a[216:219]\n\t"
"v_cvt_pk_f16_f32 v127, v187, v191\n\t"
"v_mfma_f32_16x16x32_f16 a[220:223], v[84:87], v[108:111], a[220:223]\n\t"
"buffer_load_dwordx4 v[184:187], v62, %[srdW], s43 offen\n\t"
"v_mfma_f32_16x16x32_f16 a[204:207], v[80:83], v[108:111], a[204:207]\n\t"
"s_waitcnt lgkmcnt(3)\n\t"
"v_mfma_f32_16x16x32_f16 a[236:239], v[88:91], v[108:111], a[236:239]\n\t"
"ds_read_b128 v[72:75], v58 offset:18432\n\t"
"s_add_u32 s62, s46, s57\n\t"
"s_add_u32 s49, s47, s57\n\t"
"s_add_u32 m0, s49, 98304\n\t"
"s_waitcnt lgkmcnt(3)\n\t"
"v_mfma_f32_16x16x32_f16 a[252:255], v[92:95], v[108:111], a[252:255]\n\t"
"buffer_load_dwordx4 v59, s[76:79], s62 offen sc1 lds\n\t"
"v_mfma_f32_16x16x32_f16 a[248:251], v[92:95], v[104:107], a[248:251]\n\t"
"v_mfma_f32_16x16x32_f16 a[232:235], v[88:91], v[104:107], a[232:235]\n\t"
"v_mfma_f32_16x16x32_f16 a[228:231], v[88:91], v[100:103], a[228:231]\n\t"
"ds_read_b128 v[76:79], v58 offset:19456\n\t"
"buffer_load_dwordx4 v[188:191], v63, %[srdW], s43 offen\n\t"
"v_mfma_f32_16x16x32_f16 a[244:247], v[92:95], v[100:103], a[244:247]\n\t"
"s_sub_u32 s80, s80, 1\n\t"
"s_getpc_b64 s[68:69]\n\t"
"s_cbranch_scc1 L_hnd_%=\n\t"
"v_mfma_f32_16x16x32_f16 a[240:243], v[92:95], v[96:99], a[240:243]\n\t"
"v_mfma_f32_16x16x32_f16 a[224:227], v[88:91], v[96:99], a[224:227]\n\t"
"s_waitcnt lgkmcnt(3)\n\t"
"v_mfma_f32_16x16x32_f16 a[0:3], v[64:67], v[112:115], a[0:3]\n\t"
"ds_read_b128 v[80:83], v58 offset:20480\n\t"
"s_waitcnt lgkmcnt(3)\n\t"
"v_mfma_f32_16x16x32_f16 a[16:19], v[68:71], v[112:115], a[16:19]\n\t"
"s_mov_b32 s42, s44\n\t"
"s_mov_b32 s43, s45\n\t"
"v_mfma_f32_16x16x32_f16 a[20:23], v[68:71], v[116:119], a[20:23]\n\t"
"s_waitcnt vmcnt(47)\n\t"
"v_cvt_pk_f16_f32 v96, v192, v196\n\t"
"v_mfma_f32_16x16x32_f16 a[4:7], v[64:67], v[116:119], a[4:7]\n\t"
"v_cvt_pk_f16_f32 v100, v193, v197\n\t"
"v_mfma_f32_16x16x32_f16 a[8:11], v[64:67], v[120:123], a[8:11]\n\t"
"ds_read_b128 v[84:87], v58 offset:21504\n\t"
"v_cvt_pk_f16_f32 v104, v194, v198\n\t"
"v_mfma_f32_16x16x32_f16 a[24:27], v[68:71], v[120:123], a[24:27]\n\t"
"v_cvt_pk_f16_f32 v108, v195, v199\n\t"
"v_mfma_f32_16x16x32_f16 a[28:31], v[68:71], v[124:127], a[28:31]\n\t"
"buffer_load_dwordx4 v[192:195], v60, %[srdW], s40 offen\n\t"
"v_mfma_f32_16x16x32_f16 a[12:15], v[64:67], v[124:127], a[12:15]\n\t"
"s_waitcnt lgkmcnt(3)\n\t"
"v_mfma_f32_16x16x32_f16 a[44:47], v[72:75], v[124:127], a[44:47]\n\t"
"ds_read_b128 v[88:91], v58 offset:22528\n\t"
"s_add_u32 s62, s46, s58\n\t"
"s_add_u32 s49, s47, s58\n\t"
"s_add_u32 m0, s49, 98304\n\t"
"s_waitcnt lgkmcnt(3)\n\t"
"v_mfma_f32_16x16x32_f16 a[60:63], v[76:79], v[124:127], a[60:63]\n\t"
"buffer_load_dwordx4 v59, s[76:79], s62 offen sc1 lds\n\t"
"v_mfma_f32_16x16x32_f16 a[56:59], v[76:79], v[120:123], a[56:59]\n\t"
"v_mfma_f32_16x16x32_f16 a[40:43], v[72:75], v[120:123], a[40:43]\n\t"
"v_mfma_f32_16x16x32_f16 a[36:39], v[72:75], v[116:119], a[36:39]\n\t"
"ds_read_b128 v[92:95], v58 offset:23552\n\t"
"buffer_load_dwordx4 v[196:199], v61, %[srdW], s40 offen\n\t"
"v_mfma_f32_16x16x32_f16 a[52:55], v[76:79], v[116:119], a[52:55]\n\t"
"v_mfma_f32_16x16x32_f16 a[48:51], v[76:79], v[112:115], a[48:51]\n\t"
"v_mfma_f32_16x16x32_f16 a[32:35], v[72:75], v[112:115], a[32:35]\n\t"
"s_waitcnt lgkmcnt(3)\n\t"
"v_mfma_f32_16x16x32_f16 a[64:67], v[80:83], v[112:115], a[64:67]\n\t"
"ds_read_b128 v[64:67], v58 offset:24576\n\t"
"s_waitcnt lgkmcnt(3)\n\t"
"v_mfma_f32_16x16x32_f16 a[80:83], v[84:87], v[112:115], a[80:83]\n\t"
"v_mfma_f32_16x16x32_f16 a[84:87], v[84:87], v[116:119], a[84:87]\n\t"
"s_waitcnt vmcnt(47)\n\t"
"v_cvt_pk_f16_f32 v97, v200, v204\n\t"
"v_mfma_f32_16x16x32_f16 a[68:71], v[80:83], v[116:119], a[68:71]\n\t"
"v_cvt_pk_f16_f32 v101, v201, v205\n\t"
"v_mfma_f32_16x16x32_f16 a[72:75], v[80:83], v[120:123], a[72:75]\n\t"
"ds_read_b128 v[68:71], v58 offset:25600\n\t"
"v_cvt_pk_f16_f32 v105, v202, v206\n\t"
"v_mfma_f32_16x16x32_f16 a[88:91], v[84:87], v[120:123], a[88:91]\n\t"
"v_cvt_pk_f16_f32 v109, v203, v207\n\t"
"v_mfma_f32_16x16x32_f16 a[92:95], v[84:87], v[124:127], a[92:95]\n\t"
"buffer_load_dwordx4 v[200:203], v62, %[srdW], s40 offen\n\t"
"v_mfma_f32_16x16x32_f16 a[76:79], v[80:83], v[124:127], a[76:79]\n\t"
"s_waitcnt lgkmcnt(3)\n\t"
"v_mfma_f32_16x16x32_f16 a[108:111], v[88:91], v[124:127], a[108:111]\n\t"
"ds_read_b128 v[72:75], v58 offset:26624\n\t"
"s_add_u32 s62, s46, s59\n\t"
"s_add_u32 s49, s47, s59\n\t"
"s_add_u32 m0, s49, 98304\n\t"
"s_waitcnt lgkmcnt(3)\n\t"
"v_mfma_f32_16x16x32_f16 a[124:127], v[92:95], v[124:127], a[124:127]\n\t"
"buffer_load_dwordx4 v59, s[76:79], s62 offen sc1 lds\n\t"
"v_mfma_f32_16x16x32_f16 a[120:123], v[92:95], v[120:123], a[120:123]\n\t"
"v_mfma_f32_16x16x32_f16 a[104:107], v[88:91], v[120:123], a[104:107]\n\t"
"v_mfma_f32_16x16x32_f16 a[100:103], v[88:91], v[116:119], a[100:103]\n\t"
"ds_read_b128 v[76:79], v58 offset:27648\n\t"
"buffer_load_dwordx4 v[204:207], v63, %[srdW], s40 offen\n\t"
"v_mfma_f32_16x16x32_f16 a[116:119], v[92:95], v[116:119], a[116:119]\n\t"
"v_mfma_f32_16x16x32_f16 a[112:115], v[92:95], v[112:115], a[112:115]\n\t"
"v_mfma_f32_16x16x32_f16 a[96:99], v[88:91], v[112:115], a[96:99]\n\t"
"s_waitcnt lgkmcnt(3)\n\t"
"v_mfma_f32_16x16x32_f16 a[128:131], v[64:67], v[112:115], a[128:131]\n\t"
"ds_read_b128 v[80:83], v58 offset:28672\n\t"
"s_waitcnt lgkmcnt(3)\n\t"
"v_mfma_f32_16x16x32_f16 a[144:147], v[68:71], v[112:115], a[144:147]\n\t"
"v_mfma_f32_16x16x32_f16 a[148:151], v[68:71], v[116:119], a[148:151]\n\t"
"s_waitcnt vmcnt(47)\n\t"
"v_cvt_pk_f16_f32 v98, v208, v212\n\t"
"v_mfma_f32_16x16x32_f16 a[132:135], v[64:67], v[116:119], a[132:135]\n\t"
"v_cvt_pk_f16_f32 v102, v209, v213\n\t"
"v_mfma_f32_16x16x32_f16 a[136:139], v[64:67], v[120:123], a[136:139]\n\t"
"ds_read_b128 v[84:87], v58 offset:29696\n\t"
"v_cvt_pk_f16_f32 v106, v210, v214\n\t"
"v_mfma_f32_16x16x32_f16 a[152:155], v[68:71], v[120:123], a[152:155]\n\t"
"v_cvt_pk_f16_f32 v110, v211, v215\n\t"
"v_mfma_f32_16x16x32_f16 a[156:159], v[68:71], v[124:127], a[156:159]\n\t"
"buffer_load_dwordx4 v[208:211], v60, %[srdW], s41 offen\n\t"
"v_mfma_f32_16x16x32_f16 a[140:143], v[64:67], v[124:127], a[140:143]\n\t"
"s_waitcnt lgkmcnt(3)\n\t"
"v_mfma_f32_16x16x32_f16 a[172:175], v[72:75], v[124:127], a[172:175]\n\t"
"ds_read_b128 v[88:91], v58 offset:30720\n\t"
"s_add_u32 s62, s46, s60\n\t"
"s_add_u32 s49, s47, s60\n\t"
"s_add_u32 m0, s49, 98304\n\t"
"s_waitcnt lgkmcnt(3)\n\t"
"v_mfma_f32_16x16x32_f16 a[188:191], v[76:79], v[124:127], a[188:191]\n\t"
"buffer_load_dwordx4 v59, s[76:79], s62 offen sc1 lds\n\t"
"v_mfma_f32_16x16x32_f16 a[184:187], v[76:79], v[120:123], a[184:187]\n\t"
"v_mfma_f32_16x16x32_f16 a[168:171], v[72:75], v[120:123], a[168:171]\n\t"
"v_mfma_f32_16x16x32_f16 a[164:167], v[72:75], v[116:119], a[164:167]\n\t"
"ds_read_b128 v[92:95], v58 offset:31744\n\t"
"buffer_load_dwordx4 v[212:215], v61, %[srdW], s41 offen\n\t"
"v_mfma_f32_16x16x32_f16 a[180:183], v[76:79], v[116:119], a[180:183]\n\t"
"v_mfma_f32_16x16x32_f16 a[176:179], v[76:79], v[112:115], a[176:179]\n\t"
"v_mfma_f32_16x16x32_f16 a[160:163], v[72:75], v[112:115], a[160:163]\n\t"
"s_waitcnt lgkmcnt(3)\n\t"
"v_mfma_f32_16x16x32_f16 a[192:195], v[80:83], v[112:115], a[192:195]\n\t"
"s_waitcnt lgkmcnt(2)\n\t"
"v_mfma_f32_16x16x32_f16 a[208:211], v[84:87], v[112:115], a[208:211]\n\t"
"s_memrealtime s[90:91]\n\t"
"v_mfma_f32_16x16x32_f16 a[212:215], v[84:87], v[116:119], a[212:215]\n\t"
"s_waitcnt vmcnt(47)\n\t"
"v_cvt_pk_f16_f32 v99, v216, v220\n\t"
"v_mfma_f32_16x16x32_f16 a[196:199], v[80:83], v[116:119], a[196:199]\n\t"
"v_cvt_pk_f16_f32 v103, v217, v221\n\t"
"s_waitcnt vmcnt(38) lgkmcnt(0)\n\t"
"s_sub_u32 s89, s90, s65\n\t"
"s_cmp_lt_i32 s89, 0\n\t"
"s_getpc_b64 s[68:69]\n\t"
"s_cbranch_scc1 L_pace_%=\n\t"
"s_ashr_i32 s89, s95, 3\n\t"
"s_add_u32 s65, s90, s89\n\t"
"s_sub_u32 s95, s95, %[sslope]\n\t"
"s_barrier\n\t"
"v_mfma_f32_16x16x32_f16 a[200:203], v[80:83], v[120:123], a[200:203]\n\t"
"ds_read_b128 v[64:67], v58 offset:32768\n\t"
"v_cvt_pk_f16_f32 v107, v218, v222\n\t"
"v_mfma_f32_16x16x32_f16 a[216:219], v[84:87], v[120:123], a[216:219]\n\t"
"ds_read_b64 v[54:55], v53\n\t"
"v_add_u32 v53, 8, v53\n\t"
"v_cvt_pk_f16_f32 v111, v219, v223\n\t"
"v_mfma_f32_16x16x32_f16 a[220:223], v[84:87], v[124:127], a[220:223]\n\t"
"buffer_load_dwordx4 v[216:219], v62, %[srdW], s41 offen\n\t"
"v_mfma_f32_16x16x32_f16 a[204:207], v[80:83], v[124:127], a[204:207]\n\t"
"ds_read_b128 v[68:71], v58 offset:33792\n\t"
"s_waitcnt lgkmcnt(4)\n\t"
"v_mfma_f32_16x16x32_f16 a[236:239], v[88:91], v[124:127], a[236:239]\n\t"
"s_add_u32 s62, s46, s61\n\t"
"s_add_u32 s49, s47, s61\n\t"
"s_add_u32 m0, s49, 98304\n\t"
"s_waitcnt lgkmcnt(3)\n\t"
"v_mfma_f32_16x16x32_f16 a[252:255], v[92:95], v[124:127], a[252:255]\n\t"
"buffer_load_dwordx4 v59, s[76:79], s62 offen sc1 lds\n\t"
"v_mfma_f32_16x16x32_f16 a[248:251], v[92:95], v[120:123], a[248:251]\n\t"
"ds_read_b128 v[72:75], v58 offset:34816\n\t"
"v_mfma_f32_16x16x32_f16 a[232:235], v[88:91], v[120:123], a[232:235]\n\t"
"v_mfma_f32_16x16x32_f16 a[228:231], v[88:91], v[116:119], a[228:231]\n\t"
"buffer_load_dwordx4 v[220:223], v63, %[srdW], s41 offen\n\t"
"v_mfma_f32_16x16x32_f16 a[244:247], v[92:95], v[116:119], a[244:247]\n\t"
"ds_read_b128 v[76:79], v58 offset:35840\n\t"
"s_sub_u32 s80, s80, 1\n\t"
"s_getpc_b64 s[68:69]\n\t"
"s_cbranch_scc1 L_hnd_%=\n\t"
"v_mfma_f32_16x16x32_f16 a[240:243], v[92:95], v[112:115], a[240:243]\n\t"
"v_mfma_f32_16x16x32_f16 a[224:227], v[88:91], v[112:115], a[224:227]\n\t"
"s_waitcnt lgkmcnt(4)\n\t"
"v_mfma_f32_16x16x32_f16 a[0:3], v[64:67], v[96:99], a[0:3]\n\t"
"ds_read_b128 v[80:83], v58 offset:36864\n\t"
"s_waitcnt vmcnt(23)\n\t"
"v_readfirstlane_b32 s89, v43\n\t"
"s_cmp_ge_u32 s89, s71\n\t"
"s_cbranch_scc0 L_slow1_%=\n\t"
"L_sback1_%=:\n\t"
"s_waitcnt lgkmcnt(3)\n\t"
"v_mfma_f32_16x16x32_f16 a[16:19], v[68:71], v[96:99], a[16:19]\n\t"
"s_waitcnt lgkmcnt(4)\n\t"
"v_readfirstlane_b32 s49, v54\n\t"
"v_readfirstlane_b32 s70, v55\n\t"
"s_lshl_b32 s46, s49, 14\n\t"
"s_lshl_b32 s49, s49, 20\n\t"
"v_mfma_f32_16x16x32_f16 a[20:23], v[68:71], v[100:103], a[20:23]\n\t"
"s_add_u32 s40, s49, s50\n\t"
"s_add_u32 s41, s49, s51\n\t"
"s_waitcnt vmcnt(46)\n\t"
"v_cvt_pk_f16_f32 v112, v224, v228\n\t"
"v_mfma_f32_16x16x32_f16 a[4:7], v[64:67], v[100:103], a[4:7]\n\t"
"s_add_u32 s44, s49, s52\n\t"
"s_add_u32 s45, s49, s53\n\t"
"v_cvt_pk_f16_f32 v116, v225, v229\n\t"
"v_mfma_f32_16x16x32_f16 a[8:11], v[64:67], v[104:107], a[8:11]\n\t"
"ds_read_b128 v[84:87], v58 offset:37888\n\t"
"v_cvt_pk_f16_f32 v120, v226, v230\n\t"
"v_mfma_f32_16x16x32_f16 a[24:27], v[68:71], v[104:107], a[24:27]\n\t"
"v_cvt_pk_f16_f32 v124, v227, v231\n\t"
"v_mfma_f32_16x16x32_f16 a[28:31], v[68:71], v[108:111], a[28:31]\n\t"
"buffer_load_dwordx4 v[224:227], v60, %[srdW], s42 offen\n\t"
"v_mfma_f32_16x16x32_f16 a[12:15], v[64:67], v[108:111], a[12:15]\n\t"
"buffer_load_dword v43, off, %[srdC], s70 sc1\n\t"
"s_waitcnt lgkmcnt(3)\n\t"
"v_mfma_f32_16x16x32_f16 a[44:47], v[72:75], v[108:111], a[44:47]\n\t"
"ds_read_b128 v[88:91], v58 offset:38912\n\t"
"s_add_u32 s62, s46, s54\n\t"
"s_add_u32 s49, s47, s54\n\t"
"s_add_u32 m0, s49, 0\n\t"
"s_waitcnt lgkmcnt(3)\n\t"
"v_mfma_f32_16x16x32_f16 a[60:63], v[76:79], v[108:111], a[60:63]\n\t"
"buffer_load_dwordx4 v59, s[76:79], s62 offen sc1 lds\n\t"
"v_mfma_f32_16x16x32_f16 a[56:59], v[76:79], v[104:107], a[56:59]\n\t"
"v_mfma_f32_16x16x32_f16 a[40:43], v[72:75], v[104:107], a[40:43]\n\t"
"v_mfma_f32_16x16x32_f16 a[36:39], v[72:75], v[100:103], a[36:39]\n\t"
"ds_read_b128 v[92:95], v58 offset:39936\n\t"
"buffer_load_dwordx4 v[228:231], v61, %[srdW], s42 offen\n\t"
"v_mfma_f32_16x16x32_f16 a[52:55], v[76:79], v[100:103], a[52:55]\n\t"
"v_mfma_f32_16x16x32_f16 a[48:51], v[76:79], v[96:99], a[48:51]\n\t"
"v_mfma_f32_16x16x32_f16 a[32:35], v[72:75], v[96:99], a[32:35]\n\t"
"s_waitcnt lgkmcnt(3)\n\t"
"v_mfma_f32_16x16x32_f16 a[64:67], v[80:83], v[96:99], a[64:67]\n\t"
"ds_read_b128 v[64:67], v58 offset:40960\n\t"
"s_waitcnt lgkmcnt(3)\n\t"
"v_mfma_f32_16x16x32_f16 a[80:83], v[84:87], v[96:99], a[80:83]\n\t"
"v_mfma_f32_16x16x32_f16 a[84:87], v[84:87], v[100:103], a[84:87]\n\t"
"s_waitcnt vmcnt(47)\n\t"
"v_cvt_pk_f16_f32 v113, v232, v236\n\t"
"v_mfma_f32_16x16x32_f16 a[68:71], v[80:83], v[100:103], a[68:71]\n\t"
"v_cvt_pk_f16_f32 v117, v233, v237\n\t"
"v_mfma_f32_16x16x32_f16 a[72:75], v[80:83], v[104:107], a[72:75]\n\t"
"ds_read_b128 v[68:71], v58 offset:41984\n\t"
"v_cvt_pk_f16_f32 v121, v234, v238\n\t"
"v_mfma_f32_16x16x32_f16 a[88:91], v[84:87], v[104:107], a[88:91]\n\t"
"v_cvt_pk_f16_f32 v125, v235, v239\n\t"
"v_mfma_f32_16x16x32_f16 a[92:95], v[84:87], v[108:111], a[92:95]\n\t"
"buffer_load_dwordx4 v[232:235], v62, %[srdW], s42 offen\n\t"
"v_mfma_f32_16x16x32_f16 a[76:79], v[80:83], v[108:111], a[76:79]\n\t"
"s_waitcnt lgkmcnt(3)\n\t"
"v_mfma_f32_16x16x32_f16 a[108:111], v[88:91], v[108:111], a[108:111]\n\t"
"ds_read_b128 v[72:75], v58 offset:43008\n\t"
"s_add_u32 s62, s46, s55\n\t"
"s_add_u32 s49, s47, s55\n\t"
"s_add_u32 m0, s49, 0\n\t"
"s_waitcnt lgkmcnt(3)\n\t"
"v_mfma_f32_16x16x32_f16 a[124:127], v[92:95], v[108:111], a[124:127]\n\t"
"buffer_load_dwordx4 v59, s[76:79], s62 offen sc1 lds\n\t"
"v_mfma_f32_16x16x32_f16 a[120:123], v[92:95], v[104:107], a[120:123]\n\t"
"v_mfma_f32_16x16x32_f16 a[104:107], v[88:91], v[104:107], a[104:107]\n\t"
"v_mfma_f32_16x16x32_f16 a[100:103], v[88:91], v[100:103], a[100:103]\n\t"
"ds_read_b128 v[76:79], v58 offset:44032\n\t"
"buffer_load_dwordx4 v[236:239], v63, %[srdW], s42 offen\n\t"
"v_mfma_f32_16x16x32_f16 a[116:119], v[92:95], v[100:103], a[116:119]\n\t"
"v_mfma_f32_16x16x32_f16 a[112:115], v[92:95], v[96:99], a[112:115]\n\t"
"v_mfma_f32_16x16x32_f16 a[96:99], v[88:91], v[96:99], a[96:99]\n\t"
"s_waitcnt lgkmcnt(3)\n\t"
"v_mfma_f32_16x16x32_f16 a[128:131], v[64:67], v[96:99], a[128:131]\n\t"
"ds_read_b128 v[80:83], v58 offset:45056\n\t"
"s_waitcnt lgkmcnt(3)\n\t"
"v_mfma_f32_16x16x32_f16 a[144:147], v[68:71], v[96:99], a[144:147]\n\t"
"v_mfma_f32_16x16x32_f16 a[148:151], v[68:71], v[100:103], a[148:151]\n\t"
"s_waitcnt vmcnt(47)\n\t"
"v_cvt_pk_f16_f32 v114, v240, v244\n\t"
"v_mfma_f32_16x16x32_f16 a[132:135], v[64:67], v[100:103], a[132:135]\n\t"
"v_cvt_pk_f16_f32 v118, v241, v245\n\t"
"v_mfma_f32_16x16x32_f16 a[136:139], v[64:67], v[104:107], a[136:139]\n\t"
"ds_read_b128 v[84:87], v58 offset:46080\n\t"
"v_cvt_pk_f16_f32 v122, v242, v246\n\t"
"v_mfma_f32_16x16x32_f16 a[152:155], v[68:71], v[104:107], a[152:155]\n\t"
"v_cvt_pk_f16_f32 v126, v243, v247\n\t"
"v_mfma_f32_16x16x32_f16 a[156:159], v[68:71], v[108:111], a[156:159]\n\t"
"buffer_load_dwordx4 v[240:243], v60, %[srdW], s43 offen\n\t"
"v_mfma_f32_16x16x32_f16 a[140:143], v[64:67], v[108:111], a[140:143]\n\t"
"s_waitcnt lgkmcnt(3)\n\t"
"v_mfma_f32_16x16x32_f16 a[172:175], v[72:75], v[108:111], a[172:175]\n\t"
"ds_read_b128 v[88:91], v58 offset:47104\n\t"
"s_add_u32 s62, s46, s56\n\t"
"s_add_u32 s49, s47, s56\n\t"
"s_add_u32 m0, s49, 0\n\t"
"s_waitcnt lgkmcnt(3)\n\t"
"v_mfma_f32_16x16x32_f16 a[188:191], v[76:79], v[108:111], a[188:191]\n\t"
"buffer_load_dwordx4 v59, s[76:79], s62 offen sc1 lds\n\t"
"v_mfma_f32_16x16x32_f16 a[184:187], v[76:79], v[104:107], a[184:187]\n\t"
"v_mfma_f32_16x16x32_f16 a[168:171], v[72:75], v[104:107], a[168:171]\n\t"
"v_mfma_f32_16x16x32_f16 a[164:167], v[72:75], v[100:103], a[164:167]\n\t"
"ds_read_b128 v[92:95], v58 offset:48128\n\t"
"buffer_load_dwordx4 v[244:247], v61, %[srdW], s43 offen\n\t"
"v_mfma_f32_16x16x32_f16 a[180:183], v[76:79], v[100:103], a[180:183]\n\t"
"v_mfma_f32_16x16x32_f16 a[176:179], v[76:79], v[96:99], a[176:179]\n\t"
"v_mfma_f32_16x16x32_f16 a[160:163], v[72:75], v[96:99], a[160:163]\n\t"
"s_waitcnt lgkmcnt(3)\n\t"
"v_mfma_f32_16x16x32_f16 a[192:195], v[80:83], v[96:99], a[192:195]\n\t"
"ds_read_b128 v[64:67], v58 offset:49152\n\t"
"s_waitcnt lgkmcnt(3)\n\t"
"v_mfma_f32_16x16x32_f16 a[208:211], v[84:87], v[96:99], a[208:211]\n\t"
"v_mfma_f32_16x16x32_f16 a[212:215], v[84:87], v[100:103], a[212:215]\n\t"
"s_waitcnt vmcnt(47)\n\t"
"v_cvt_pk_f16_f32 v115, v248, v252\n\t"
"v_mfma_f32_16x16x32_f16 a[196:199], v[80:83], v[100:103], a[196:199]\n\t"
"v_cvt_pk_f16_f32 v119, v249, v253\n\t"
"v_mfma_f32_16x16x32_f16 a[200:203], v[80:83], v[104:107], a[200:203]\n\t"
"ds_read_b128 v[68:71], v58 offset:50176\n\t"
"v_cvt_pk_f16_f32 v123, v250, v254\n\t"
"v_mfma_f32_16x16x32_f16 a[216:219], v[84:87], v[104:107], a[216:219]\n\t"
"v_cvt_pk_f16_f32 v127, v251, v255\n\t"
"v_mfma_f32_16x16x32_f16 a[220:223], v[84:87], v[108:111], a[220:223]\n\t"
"buffer_load_dwordx4 v[248:251], v62, %[srdW], s43 offen\n\t"
"v_mfma_f32_16x16x32_f16 a[204:207], v[80:83], v[108:111], a[204:207]\n\t"
"s_waitcnt lgkmcnt(3)\n\t"
"v_mfma_f32_16x16x32_f16 a[236:239], v[88:91], v[108:111], a[236:239]\n\t"
"ds_read_b128 v[72:75], v58 offset:51200\n\t"
"s_add_u32 s62, s46, s57\n\t"
"s_add_u32 s49, s47, s57\n\t"
"s_add_u32 m0, s49, 0\n\t"
"s_waitcnt lgkmcnt(3)\n\t"
"v_mfma_f32_16x16x32_f16 a[252:255], v[92:95], v[108:111], a[252:255]\n\t"
"buffer_load_dwordx4 v59, s[76:79], s62 offen sc1 lds\n\t"
"v_mfma_f32_16x16x32_f16 a[248:251], v[92:95], v[104:107], a[248:251]\n\t"
"v_mfma_f32_16x16x32_f16 a[232:235], v[88:91], v[104:107], a[232:235]\n\t"
"v_mfma_f32_16x16x32_f16 a[228:231], v[88:91], v[100:103], a[228:231]\n\t"
"ds_read_b128 v[76:79], v58 offset:52224\n\t"
"buffer_load_dwordx4 v[252:255], v63, %[srdW], s43 offen\n\t"
"v_mfma_f32_16x16x32_f16 a[244:247], v[92:95], v[100:103], a[244:247]\n\t"
"s_sub_u32 s80, s80, 1\n\t"
"s_getpc_b64 s[68:69]\n\t"
"s_cbranch_scc1 L_hnd_%=\n\t"
"v_mfma_f32_16x16x32_f16 a[240:243], v[92:95], v[96:99], a[240:243]\n\t"
"v_mfma_f32_16x16x32_f16 a[224:227], v[88:91], v[96:99], a[224:227]\n\t"
"s_waitcnt lgkmcnt(3)\n\t"
"v_mfma_f32_16x16x32_f16 a[0:3], v[64:67], v[112:115], a[0:3]\n\t"
"ds_read_b128 v[80:83], v58 offset:53248\n\t"
"s_waitcnt lgkmcnt(3)\n\t"
"v_mfma_f32_16x16x32_f16 a[16:19], v[68:71], v[112:115], a[16:19]\n\t"
"s_mov_b32 s42, s44\n\t"
"s_mov_b32 s43, s45\n\t"
"v_mfma_f32_16x16x32_f16 a[20:23], v[68:71], v[116:119], a[20:23]\n\t"
"s_waitcnt vmcnt(47)\n\t"
"v_cvt_pk_f16_f32 v96, v128, v132\n\t"
"v_mfma_f32_16x16x32_f16 a[4:7], v[64:67], v[116:119], a[4:7]\n\t"
"v_cvt_pk_f16_f32 v100, v129, v133\n\t"
"v_mfma_f32_16x16x32_f16 a[8:11], v[64:67], v[120:123], a[8:11]\n\t"
"ds_read_b128 v[84:87], v58 offset:54272\n\t"
"v_cvt_pk_f16_f32 v104, v130, v134\n\t"
"v_mfma_f32_16x16x32_f16 a[24:27], v[68:71], v[120:123], a[24:27]\n\t"
"v_cvt_pk_f16_f32 v108, v131, v135\n\t"
"v_mfma_f32_16x16x32_f16 a[28:31], v[68:71], v[124:127], a[28:31]\n\t"
"buffer_load_dwordx4 v[128:131], v60, %[srdW], s40 offen\n\t"
"v_mfma_f32_16x16x32_f16 a[12:15], v[64:67], v[124:127], a[12:15]\n\t"
"s_waitcnt lgkmcnt(3)\n\t"
"v_mfma_f32_16x16x32_f16 a[44:47], v[72:75], v[124:127], a[44:47]\n\t"
"ds_read_b128 v[88:91], v58 offset:55296\n\t"
"s_add_u32 s62, s46, s58\n\t"
"s_add_u32 s49, s47, s58\n\t"
"s_add_u32 m0, s49, 0\n\t"
"s_waitcnt lgkmcnt(3)\n\t"
"v_mfma_f32_16x16x32_f16 a[60:63], v[76:79], v[124:127], a[60:63]\n\t"
"buffer_load_dwordx4 v59, s[76:79], s62 offen sc1 lds\n\t"
"v_mfma_f32_16x16x32_f16 a[56:59], v[76:79], v[120:123], a[56:59]\n\t"
"v_mfma_f32_16x16x32_f16 a[40:43], v[72:75], v[120:123], a[40:43]\n\t"
"v_mfma_f32_16x16x32_f16 a[36:39], v[72:75], v[116:119], a[36:39]\n\t"
"ds_read_b128 v[92:95], v58 offset:56320\n\t"
"buffer_load_dwordx4 v[132:135], v61, %[srdW], s40 offen\n\t"
"v_mfma_f32_16x16x32_f16 a[52:55], v[76:79], v[116:119], a[52:55]\n\t"
"v_mfma_f32_16x16x32_f16 a[48:51], v[76:79], v[112:115], a[48:51]\n\t"
"v_mfma_f32_16x16x32_f16 a[32:35], v[72:75], v[112:115], a[32:35]\n\t"
"s_waitcnt lgkmcnt(3)\n\t"
"v_mfma_f32_16x16x32_f16 a[64:67], v[80:83], v[112:115], a[64:67]\n\t"
"ds_read_b128 v[64:67], v58 offset:57344\n\t"
"s_waitcnt lgkmcnt(3)\n\t"
"v_mfma_f32_16x16x32_f16 a[80:83], v[84:87], v[112:115], a[80:83]\n\t"
"v_mfma_f32_16x16x32_f16 a[84:87], v[84:87], v[116:119], a[84:87]\n\t"
"s_waitcnt vmcnt(47)\n\t"
"v_cvt_pk_f16_f32 v97, v136, v140\n\t"
"v_mfma_f32_16x16x32_f16 a[68:71], v[80:83], v[116:119], a[68:71]\n\t"
"v_cvt_pk_f16_f32 v101, v137, v141\n\t"
"v_mfma_f32_16x16x32_f16 a[72:75], v[80:83], v[120:123], a[72:75]\n\t"
"ds_read_b128 v[68:71], v58 offset:58368\n\t"
"v_cvt_pk_f16_f32 v105, v138, v142\n\t"
"v_mfma_f32_16x16x32_f16 a[88:91], v[84:87], v[120:123], a[88:91]\n\t"
"v_cvt_pk_f16_f32 v109, v139, v143\n\t"
"v_mfma_f32_16x16x32_f16 a[92:95], v[84:87], v[124:127], a[92:95]\n\t"
"buffer_load_dwordx4 v[136:139], v62, %[srdW], s40 offen\n\t"
"v_mfma_f32_16x16x32_f16 a[76:79], v[80:83], v[124:127], a[76:79]\n\t"
"s_waitcnt lgkmcnt(3)\n\t"
"v_mfma_f32_16x16x32_f16 a[108:111], v[88:91], v[124:127], a[108:111]\n\t"
"ds_read_b128 v[72:75], v58 offset:59392\n\t"
"s_add_u32 s62, s46, s59\n\t"
"s_add_u32 s49, s47, s59\n\t"
"s_add_u32 m0, s49, 0\n\t"
"s_waitcnt lgkmcnt(3)\n\t"
"v_mfma_f32_16x16x32_f16 a[124:127], v[92:95], v[124:127], a[124:127]\n\t"
"buffer_load_dwordx4 v59, s[76:79], s62 offen sc1 lds\n\t"
"v_mfma_f32_16x16x32_f16 a[120:123], v[92:95], v[120:123], a[120:123]\n\t"
"v_mfma_f32_16x16x32_f16 a[104:107], v[88:91], v[120:123], a[104:107]\n\t"
"v_mfma_f32_16x16x32_f16 a[100:103], v[88:91], v[116:119], a[100:103]\n\t"
"ds_read_b128 v[76:79], v58 offset:60416\n\t"
"buffer_load_dwordx4 v[140:143], v63, %[srdW], s40 offen\n\t"
"v_mfma_f32_16x16x32_f16 a[116:119], v[92:95], v[116:119], a[116:119]\n\t"
"v_mfma_f32_16x16x32_f16 a[112:115], v[92:95], v[112:115], a[112:115]\n\t"
"v_mfma_f32_16x16x32_f16 a[96:99], v[88:91], v[112:115], a[96:99]\n\t"
"s_waitcnt lgkmcnt(3)\n\t"
"v_mfma_f32_16x16x32_f16 a[128:131], v[64:67], v[112:115], a[128:131]\n\t"
"ds_read_b128 v[80:83], v58 offset:61440\n\t"
"s_waitcnt lgkmcnt(3)\n\t"
"v_mfma_f32_16x16x32_f16 a[144:147], v[68:71], v[112:115], a[144:147]\n\t"
"v_mfma_f32_16x16x32_f16 a[148:151], v[68:71], v[116:119], a[148:151]\n\t"
"s_waitcnt vmcnt(47)\n\t"
"v_cvt_pk_f16_f32 v98, v144, v148\n\t"
"v_mfma_f32_16x16x32_f16 a[132:135], v[64:67], v[116:119], a[132:135]\n\t"
"v_cvt_pk_f16_f32 v102, v145, v149\n\t"
"v_mfma_f32_16x16x32_f16 a[136:139], v[64:67], v[120:123], a[136:139]\n\t"
"ds_read_b128 v[84:87], v58 offset:62464\n\t"
"v_cvt_pk_f16_f32 v106, v146, v150\n\t"
"v_mfma_f32_16x16x32_f16 a[152:155], v[68:71], v[120:123], a[152:155]\n\t"
"v_cvt_pk_f16_f32 v110, v147, v151\n\t"
"v_mfma_f32_16x16x32_f16 a[156:159], v[68:71], v[124:127], a[156:159]\n\t"
"buffer_load_dwordx4 v[144:147], v60, %[srdW], s41 offen\n\t"
"v_mfma_f32_16x16x32_f16 a[140:143], v[64:67], v[124:127], a[140:143]\n\t"
"s_waitcnt lgkmcnt(3)\n\t"
"v_mfma_f32_16x16x32_f16 a[172:175], v[72:75], v[124:127], a[172:175]\n\t"
"ds_read_b128 v[88:91], v58 offset:63488\n\t"
"s_add_u32 s62, s46, s60\n\t"
"s_add_u32 s49, s47, s60\n\t"
"s_add_u32 m0, s49, 0\n\t"
"s_waitcnt lgkmcnt(3)\n\t"
"v_mfma_f32_16x16x32_f16 a[188:191], v[76:79], v[124:127], a[188:191]\n\t"
"buffer_load_dwordx4 v59, s[76:79], s62 offen sc1 lds\n\t"
"v_mfma_f32_16x16x32_f16 a[184:187], v[76:79], v[120:123], a[184:187]\n\t"
"v_mfma_f32_16x16x32_f16 a[168:171], v[72:75], v[120:123], a[168:171]\n\t"
"v_mfma_f32_16x16x32_f16 a[164:167], v[72:75], v[116:119], a[164:167]\n\t"
"ds_read_b128 v[92:95], v58 offset:64512\n\t"
"buffer_load_dwordx4 v[148:151], v61, %[srdW], s41 offen\n\t"
"v_mfma_f32_16x16x32_f16 a[180:183], v[76:79], v[116:119], a[180:183]\n\t"
"v_mfma_f32_16x16x32_f16 a[176:179], v[76:79], v[112:115], a[176:179]\n\t"
"v_mfma_f32_16x16x32_f16 a[160:163], v[72:75], v[112:115], a[160:163]\n\t"
"s_waitcnt lgkmcnt(3)\n\t"
"v_mfma_f32_16x16x32_f16 a[192:195], v[80:83], v[112:115], a[192:195]\n\t"
"s_waitcnt lgkmcnt(2)\n\t"
"v_mfma_f32_16x16x32_f16 a[208:211], v[84:87], v[112:115], a[208:211]\n\t"
"s_memrealtime s[90:91]\n\t"
"v_mfma_f32_16x16x32_f16 a[212:215], v[84:87], v[116:119], a[212:215]\n\t"
"s_waitcnt vmcnt(47)\n\t"
"v_cvt_pk_f16_f32 v99, v152, v156\n\t"
"v_mfma_f32_16x16x32_f16 a[196:199], v[80:83], v[116:119], a[196:199]\n\t"
"v_cvt_pk_f16_f32 v103, v153, v157\n\t"
"s_waitcnt vmcnt(47) lgkmcnt(0)\n\t"
"s_sub_u32 s89, s90, s65\n\t"
"s_cmp_lt_i32 s89, 0\n\t"
"s_getpc_b64 s[68:69]\n\t"
"s_cbranch_scc1 L_pace_%=\n\t"
"s_ashr_i32 s89, s95, 3\n\t"
"s_add_u32 s65, s90, s89\n\t"
"s_sub_u32 s95, s95, %[sslope]\n\t"
"s_barrier\n\t"
"v_mfma_f32_16x16x32_f16 a[200:203], v[80:83], v[120:123], a[200:203]\n\t"
"ds_read_b128 v[64:67], v57 offset:0\n\t"
"v_cvt_pk_f16_f32 v107, v154, v158\n\t"
"v_mfma_f32_16x16x32_f16 a[216:219], v[84:87], v[120:123], a[216:219]\n\t"
"ds_read_b64 v[54:55], v53\n\t"
"v_add_u32 v53, 8, v53\n\t"
"v_cvt_pk_f16_f32 v111, v155, v159\n\t"
"v_mfma_f32_16x16x32_f16 a[220:223], v[84:87], v[124:127], a[220:223]\n\t"
"buffer_load_dwordx4 v[152:155], v62, %[srdW], s41 offen\n\t"
"v_mfma_f32_16x16x32_f16 a[204:207], v[80:83], v[124:127], a[204:207]\n\t"
"ds_read_b128 v[68:71], v57 offset:1024\n\t"
"s_waitcnt lgkmcnt(4)\n\t"
"v_mfma_f32_16x16x32_f16 a[236:239], v[88:91], v[124:127], a[236:239]\n\t"
"s_add_u32 s62, s46, s61\n\t"
"s_add_u32 s49, s47, s61\n\t"
"s_add_u32 m0, s49, 0\n\t"
"s_waitcnt lgkmcnt(3)\n\t"
"v_mfma_f32_16x16x32_f16 a[252:255], v[92:95], v[124:127], a[252:255]\n\t"
"buffer_load_dwordx4 v59, s[76:79], s62 offen sc1 lds\n\t"
"v_mfma_f32_16x16x32_f16 a[248:251], v[92:95], v[120:123], a[248:251]\n\t"
"ds_read_b128 v[72:75], v57 offset:2048\n\t"
"v_mfma_f32_16x16x32_f16 a[232:235], v[88:91], v[120:123], a[232:235]\n\t"
"v_mfma_f32_16x16x32_f16 a[228:231], v[88:91], v[116:119], a[228:231]\n\t"
"buffer_load_dwordx4 v[156:159], v63, %[srdW], s41 offen\n\t"
"v_mfma_f32_16x16x32_f16 a[244:247], v[92:95], v[116:119], a[244:247]\n\t"
"ds_read_b128 v[76:79], v57 offset:3072\n\t"
"s_sub_u32 s80, s80, 1\n\t"
"s_getpc_b64 s[68:69]\n\t"
"s_cbranch_scc1 L_hnd_%=\n\t"
"v_mfma_f32_16x16x32_f16 a[240:243], v[92:95], v[112:115], a[240:243]\n\t"
"v_mfma_f32_16x16x32_f16 a[224:227], v[88:91], v[112:115], a[224:227]\n\t"
"s_waitcnt lgkmcnt(4)\n\t"
"v_mfma_f32_16x16x32_f16 a[0:3], v[64:67], v[96:99], a[0:3]\n\t"
"ds_read_b128 v[80:83], v57 offset:4096\n\t"
"s_waitcnt vmcnt(23)\n\t"
"v_readfirstlane_b32 s89, v43\n\t"
"s_cmp_ge_u32 s89, s71\n\t"
"s_cbranch_scc0 L_slow2_%=\n\t"
"L_sback2_%=:\n\t"
"s_waitcnt lgkmcnt(3)\n\t"
"v_mfma_f32_16x16x32_f16 a[16:19], v[68:71], v[96:99], a[16:19]\n\t"
"s_waitcnt lgkmcnt(4)\n\t"
"v_readfirstlane_b32 s49, v54\n\t"
"v_readfirstlane_b32 s70, v55\n\t"
"s_lshl_b32 s46, s49, 14\n\t"
"s_lshl_b32 s49, s49, 20\n\t"
"v_mfma_f32_16x16x32_f16 a[20:23], v[68:71], v[100:103], a[20:23]\n\t"
"s_add_u32 s40, s49, s50\n\t"
"s_add_u32 s41, s49, s51\n\t"
"s_waitcnt vmcnt(46)\n\t"
"v_cvt_pk_f16_f32 v112, v160, v164\n\t"
"v_mfma_f32_16x16x32_f16 a[4:7], v[64:67], v[100:103], a[4:7]\n\t"
"s_add_u32 s44, s49, s52\n\t"
"s_add_u32 s45, s49, s53\n\t"
"v_cvt_pk_f16_f32 v116, v161, v165\n\t"
"v_mfma_f32_16x16x32_f16 a[8:11], v[64:67], v[104:107], a[8:11]\n\t"
"ds_read_b128 v[84:87], v57 offset:5120\n\t"
"v_cvt_pk_f16_f32 v120, v162, v166\n\t"
"v_mfma_f32_16x16x32_f16 a[24:27], v[68:71], v[104:107], a[24:27]\n\t"
"v_cvt_pk_f16_f32 v124, v163, v167\n\t"
"v_mfma_f32_16x16x32_f16 a[28:31], v[68:71], v[108:111], a[28:31]\n\t"
"buffer_load_dwordx4 v[160:163], v60, %[srdW], s42 offen\n\t"
"v_mfma_f32_16x16x32_f16 a[12:15], v[64:67], v[108:111], a[12:15]\n\t"
"buffer_load_dword v43, off, %[srdC], s70 sc1\n\t"
"s_waitcnt lgkmcnt(3)\n\t"
"v_mfma_f32_16x16x32_f16 a[44:47], v[72:75], v[108:111], a[44:47]\n\t"
"ds_read_b128 v[88:91], v57 offset:6144\n\t"
"s_add_u32 s62, s46, s54\n\t"
"s_add_u32 s49, s47, s54\n\t"
"s_add_u32 m0, s49, 32768\n\t"
"s_waitcnt lgkmcnt(3)\n\t"
"v_mfma_f32_16x16x32_f16 a[60:63], v[76:79], v[108:111], a[60:63]\n\t"
"buffer_load_dwordx4 v59, s[76:79], s62 offen sc1 lds\n\t"
"v_mfma_f32_16x16x32_f16 a[56:59], v[76:79], v[104:107], a[56:59]\n\t"
"v_mfma_f32_16x16x32_f16 a[40:43], v[72:75], v[104:107], a[40:43]\n\t"
"v_mfma_f32_16x16x32_f16 a[36:39], v[72:75], v[100:103], a[36:39]\n\t"
"ds_read_b128 v[92:95], v57 offset:7168\n\t"
"buffer_load_dwordx4 v[164:167], v61, %[srdW], s42 offen\n\t"
"v_mfma_f32_16x16x32_f16 a[52:55], v[76:79], v[100:103], a[52:55]\n\t"
"v_mfma_f32_16x16x32_f16 a[48:51], v[76:79], v[96:99], a[48:51]\n\t"
"v_mfma_f32_16x16x32_f16 a[32:35], v[72:75], v[96:99], a[32:35]\n\t"
"s_waitcnt lgkmcnt(3)\n\t"
"v_mfma_f32_16x16x32_f16 a[64:67], v[80:83], v[96:99], a[64:67]\n\t"
"ds_read_b128 v[64:67], v57 offset:8192\n\t"
"s_waitcnt lgkmcnt(3)\n\t"
"v_mfma_f32_16x16x32_f16 a[80:83], v[84:87], v[96:99], a[80:83]\n\t"
"v_mfma_f32_16x16x32_f16 a[84:87], v[84:87], v[100:103], a[84:87]\n\t"
"s_waitcnt vmcnt(47)\n\t"
"v_cvt_pk_f16_f32 v113, v168, v172\n\t"
"v_mfma_f32_16x16x32_f16 a[68:71], v[80:83], v[100:103], a[68:71]\n\t"
"v_cvt_pk_f16_f32 v117, v169, v173\n\t"
"v_mfma_f32_16x16x32_f16 a[72:75], v[80:83], v[104:107], a[72:75]\n\t"
"ds_read_b128 v[68:71], v57 offset:9216\n\t"
"v_cvt_pk_f16_f32 v121, v170, v174\n\t"
"v_mfma_f32_16x16x32_f16 a[88:91], v[84:87], v[104:107], a[88:91]\n\t"
"v_cvt_pk_f16_f32 v125, v171, v175\n\t"
"v_mfma_f32_16x16x32_f16 a[92:95], v[84:87], v[108:111], a[92:95]\n\t"
"buffer_load_dwordx4 v[168:171], v62, %[srdW], s42 offen\n\t"
"v_mfma_f32_16x16x32_f16 a[76:79], v[80:83], v[108:111], a[76:79]\n\t"
"s_waitcnt lgkmcnt(3)\n\t"
"v_mfma_f32_16x16x32_f16 a[108:111], v[88:91], v[108:111], a[108:111]\n\t"
"ds_read_b128 v[72:75], v57 offset:10240\n\t"
"s_add_u32 s62, s46, s55\n\t"
"s_add_u32 s49, s47, s55\n\t"
"s_add_u32 m0, s49, 32768\n\t"
"s_waitcnt lgkmcnt(3)\n\t"
"v_mfma_f32_16x16x32_f16 a[124:127], v[92:95], v[108:111], a[124:127]\n\t"
"buffer_load_dwordx4 v59, s[76:79], s62 offen sc1 lds\n\t"
"v_mfma_f32_16x16x32_f16 a[120:123], v[92:95], v[104:107], a[120:123]\n\t"
"v_mfma_f32_16x16x32_f16 a[104:107], v[88:91], v[104:107], a[104:107]\n\t"
"v_mfma_f32_16x16x32_f16 a[100:103], v[88:91], v[100:103], a[100:103]\n\t"
"ds_read_b128 v[76:79], v57 offset:11264\n\t"
"buffer_load_dwordx4 v[172:175], v63, %[srdW], s42 offen\n\t"
"v_mfma_f32_16x16x32_f16 a[116:119], v[92:95], v[100:103], a[116:119]\n\t"
"v_mfma_f32_16x16x32_f16 a[112:115], v[92:95], v[96:99], a[112:115]\n\t"
"v_mfma_f32_16x16x32_f16 a[96:99], v[88:91], v[96:99], a[96:99]\n\t"
"s_waitcnt lgkmcnt(3)\n\t"
"v_mfma_f32_16x16x32_f16 a[128:131], v[64:67], v[96:99], a[128:131]\n\t"
"ds_read_b128 v[80:83], v57 offset:12288\n\t"
"s_waitcnt lgkmcnt(3)\n\t"
"v_mfma_f32_16x16x32_f16 a[144:147], v[68:71], v[96:99], a[144:147]\n\t"
"v_mfma_f32_16x16x32_f16 a[148:151], v[68:71], v[100:103], a[148:151]\n\t"
"s_waitcnt vmcnt(47)\n\t"
"v_cvt_pk_f16_f32 v114, v176, v180\n\t"
"v_mfma_f32_16x16x32_f16 a[132:135], v[64:67], v[100:103], a[132:135]\n\t"
"v_cvt_pk_f16_f32 v118, v177, v181\n\t"
"v_mfma_f32_16x16x32_f16 a[136:139], v[64:67], v[104:107], a[136:139]\n\t"
"ds_read_b128 v[84:87], v57 offset:13312\n\t"
"v_cvt_pk_f16_f32 v122, v178, v182\n\t"
"v_mfma_f32_16x16x32_f16 a[152:155], v[68:71], v[104:107], a[152:155]\n\t"
"v_cvt_pk_f16_f32 v126, v179, v183\n\t"
"v_mfma_f32_16x16x32_f16 a[156:159], v[68:71], v[108:111], a[156:159]\n\t"
"buffer_load_dwordx4 v[176:179], v60, %[srdW], s43 offen\n\t"
"v_mfma_f32_16x16x32_f16 a[140:143], v[64:67], v[108:111], a[140:143]\n\t"
"s_waitcnt lgkmcnt(3)\n\t"
"v_mfma_f32_16x16x32_f16 a[172:175], v[72:75], v[108:111], a[172:175]\n\t"
"ds_read_b128 v[88:91], v57 offset:14336\n\t"
"s_add_u32 s62, s46, s56\n\t"
"s_add_u32 s49, s47, s56\n\t"
"s_add_u32 m0, s49, 32768\n\t"
"s_waitcnt lgkmcnt(3)\n\t"
"v_mfma_f32_16x16x32_f16 a[188:191], v[76:79], v[108:111], a[188:191]\n\t"
"buffer_load_dwordx4 v59, s[76:79], s62 offen sc1 lds\n\t"
"v_mfma_f32_16x16x32_f16 a[184:187], v[76:79], v[104:107], a[184:187]\n\t"
"v_mfma_f32_16x16x32_f16 a[168:171], v[72:75], v[104:107], a[168:171]\n\t"
"v_mfma_f32_16x16x32_f16 a[164:167], v[72:75], v[100:103], a[164:167]\n\t"
"ds_read_b128 v[92:95], v57 offset:15360\n\t"
"buffer_load_dwordx4 v[180:183], v61, %[srdW], s43 offen\n\t"
"v_mfma_f32_16x16x32_f16 a[180:183], v[76:79], v[100:103], a[180:183]\n\t"
"v_mfma_f32_16x16x32_f16 a[176:179], v[76:79], v[96:99], a[176:179]\n\t"
"v_mfma_f32_16x16x32_f16 a[160:163], v[72:75], v[96:99], a[160:163]\n\t"
"s_waitcnt lgkmcnt(3)\n\t"
"v_mfma_f32_16x16x32_f16 a[192:195], v[80:83], v[96:99], a[192:195]\n\t"
"ds_read_b128 v[64:67], v57 offset:16384\n\t"
"s_waitcnt lgkmcnt(3)\n\t"
"v_mfma_f32_16x16x32_f16 a[208:211], v[84:87], v[96:99], a[208:211]\n\t"
"v_mfma_f32_16x16x32_f16 a[212:215], v[84:87], v[100:103], a[212:215]\n\t"
"s_waitcnt vmcnt(47)\n\t"
"v_cvt_pk_f16_f32 v115, v184, v188\n\t"
"v_mfma_f32_16x16x32_f16 a[196:199], v[80:83], v[100:103], a[196:199]\n\t"
"v_cvt_pk_f16_f32 v119, v185, v189\n\t"
"v_mfma_f32_16x16x32_f16 a[200:203], v[80:83], v[104:107], a[200:203]\n\t"
"ds_read_b128 v[68:71], v57 offset:17408\n\t"
"v_cvt_pk_f16_f32 v123, v186, v190\n\t"
"v_mfma_f32_16x16x32_f16 a[216:219], v[84:87], v[104:107], a[216:219]\n\t"
"v_cvt_pk_f16_f32 v127, v187, v191\n\t"
"v_mfma_f32_16x16x32_f16 a[220:223], v[84:87], v[108:111], a[220:223]\n\t"
"buffer_load_dwordx4 v[184:187], v62, %[srdW], s43 offen\n\t"
"v_mfma_f32_16x16x32_f16 a[204:207], v[80:83], v[108:111], a[204:207]\n\t"
"s_waitcnt lgkmcnt(3)\n\t"
"v_mfma_f32_16x16x32_f16 a[236:239], v[88:91], v[108:111], a[236:239]\n\t"
"ds_read_b128 v[72:75], v57 offset:18432\n\t"
"s_add_u32 s62, s46, s57\n\t"
"s_add_u32 s49, s47, s57\n\t"
"s_add_u32 m0, s49, 32768\n\t"
"s_waitcnt lgkmcnt(3)\n\t"
"v_mfma_f32_16x16x32_f16 a[252:255], v[92:95], v[108:111], a[252:255]\n\t"
"buffer_load_dwordx4 v59, s[76:79], s62 offen sc1 lds\n\t"
"v_mfma_f32_16x16x32_f16 a[248:251], v[92:95], v[104:107], a[248:251]\n\t"
"v_mfma_f32_16x16x32_f16 a[232:235], v[88:91], v[104:107], a[232:235]\n\t"
"v_mfma_f32_16x16x32_f16 a[228:231], v[88:91], v[100:103], a[228:231]\n\t"
"ds_read_b128 v[76:79], v57 offset:19456\n\t"
"buffer_load_dwordx4 v[188:191], v63, %[srdW], s43 offen\n\t"
"v_mfma_f32_16x16x32_f16 a[244:247], v[92:95], v[100:103], a[244:247]\n\t"
"s_sub_u32 s80, s80, 1\n\t"
"s_getpc_b64 s[68:69]\n\t"
"s_cbranch_scc1 L_hnd_%=\n\t"
"v_mfma_f32_16x16x32_f16 a[240:243], v[92:95], v[96:99], a[240:243]\n\t"
"v_mfma_f32_16x16x32_f16 a[224:227], v[88:91], v[96:99], a[224:227]\n\t"
"s_waitcnt lgkmcnt(3)\n\t"
"v_mfma_f32_16x16x32_f16 a[0:3], v[64:67], v[112:115], a[0:3]\n\t"
"ds_read_b128 v[80:83], v57 offset:20480\n\t"
"s_waitcnt lgkmcnt(3)\n\t"
"v_mfma_f32_16x16x32_f16 a[16:19], v[68:71], v[112:115], a[16:19]\n\t"
"s_mov_b32 s42, s44\n\t"
"s_mov_b32 s43, s45\n\t"
"v_mfma_f32_16x16x32_f16 a[20:23], v[68:71], v[116:119], a[20:23]\n\t"
"s_waitcnt vmcnt(47)\n\t"
"v_cvt_pk_f16_f32 v96, v192, v196\n\t"
"v_mfma_f32_16x16x32_f16 a[4:7], v[64:67], v[116:119], a[4:7]\n\t"
"v_cvt_pk_f16_f32 v100, v193, v197\n\t"
"v_mfma_f32_16x16x32_f16 a[8:11], v[64:67], v[120:123], a[8:11]\n\t"
"ds_read_b128 v[84:87], v57 offset:21504\n\t"
"v_cvt_pk_f16_f32 v104, v194, v198\n\t"
"v_mfma_f32_16x16x32_f16 a[24:27], v[68:71], v[120:123], a[24:27]\n\t"
"v_cvt_pk_f16_f32 v108, v195, v199\n\t"
"v_mfma_f32_16x16x32_f16 a[28:31], v[68:71], v[124:127], a[28:31]\n\t"
"buffer_load_dwordx4 v[192:195], v60, %[srdW], s40 offen\n\t"
"v_mfma_f32_16x16x32_f16 a[12:15], v[64:67], v[124:127], a[12:15]\n\t"
"s_waitcnt lgkmcnt(3)\n\t"
"v_mfma_f32_16x16x32_f16 a[44:47], v[72:75], v[124:127], a[44:47]\n\t"
"ds_read_b128 v[88:91], v57 offset:22528\n\t"
"s_add_u32 s62, s46, s58\n\t"
"s_add_u32 s49, s47, s58\n\t"
"s_add_u32 m0, s49, 32768\n\t"
"s_waitcnt lgkmcnt(3)\n\t"
"v_mfma_f32_16x16x32_f16 a[60:63], v[76:79], v[124:127], a[60:63]\n\t"
"buffer_load_dwordx4 v59, s[76:79], s62 offen sc1 lds\n\t"
"v_mfma_f32_16x16x32_f16 a[56:59], v[76:79], v[120:123], a[56:59]\n\t"
"v_mfma_f32_16x16x32_f16 a[40:43], v[72:75], v[120:123], a[40:43]\n\t"
"v_mfma_f32_16x16x32_f16 a[36:39], v[72:75], v[116:119], a[36:39]\n\t"
"ds_read_b128 v[92:95], v57 offset:23552\n\t"
"buffer_load_dwordx4 v[196:199], v61, %[srdW], s40 offen\n\t"
"v_mfma_f32_16x16x32_f16 a[52:55], v[76:79], v[116:119], a[52:55]\n\t"
"v_mfma_f32_16x16x32_f16 a[48:51], v[76:79], v[112:115], a[48:51]\n\t"
"v_mfma_f32_16x16x32_f16 a[32:35], v[72:75], v[112:115], a[32:35]\n\t"
"s_waitcnt lgkmcnt(3)\n\t"
"v_mfma_f32_16x16x32_f16 a[64:67], v[80:83], v[112:115], a[64:67]\n\t"
"ds_read_b128 v[64:67], v57 offset:24576\n\t"
"s_waitcnt lgkmcnt(3)\n\t"
"v_mfma_f32_16x16x32_f16 a[80:83], v[84:87], v[112:115], a[80:83]\n\t"
"v_mfma_f32_16x16x32_f16 a[84:87], v[84:87], v[116:119], a[84:87]\n\t"
"s_waitcnt vmcnt(47)\n\t"
"v_cvt_pk_f16_f32 v97, v200, v204\n\t"
"v_mfma_f32_16x16x32_f16 a[68:71], v[80:83], v[116:119], a[68:71]\n\t"
"v_cvt_pk_f16_f32 v101, v201, v205\n\t"
"v_mfma_f32_16x16x32_f16 a[72:75], v[80:83], v[120:123], a[72:75]\n\t"
"ds_read_b128 v[68:71], v57 offset:25600\n\t"
"v_cvt_pk_f16_f32 v105, v202, v206\n\t"
"v_mfma_f32_16x16x32_f16 a[88:91], v[84:87], v[120:123], a[88:91]\n\t"
"v_cvt_pk_f16_f32 v109, v203, v207\n\t"
"v_mfma_f32_16x16x32_f16 a[92:95], v[84:87], v[124:127], a[92:95]\n\t"
"buffer_load_dwordx4 v[200:203], v62, %[srdW], s40 offen\n\t"
"v_mfma_f32_16x16x32_f16 a[76:79], v[80:83], v[124:127], a[76:79]\n\t"
"s_waitcnt lgkmcnt(3)\n\t"
"v_mfma_f32_16x16x32_f16 a[108:111], v[88:91], v[124:127], a[108:111]\n\t"
"ds_read_b128 v[72:75], v57 offset:26624\n\t"
"s_add_u32 s62, s46, s59\n\t"
"s_add_u32 s49, s47, s59\n\t"
"s_add_u32 m0, s49, 32768\n\t"
"s_waitcnt lgkmcnt(3)\n\t"
"v_mfma_f32_16x16x32_f16 a[124:127], v[92:95], v[124:127], a[124:127]\n\t"
"buffer_load_dwordx4 v59, s[76:79], s62 offen sc1 lds\n\t"
"v_mfma_f32_16x16x32_f16 a[120:123], v[92:95], v[120:123], a[120:123]\n\t"
"v_mfma_f32_16x16x32_f16 a[104:107], v[88:91], v[120:123], a[104:107]\n\t"
"v_mfma_f32_16x16x32_f16 a[100:103], v[88:91], v[116:119], a[100:103]\n\t"
"ds_read_b128 v[76:79], v57 offset:27648\n\t"
"buffer_load_dwordx4 v[204:207], v63, %[srdW], s40 offen\n\t"
"v_mfma_f32_16x16x32_f16 a[116:119], v[92:95], v[116:119], a[116:119]\n\t"
"v_mfma_f32_16x16x32_f16 a[112:115], v[92:95], v[112:115], a[112:115]\n\t"
"v_mfma_f32_16x16x32_f16 a[96:99], v[88:91], v[112:115], a[96:99]\n\t"
"s_waitcnt lgkmcnt(3)\n\t"
"v_mfma_f32_16x16x32_f16 a[128:131], v[64:67], v[112:115], a[128:131]\n\t"
"ds_read_b128 v[80:83], v57 offset:28672\n\t"
"s_waitcnt lgkmcnt(3)\n\t"
"v_mfma_f32_16x16x32_f16 a[144:147], v[68:71], v[112:115], a[144:147]\n\t"
"v_mfma_f32_16x16x32_f16 a[148:151], v[68:71], v[116:119], a[148:151]\n\t"
"s_waitcnt vmcnt(47)\n\t"
"v_cvt_pk_f16_f32 v98, v208, v212\n\t"
"v_mfma_f32_16x16x32_f16 a[132:135], v[64:67], v[116:119], a[132:135]\n\t"
"v_cvt_pk_f16_f32 v102, v209, v213\n\t"
"v_mfma_f32_16x16x32_f16 a[136:139], v[64:67], v[120:123], a[136:139]\n\t"
"ds_read_b128 v[84:87], v57 offset:29696\n\t"
"v_cvt_pk_f16_f32 v106, v210, v214\n\t"
"v_mfma_f32_16x16x32_f16 a[152:155], v[68:71], v[120:123], a[152:155]\n\t"
"v_cvt_pk_f16_f32 v110, v211, v215\n\t"
"v_mfma_f32_16x16x32_f16 a[156:159], v[68:71], v[124:127], a[156:159]\n\t"
"buffer_load_dwordx4 v[208:211], v60, %[srdW], s41 offen\n\t"
"v_mfma_f32_16x16x32_f16 a[140:143], v[64:67], v[124:127], a[140:143]\n\t"
"s_waitcnt lgkmcnt(3)\n\t"
"v_mfma_f32_16x16x32_f16 a[172:175], v[72:75], v[124:127], a[172:175]\n\t"
"ds_read_b128 v[88:91], v57 offset:30720\n\t"
"s_add_u32 s62, s46, s60\n\t"
"s_add_u32 s49, s47, s60\n\t"
"s_add_u32 m0, s49, 32768\n\t"
"s_waitcnt lgkmcnt(3)\n\t"
"v_mfma_f32_16x16x32_f16 a[188:191], v[76:79], v[124:127], a[188:191]\n\t"
"buffer_load_dwordx4 v59, s[76:79], s62 offen sc1 lds\n\t"
"v_mfma_f32_16x16x32_f16 a[184:187], v[76:79], v[120:123], a[184:187]\n\t"
"v_mfma_f32_16x16x32_f16 a[168:171], v[72:75], v[120:123], a[168:171]\n\t"
"v_mfma_f32_16x16x32_f16 a[164:167], v[72:75], v[116:119], a[164:167]\n\t"
"ds_read_b128 v[92:95], v57 offset:31744\n\t"
"buffer_load_dwordx4 v[212:215], v61, %[srdW], s41 offen\n\t"
"v_mfma_f32_16x16x32_f16 a[180:183], v[76:79], v[116:119], a[180:183]\n\t"
"v_mfma_f32_16x16x32_f16 a[176:179], v[76:79], v[112:115], a[176:179]\n\t"
"v_mfma_f32_16x16x32_f16 a[160:163], v[72:75], v[112:115], a[160:163]\n\t"
"s_waitcnt lgkmcnt(3)\n\t"
"v_mfma_f32_16x16x32_f16 a[192:195], v[80:83], v[112:115], a[192:195]\n\t"
"s_waitcnt lgkmcnt(2)\n\t"
"v_mfma_f32_16x16x32_f16 a[208:211], v[84:87], v[112:115], a[208:211]\n\t"
"s_memrealtime s[90:91]\n\t"
"v_mfma_f32_16x16x32_f16 a[212:215], v[84:87], v[116:119], a[212:215]\n\t"
"s_waitcnt vmcnt(47)\n\t"
"v_cvt_pk_f16_f32 v99, v216, v220\n\t"
"v_mfma_f32_16x16x32_f16 a[196:199], v[80:83], v[116:119], a[196:199]\n\t"
"v_cvt_pk_f16_f32 v103, v217, v221\n\t"
"s_waitcnt vmcnt(48) lgkmcnt(0)\n\t"
"s_sub_u32 s89, s90, s65\n\t"
"s_cmp_lt_i32 s89, 0\n\t"
"s_getpc_b64 s[68:69]\n\t"
"s_cbranch_scc1 L_pace_%=\n\t"
"s_ashr_i32 s89, s95, 3\n\t"
"s_add_u32 s65, s90, s89\n\t"
"s_sub_u32 s95, s95, %[sslope]\n\t"
"s_barrier\n\t"
"v_mfma_f32_16x16x32_f16 a[200:203], v[80:83], v[120:123], a[200:203]\n\t"
"ds_read_b128 v[64:67], v57 offset:32768\n\t"
"v_cvt_pk_f16_f32 v107, v218, v222\n\t"
"v_mfma_f32_16x16x32_f16 a[216:219], v[84:87], v[120:123], a[216:219]\n\t"
"ds_read_b64 v[54:55], v53\n\t"
"v_add_u32 v53, 8, v53\n\t"
"v_cvt_pk_f16_f32 v111, v219, v223\n\t"
"v_mfma_f32_16x16x32_f16 a[220:223], v[84:87], v[124:127], a[220:223]\n\t"
"buffer_load_dwordx4 v[216:219], v62, %[srdW], s41 offen\n\t"
"v_mfma_f32_16x16x32_f16 a[204:207], v[80:83], v[124:127], a[204:207]\n\t"
"ds_read_b128 v[68:71], v57 offset:33792\n\t"
"s_waitcnt lgkmcnt(4)\n\t"
"v_mfma_f32_16x16x32_f16 a[236:239], v[88:91], v[124:127], a[236:239]\n\t"
"s_add_u32 s62, s46, s61\n\t"
"s_add_u32 s49, s47, s61\n\t"
"s_add_u32 m0, s49, 32768\n\t"
"s_waitcnt lgkmcnt(3)\n\t"
"v_mfma_f32_16x16x32_f16 a[252:255], v[92:95], v[124:127], a[252:255]\n\t"
"buffer_load_dwordx4 v59, s[76:79], s62 offen sc1 lds\n\t"
"v_mfma_f32_16x16x32_f16 a[248:251], v[92:95], v[120:123], a[248:251]\n\t"
"ds_read_b128 v[72:75], v57 offset:34816\n\t"
"v_mfma_f32_16x16x32_f16 a[232:235], v[88:91], v[120:123], a[232:235]\n\t"
"v_mfma_f32_16x16x32_f16 a[228:231], v[88:91], v[116:119], a[228:231]\n\t"
"buffer_load_dwordx4 v[220:223], v63, %[srdW], s41 offen\n\t"
"v_mfma_f32_16x16x32_f16 a[244:247], v[92:95], v[116:119], a[244:247]\n\t"
"ds_read_b128 v[76:79], v57 offset:35840\n\t"
"s_sub_u32 s80, s80, 1\n\t"
"s_getpc_b64 s[68:69]\n\t"
"s_cbranch_scc1 L_hnd_%=\n\t"
"v_mfma_f32_16x16x32_f16 a[240:243], v[92:95], v[112:115], a[240:243]\n\t"
"v_mfma_f32_16x16x32_f16 a[224:227], v[88:91], v[112:115], a[224:227]\n\t"
"s_waitcnt lgkmcnt(4)\n\t"
"v_mfma_f32_16x16x32_f16 a[0:3], v[64:67], v[96:99], a[0:3]\n\t"
"ds_read_b128 v[80:83], v57 offset:36864\n\t"
"s_waitcnt vmcnt(23)\n\t"
"v_readfirstlane_b32 s89, v43\n\t"
"s_cmp_ge_u32 s89, s71\n\t"
"s_cbranch_scc0 L_slow3_%=\n\t"
"L_sback3_%=:\n\t"
"s_waitcnt lgkmcnt(3)\n\t"
"v_mfma_f32_16x16x32_f16 a[16:19], v[68:71], v[96:99], a[16:19]\n\t"
"s_waitcnt lgkmcnt(4)\n\t"
"v_readfirstlane_b32 s49, v54\n\t"
"v_readfirstlane_b32 s70, v55\n\t"
"s_lshl_b32 s46, s49, 14\n\t"
"s_lshl_b32 s49, s49, 20\n\t"
"v_mfma_f32_16x16x32_f16 a[20:23], v[68:71], v[100:103], a[20:23]\n\t"
"s_add_u32 s40, s49, s50\n\t"
"s_add_u32 s41, s49, s51\n\t"
"s_waitcnt vmcnt(46)\n\t"
"v_cvt_pk_f16_f32 v112, v224, v228\n\t"
"v_mfma_f32_16x16x32_f16 a[4:7], v[64:67], v[100:103], a[4:7]\n\t"
"s_add_u32 s44, s49, s52\n\t"
"s_add_u32 s45, s49, s53\n\t"
"v_cvt_pk_f16_f32 v116, v225, v229\n\t"
"v_mfma_f32_16x16x32_f16 a[8:11], v[64:67], v[104:107], a[8:11]\n\t"
"ds_read_b128 v[84:87], v57 offset:37888\n\t"
"v_cvt_pk_f16_f32 v120, v226, v230\n\t"
"v_mfma_f32_16x16x32_f16 a[24:27], v[68:71], v[104:107], a[24:27]\n\t"
"v_cvt_pk_f16_f32 v124, v227, v231\n\t"
"v_mfma_f32_16x16x32_f16 a[28:31], v[68:71], v[108:111], a[28:31]\n\t"
"buffer_load_dwordx4 v[224:227], v60, %[srdW], s42 offen\n\t"
"v_mfma_f32_16x16x32_f16 a[12:15], v[64:67], v[108:111], a[12:15]\n\t"
"buffer_load_dword v43, off, %[srdC], s70 sc1\n\t"
"s_waitcnt lgkmcnt(3)\n\t"
"v_mfma_f32_16x16x32_f16 a[44:47], v[72:75], v[108:111], a[44:47]\n\t"
"ds_read_b128 v[88:91], v57 offset:38912\n\t"
"s_add_u32 s62, s46, s54\n\t"
"s_add_u32 s49, s47, s54\n\t"
"s_add_u32 m0, s49, 65536\n\t"
"s_waitcnt lgkmcnt(3)\n\t"
"v_mfma_f32_16x16x32_f16 a[60:63], v[76:79], v[108:111], a[60:63]\n\t"
"buffer_load_dwordx4 v59, s[76:79], s62 offen sc1 lds\n\t"
"v_mfma_f32_16x16x32_f16 a[56:59], v[76:79], v[104:107], a[56:59]\n\t"
"v_mfma_f32_16x16x32_f16 a[40:43], v[72:75], v[104:107], a[40:43]\n\t"
"v_mfma_f32_16x16x32_f16 a[36:39], v[72:75], v[100:103], a[36:39]\n\t"
"ds_read_b128 v[92:95], v57 offset:39936\n\t"
"buffer_load_dwordx4 v[228:231], v61, %[srdW], s42 offen\n\t"
"v_mfma_f32_16x16x32_f16 a[52:55], v[76:79], v[100:103], a[52:55]\n\t"
"v_mfma_f32_16x16x32_f16 a[48:51], v[76:79], v[96:99], a[48:51]\n\t"
"v_mfma_f32_16x16x32_f16 a[32:35], v[72:75], v[96:99], a[32:35]\n\t"
"s_waitcnt lgkmcnt(3)\n\t"
"v_mfma_f32_16x16x32_f16 a[64:67], v[80:83], v[96:99], a[64:67]\n\t"
"ds_read_b128 v[64:67], v57 offset:40960\n\t"
"s_waitcnt lgkmcnt(3)\n\t"
"v_mfma_f32_16x16x32_f16 a[80:83], v[84:87], v[96:99], a[80:83]\n\t"
"v_mfma_f32_16x16x32_f16 a[84:87], v[84:87], v[100:103], a[84:87]\n\t"
"s_waitcnt vmcnt(47)\n\t"
"v_cvt_pk_f16_f32 v113, v232, v236\n\t"
"v_mfma_f32_16x16x32_f16 a[68:71], v[80:83], v[100:103], a[68:71]\n\t"
"v_cvt_pk_f16_f32 v117, v233, v237\n\t"
"v_mfma_f32_16x16x32_f16 a[72:75], v[80:83], v[104:107], a[72:75]\n\t"
"ds_read_b128 v[68:71], v57 offset:41984\n\t"
"v_cvt_pk_f16_f32 v121, v234, v238\n\t"
"v_mfma_f32_16x16x32_f16 a[88:91], v[84:87], v[104:107], a[88:91]\n\t"
"v_cvt_pk_f16_f32 v125, v235, v239\n\t"
"v_mfma_f32_16x16x32_f16 a[92:95], v[84:87], v[108:111], a[92:95]\n\t"
"buffer_load_dwordx4 v[232:235], v62, %[srdW], s42 offen\n\t"
"v_mfma_f32_16x16x32_f16 a[76:79], v[80:83], v[108:111], a[76:79]\n\t"
"s_waitcnt lgkmcnt(3)\n\t"
"v_mfma_f32_16x16x32_f16 a[108:111], v[88:91], v[108:111], a[108:111]\n\t"
"ds_read_b128 v[72:75], v57 offset:43008\n\t"
"s_add_u32 s62, s46, s55\n\t"
"s_add_u32 s49, s47, s55\n\t"
"s_add_u32 m0, s49, 65536\n\t"
"s_waitcnt lgkmcnt(3)\n\t"
"v_mfma_f32_16x16x32_f16 a[124:127], v[92:95], v[108:111], a[124:127]\n\t"
"buffer_load_dwordx4 v59, s[76:79], s62 offen sc1 lds\n\t"
"v_mfma_f32_16x16x32_f16 a[120:123], v[92:95], v[104:107], a[120:123]\n\t"
"v_mfma_f32_16x16x32_f16 a[104:107], v[88:91], v[104:107], a[104:107]\n\t"
"v_mfma_f32_16x16x32_f16 a[100:103], v[88:91], v[100:103], a[100:103]\n\t"
"ds_read_b128 v[76:79], v57 offset:44032\n\t"
"buffer_load_dwordx4 v[236:239], v63, %[srdW], s42 offen\n\t"
"v_mfma_f32_16x16x32_f16 a[116:119], v[92:95], v[100:103], a[116:119]\n\t"
"v_mfma_f32_16x16x32_f16 a[112:115], v[92:95], v[96:99], a[112:115]\n\t"
"v_mfma_f32_16x16x32_f16 a[96:99], v[88:91], v[96:99], a[96:99]\n\t"
"s_waitcnt lgkmcnt(3)\n\t"
"v_mfma_f32_16x16x32_f16 a[128:131], v[64:67], v[96:99], a[128:131]\n\t"
"ds_read_b128 v[80:83], v57 offset:45056\n\t"
"s_waitcnt lgkmcnt(3)\n\t"
"v_mfma_f32_16x16x32_f16 a[144:147], v[68:71], v[96:99], a[144:147]\n\t"
"v_mfma_f32_16x16x32_f16 a[148:151], v[68:71], v[100:103], a[148:151]\n\t"
"s_waitcnt vmcnt(47)\n\t"
"v_cvt_pk_f16_f32 v114, v240, v244\n\t"
"v_mfma_f32_16x16x32_f16 a[132:135], v[64:67], v[100:103], a[132:135]\n\t"
"v_cvt_pk_f16_f32 v118, v241, v245\n\t"
"v_mfma_f32_16x16x32_f16 a[136:139], v[64:67], v[104:107], a[136:139]\n\t"
"ds_read_b128 v[84:87], v57 offset:46080\n\t"
"v_cvt_pk_f16_f32 v122, v242, v246\n\t"
"v_mfma_f32_16x16x32_f16 a[152:155], v[68:71], v[104:107], a[152:155]\n\t"
"v_cvt_pk_f16_f32 v126, v243, v247\n\t"
"v_mfma_f32_16x16x32_f16 a[156:159], v[68:71], v[108:111], a[156:159]\n\t"
"buffer_load_dwordx4 v[240:243], v60, %[srdW], s43 offen\n\t"
"v_mfma_f32_16x16x32_f16 a[140:143], v[64:67], v[108:111], a[140:143]\n\t"
"s_waitcnt lgkmcnt(3)\n\t"
"v_mfma_f32_16x16x32_f16 a[172:175], v[72:75], v[108:111], a[172:175]\n\t"
"ds_read_b128 v[88:91], v57 offset:47104\n\t"
"s_add_u32 s62, s46, s56\n\t"
"s_add_u32 s49, s47, s56\n\t"
"s_add_u32 m0, s49, 65536\n\t"
"s_waitcnt lgkmcnt(3)\n\t"
"v_mfma_f32_16x16x32_f16 a[188:191], v[76:79], v[108:111], a[188:191]\n\t"
"buffer_load_dwordx4 v59, s[76:79], s62 offen sc1 lds\n\t"
"v_mfma_f32_16x16x32_f16 a[184:187], v[76:79], v[104:107], a[184:187]\n\t"
"v_mfma_f32_16x16x32_f16 a[168:171], v[72:75], v[104:107], a[168:171]\n\t"
"v_mfma_f32_16x16x32_f16 a[164:167], v[72:75], v[100:103], a[164:167]\n\t"
"ds_read_b128 v[92:95], v57 offset:48128\n\t"
"buffer_load_dwordx4 v[244:247], v61, %[srdW], s43 offen\n\t"
"v_mfma_f32_16x16x32_f16 a[180:183], v[76:79], v[100:103], a[180:183]\n\t"
"v_mfma_f32_16x16x32_f16 a[176:179], v[76:79], v[96:99], a[176:179]\n\t"
"v_mfma_f32_16x16x32_f16 a[160:163], v[72:75], v[96:99], a[160:163]\n\t"
"s_waitcnt lgkmcnt(3)\n\t"
"v_mfma_f32_16x16x32_f16 a[192:195], v[80:83], v[96:99], a[192:195]\n\t"
"ds_read_b128 v[64:67], v57 offset:49152\n\t"
"s_waitcnt lgkmcnt(3)\n\t"
"v_mfma_f32_16x16x32_f16 a[208:211], v[84:87], v[96:99], a[208:211]\n\t"
"v_mfma_f32_16x16x32_f16 a[212:215], v[84:87], v[100:103], a[212:215]\n\t"
"s_waitcnt vmcnt(47)\n\t"
"v_cvt_pk_f16_f32 v115, v248, v252\n\t"
"v_mfma_f32_16x16x32_f16 a[196:199], v[80:83], v[100:103], a[196:199]\n\t"
"v_cvt_pk_f16_f32 v119, v249, v253\n\t"
"v_mfma_f32_16x16x32_f16 a[200:203], v[80:83], v[104:107], a[200:203]\n\t"
"ds_read_b128 v[68:71], v57 offset:50176\n\t"
"v_cvt_pk_f16_f32 v123, v250, v254\n\t"
"v_mfma_f32_16x16x32_f16 a[216:219], v[84:87], v[104:107], a[216:219]\n\t"
"v_cvt_pk_f16_f32 v127, v251, v255\n\t"
"v_mfma_f32_16x16x32_f16 a[220:223], v[84:87], v[108:111], a[220:223]\n\t"
"buffer_load_dwordx4 v[248:251], v62, %[srdW], s43 offen\n\t"
"v_mfma_f32_16x16x32_f16 a[204:207], v[80:83], v[108:111], a[204:207]\n\t"
"s_waitcnt lgkmcnt(3)\n\t"
"v_mfma_f32_16x16x32_f16 a[236:239], v[88:91], v[108:111], a[236:239]\n\t"
"ds_read_b128 v[72:75], v57 offset:51200\n\t"
"s_add_u32 s62, s46, s57\n\t"
"s_add_u32 s49, s47, s57\n\t"
"s_add_u32 m0, s49, 65536\n\t"
"s_waitcnt lgkmcnt(3)\n\t"
"v_mfma_f32_16x16x32_f16 a[252:255], v[92:95], v[108:111], a[252:255]\n\t"
"buffer_load_dwordx4 v59, s[76:79], s62 offen sc1 lds\n\t"
"v_mfma_f32_16x16x32_f16 a[248:251], v[92:95], v[104:107], a[248:251]\n\t"
"v_mfma_f32_16x16x32_f16 a[232:235], v[88:91], v[104:107], a[232:235]\n\t"
"v_mfma_f32_16x16x32_f16 a[228:231], v[88:91], v[100:103], a[228:231]\n\t"
"ds_read_b128 v[76:79], v57 offset:52224\n\t"
"buffer_load_dwordx4 v[252:255], v63, %[srdW], s43 offen\n\t"
"v_mfma_f32_16x16x32_f16 a[244:247], v[92:95], v[100:103], a[244:247]\n\t"
"s_sub_u32 s80, s80, 1\n\t"
"s_getpc_b64 s[68:69]\n\t"
"s_cbranch_scc1 L_hnd_%=\n\t"
"v_mfma_f32_16x16x32_f16 a[240:243], v[92:95], v[96:99], a[240:243]\n\t"
"v_mfma_f32_16x16x32_f16 a[224:227], v[88:91], v[96:99], a[224:227]\n\t"
"s_waitcnt lgkmcnt(3)\n\t"
"v_mfma_f32_16x16x32_f16 a[0:3], v[64:67], v[112:115], a[0:3]\n\t"
"ds_read_b128 v[80:83], v57 offset:53248\n\t"
"s_waitcnt lgkmcnt(3)\n\t"
"v_mfma_f32_16x16x32_f16 a[16:19], v[68:71], v[112:115], a[16:19]\n\t"
"s_mov_b32 s42, s44\n\t"
"s_mov_b32 s43, s45\n\t"
"v_mfma_f32_16x16x32_f16 a[20:23], v[68:71], v[116:119], a[20:23]\n\t"
"s_waitcnt vmcnt(47)\n\t"
"v_cvt_pk_f16_f32 v96, v128, v132\n\t"
"v_mfma_f32_16x16x32_f16 a[4:7], v[64:67], v[116:119], a[4:7]\n\t"
"v_cvt_pk_f16_f32 v100, v129, v133\n\t"
"v_mfma_f32_16x16x32_f16 a[8:11], v[64:67], v[120:123], a[8:11]\n\t"
"ds_read_b128 v[84:87], v57 offset:54272\n\t"
"v_cvt_pk_f16_f32 v104, v130, v134\n\t"
"v_mfma_f32_16x16x32_f16 a[24:27], v[68:71], v[120:123], a[24:27]\n\t"
"v_cvt_pk_f16_f32 v108, v131, v135\n\t"
"v_mfma_f32_16x16x32_f16 a[28:31], v[68:71], v[124:127], a[28:31]\n\t"
"buffer_load_dwordx4 v[128:131], v60, %[srdW], s40 offen\n\t"
"v_mfma_f32_16x16x32_f16 a[12:15], v[64:67], v[124:127], a[12:15]\n\t"
"s_waitcnt lgkmcnt(3)\n\t"
"v_mfma_f32_16x16x32_f16 a[44:47], v[72:75], v[124:127], a[44:47]\n\t"
"ds_read_b128 v[88:91], v57 offset:55296\n\t"
"s_add_u32 s62, s46, s58\n\t"
"s_add_u32 s49, s47, s58\n\t"
"s_add_u32 m0, s49, 65536\n\t"
"s_waitcnt lgkmcnt(3)\n\t"
"v_mfma_f32_16x16x32_f16 a[60:63], v[76:79], v[124:127], a[60:63]\n\t"
"buffer_load_dwordx4 v59, s[76:79], s62 offen sc1 lds\n\t"
"v_mfma_f32_16x16x32_f16 a[56:59], v[76:79], v[120:123], a[56:59]\n\t"
"v_mfma_f32_16x16x32_f16 a[40:43], v[72:75], v[120:123], a[40:43]\n\t"
"v_mfma_f32_16x16x32_f16 a[36:39], v[72:75], v[116:119], a[36:39]\n\t"
"ds_read_b128 v[92:95], v57 offset:56320\n\t"
"buffer_load_dwordx4 v[132:135], v61, %[srdW], s40 offen\n\t"
"v_mfma_f32_16x16x32_f16 a[52:55], v[76:79], v[116:119], a[52:55]\n\t"
"v_mfma_f32_16x16x32_f16 a[48:51], v[76:79], v[112:115], a[48:51]\n\t"
"v_mfma_f32_16x16x32_f16 a[32:35], v[72:75], v[112:115], a[32:35]\n\t"
"s_waitcnt lgkmcnt(3)\n\t"
"v_mfma_f32_16x16x32_f16 a[64:67], v[80:83], v[112:115], a[64:67]\n\t"
"ds_read_b128 v[64:67], v57 offset:57344\n\t"
"s_waitcnt lgkmcnt(3)\n\t"
"v_mfma_f32_16x16x32_f16 a[80:83], v[84:87], v[112:115], a[80:83]\n\t"
"v_mfma_f32_16x16x32_f16 a[84:87], v[84:87], v[116:119], a[84:87]\n\t"
"s_waitcnt vmcnt(47)\n\t"
"v_cvt_pk_f16_f32 v97, v136, v140\n\t"
"v_mfma_f32_16x16x32_f16 a[68:71], v[80:83], v[116:119], a[68:71]\n\t"
"v_cvt_pk_f16_f32 v101, v137, v141\n\t"
"v_mfma_f32_16x16x32_f16 a[72:75], v[80:83], v[120:123], a[72:75]\n\t"
"ds_read_b128 v[68:71], v57 offset:58368\n\t"
"v_cvt_pk_f16_f32 v105, v138, v142\n\t"
"v_mfma_f32_16x16x32_f16 a[88:91], v[84:87], v[120:123], a[88:91]\n\t"
"v_cvt_pk_f16_f32 v109, v139, v143\n\t"
"v_mfma_f32_16x16x32_f16 a[92:95], v[84:87], v[124:127], a[92:95]\n\t"
"buffer_load_dwordx4 v[136:139], v62, %[srdW], s40 offen\n\t"
"v_mfma_f32_16x16x32_f16 a[76:79], v[80:83], v[124:127], a[76:79]\n\t"
"s_waitcnt lgkmcnt(3)\n\t"
"v_mfma_f32_16x16x32_f16 a[108:111], v[88:91], v[124:127], a[108:111]\n\t"
"ds_read_b128 v[72:75], v57 offset:59392\n\t"
"s_add_u32 s62, s46, s59\n\t"
"s_add_u32 s49, s47, s59\n\t"
"s_add_u32 m0, s49, 65536\n\t"
"s_waitcnt lgkmcnt(3)\n\t"
"v_mfma_f32_16x16x32_f16 a[124:127], v[92:95], v[124:127], a[124:127]\n\t"
"buffer_load_dwordx4 v59, s[76:79], s62 offen sc1 lds\n\t"
"v_mfma_f32_16x16x32_f16 a[120:123], v[92:95], v[120:123], a[120:123]\n\t"
"v_mfma_f32_16x16x32_f16 a[104:107], v[88:91], v[120:123], a[104:107]\n\t"
"v_mfma_f32_16x16x32_f16 a[100:103], v[88:91], v[116:119], a[100:103]\n\t"
"ds_read_b128 v[76:79], v57 offset:60416\n\t"
"buffer_load_dwordx4 v[140:143], v63, %[srdW], s40 offen\n\t"
"v_mfma_f32_16x16x32_f16 a[116:119], v[92:95], v[116:119], a[116:119]\n\t"
"v_mfma_f32_16x16x32_f16 a[112:115], v[92:95], v[112:115], a[112:115]\n\t"
"v_mfma_f32_16x16x32_f16 a[96:99], v[88:91], v[112:115], a[96:99]\n\t"
"s_waitcnt lgkmcnt(3)\n\t"
"v_mfma_f32_16x16x32_f16 a[128:131], v[64:67], v[112:115], a[128:131]\n\t"
"ds_read_b128 v[80:83], v57 offset:61440\n\t"
"s_waitcnt lgkmcnt(3)\n\t"
"v_mfma_f32_16x16x32_f16 a[144:147], v[68:71], v[112:115], a[144:147]\n\t"
"v_mfma_f32_16x16x32_f16 a[148:151], v[68:71], v[116:119], a[148:151]\n\t"
"s_waitcnt vmcnt(47)\n\t"
"v_cvt_pk_f16_f32 v98, v144, v148\n\t"
"v_mfma_f32_16x16x32_f16 a[132:135], v[64:67], v[116:119], a[132:135]\n\t"
"v_cvt_pk_f16_f32 v102, v145, v149\n\t"
"v_mfma_f32_16x16x32_f16 a[136:139], v[64:67], v[120:123], a[136:139]\n\t"
"ds_read_b128 v[84:87], v57 offset:62464\n\t"
"v_cvt_pk_f16_f32 v106, v146, v150\n\t"
"v_mfma_f32_16x16x32_f16 a[152:155], v[68:71], v[120:123], a[152:155]\n\t"
"v_cvt_pk_f16_f32 v110, v147, v151\n\t"
"v_mfma_f32_16x16x32_f16 a[156:159], v[68:71], v[124:127], a[156:159]\n\t"
"buffer_load_dwordx4 v[144:147], v60, %[srdW], s41 offen\n\t"
"v_mfma_f32_16x16x32_f16 a[140:143], v[64:67], v[124:127], a[140:143]\n\t"
"s_waitcnt lgkmcnt(3)\n\t"
"v_mfma_f32_16x16x32_f16 a[172:175], v[72:75], v[124:127], a[172:175]\n\t"
"ds_read_b128 v[88:91], v57 offset:63488\n\t"
"s_add_u32 s62, s46, s60\n\t"
"s_add_u32 s49, s47, s60\n\t"
"s_add_u32 m0, s49, 65536\n\t"
"s_waitcnt lgkmcnt(3)\n\t"
"v_mfma_f32_16x16x32_f16 a[188:191], v[76:79], v[124:127], a[188:191]\n\t"
"buffer_load_dwordx4 v59, s[76:79], s62 offen sc1 lds\n\t"
"v_mfma_f32_16x16x32_f16 a[184:187], v[76:79], v[120:123], a[184:187]\n\t"
"v_mfma_f32_16x16x32_f16 a[168:171], v[72:75], v[120:123], a[168:171]\n\t"
"v_mfma_f32_16x16x32_f16 a[164:167], v[72:75], v[116:119], a[164:167]\n\t"
"ds_read_b128 v[92:95], v57 offset:64512\n\t"
"buffer_load_dwordx4 v[148:151], v61, %[srdW], s41 offen\n\t"
"v_mfma_f32_16x16x32_f16 a[180:183], v[76:79], v[116:119], a[180:183]\n\t"
"v_mfma_f32_16x16x32_f16 a[176:179], v[76:79], v[112:115], a[176:179]\n\t"
"v_mfma_f32_16x16x32_f16 a[160:163], v[72:75], v[112:115], a[160:163]\n\t"
"s_waitcnt lgkmcnt(3)\n\t"
"v_mfma_f32_16x16x32_f16 a[192:195], v[80:83], v[112:115], a[192:195]\n\t"
"s_waitcnt lgkmcnt(2)\n\t"
"v_mfma_f32_16x16x32_f16 a[208:211], v[84:87], v[112:115], a[208:211]\n\t"
"s_memrealtime s[90:91]\n\t"
"v_mfma_f32_16x16x32_f16 a[212:215], v[84:87], v[116:119], a[212:215]\n\t"
"s_waitcnt vmcnt(47)\n\t"
"v_cvt_pk_f16_f32 v99, v152, v156\n\t"
"v_mfma_f32_16x16x32_f16 a[196:199], v[80:83], v[116:119], a[196:199]\n\t"
"v_cvt_pk_f16_f32 v103, v153, v157\n\t"
"s_waitcnt vmcnt(48) lgkmcnt(0)\n\t"
"s_sub_u32 s89, s90, s65\n\t"
"s_cmp_lt_i32 s89, 0\n\t"
"s_getpc_b64 s[68:69]\n\t"
"s_cbranch_scc1 L_pace_%=\n\t"
"s_ashr_i32 s89, s95, 3\n\t"
"s_add_u32 s65, s90, s89\n\t"
"s_sub_u32 s95, s95, %[sslope]\n\t"
"s_barrier\n\t"
"v_mfma_f32_16x16x32_f16 a[200:203], v[80:83], v[120:123], a[200:203]\n\t"
"ds_read_b128 v[64:67], v58 offset:0\n\t"
"v_cvt_pk_f16_f32 v107, v154, v158\n\t"
"v_mfma_f32_16x16x32_f16 a[216:219], v[84:87], v[120:123], a[216:219]\n\t"
"ds_read_b64 v[54:55], v53\n\t"
"v_add_u32 v53, 8, v53\n\t"
"v_cvt_pk_f16_f32 v111, v155, v159\n\t"
"v_mfma_f32_16x16x32_f16 a[220:223], v[84:87], v[124:127], a[220:223]\n\t"
"buffer_load_dwordx4 v[152:155], v62, %[srdW], s41 offen\n\t"
"v_mfma_f32_16x16x32_f16 a[204:207], v[80:83], v[124:127], a[204:207]\n\t"
"ds_read_b128 v[68:71], v58 offset:1024\n\t"
"s_waitcnt lgkmcnt(4)\n\t"
"v_mfma_f32_16x16x32_f16 a[236:239], v[88:91], v[124:127], a[236:239]\n\t"
"s_add_u32 s62, s46, s61\n\t"
"s_add_u32 s49, s47, s61\n\t"
"s_add_u32 m0, s49, 65536\n\t"
"s_waitcnt lgkmcnt(3)\n\t"
"v_mfma_f32_16x16x32_f16 a[252:255], v[92:95], v[124:127], a[252:255]\n\t"
"buffer_load_dwordx4 v59, s[76:79], s62 offen sc1 lds\n\t"
"v_mfma_f32_16x16x32_f16 a[248:251], v[92:95], v[120:123], a[248:251]\n\t"
"ds_read_b128 v[72:75], v58 offset:2048\n\t"
"v_mfma_f32_16x16x32_f16 a[232:235], v[88:91], v[120:123], a[232:235]\n\t"
"v_mfma_f32_16x16x32_f16 a[228:231], v[88:91], v[116:119], a[228:231]\n\t"
"buffer_load_dwordx4 v[156:159], v63, %[srdW], s41 offen\n\t"
"v_mfma_f32_16x16x32_f16 a[244:247], v[92:95], v[116:119], a[244:247]\n\t"
"ds_read_b128 v[76:79], v58 offset:3072\n\t"
"s_sub_u32 s80, s80, 1\n\t"
"s_getpc_b64 s[68:69]\n\t"
"s_cbranch_scc1 L_hnd_%=\n\t"
"v_mfma_f32_16x16x32_f16 a[240:243], v[92:95], v[112:115], a[240:243]\n\t"
"v_mfma_f32_16x16x32_f16 a[224:227], v[88:91], v[112:115], a[224:227]\n\t"
"s_sub_u32 s48, s48, 1\n\t"
"s_cmp_lg_u32 s48, 0\n\t"
"s_cbranch_scc1 L_loop_%=\n\t"
"s_branch L_done_%=\n\t"
"L_hnd_%=:\n\t"
"s_add_u32 s68, s68, 4\n\t"
"s_addc_u32 s69, s69, 0\n\t"
"s_cmp_eq_u32 s81, 1\n\t"
"s_cbranch_scc1 L_hB_%=\n\t"
"s_cmp_eq_u32 s81, 2\n\t"
"s_cbranch_scc1 L_hC_%=\n\t"
"v_readlane_b32 s84, %[vt0], s82\n\t"
"v_readlane_b32 s85, %[vt5], s82\n\t"
"s_and_b32 s85, s85, 3\n\t"
"s_add_u32 s89, s85, 0\n\t"
"s_and_b32 s89, s89, 3\n\t"
"s_lshl_b32 s89, s89, 6\n\t"
"s_add_u32 s89, s89, s84\n\t"
"buffer_load_dwordx4 v[16:19], v40, %[srdx], s89 offen\n\t"
"s_add_u32 s89, s85, 1\n\t"
"s_and_b32 s89, s89, 3\n\t"
"s_lshl_b32 s89, s89, 6\n\t"
"s_add_u32 s89, s89, s84\n\t"
"buffer_load_dwordx4 v[20:23], v40, %[srdx], s89 offen\n\t"
"s_add_u32 s89, s85, 2\n\t"
"s_and_b32 s89, s89, 3\n\t"
"s_lshl_b32 s89, s89, 6\n\t"
"s_add_u32 s89, s89, s84\n\t"
"buffer_load_dwordx4 v[24:27], v40, %[srdx], s89 offen\n\t"
"s_add_u32 s89, s85, 3\n\t"
"s_and_b32 s89, s89, 3\n\t"
"s_lshl_b32 s89, s89, 6\n\t"
"s_add_u32 s89, s89, s84\n\t"
"buffer_load_dwordx4 v[28:31], v40, %[srdx], s89 offen\n\t"
"s_mov_b32 s81, 1\n\t"
"s_mov_b32 s80, 1\n\t"
"s_setpc_b64 s[68:69]\n\t"
"L_hB_%=:\n\t"
"s_waitcnt vmcnt(24)\n\t"
"v_readlane_b32 s72, %[vt3], s82\n\t"
"v_readlane_b32 s73, %[vt4], s82\n\t"
"v_readlane_b32 s84, %[vt1], s82\n\t"
"v_cvt_pk_f16_f32 v32, v16, v17\n\t"
"v_cvt_pk_f16_f32 v33, v18, v19\n\t"
"v_cvt_pk_f16_f32 v34, v20, v21\n\t"
"v_cvt_pk_f16_f32 v35, v22, v23\n\t"
"v_cvt_pk_f16_f32 v36, v24, v25\n\t"
"v_cvt_pk_f16_f32 v37, v26, v27\n\t"
"v_cvt_pk_f16_f32 v38, v28, v29\n\t"
"v_cvt_pk_f16_f32 v39, v30, v31\n\t"
"s_add_u32 s84, s84, s83\n\t"
"s_add_u32 s85, s84, 0x4000\n\t"
"buffer_store_dwordx4 v[32:35], v41, s[72:75], s84 offen sc0 sc1\n\t"
"buffer_store_dwordx4 v[36:39], v41, s[72:75], s85 offen sc0 sc1\n\t"
"s_mov_b32 s81, 2\n\t"
"s_mov_b32 s80, 1\n\t"
"s_setpc_b64 s[68:69]\n\t"
"L_hC_%=:\n\t"
"s_waitcnt vmcnt(24)\n\t"
"v_readlane_b32 s84, %[vt2], s82\n\t"
"s_mov_b64 exec, 1\n\t"
"s_nop 3\n\t"
"buffer_atomic_add v42, off, s[72:75], s84 sc1\n\t"
"s_mov_b64 exec, -1\n\t"
"v_readlane_b32 s80, %[vt5], s82\n\t"
"s_add_u32 s82, s82, 1\n\t"
"s_mov_b32 s81, 0\n\t"
"s_lshr_b32 s80, s80, 8\n\t"
"s_setpc_b64 s[68:69]\n\t"
"L_slow0_%=:\n\t"
"s_cmp_lg_u32 s88, 0\n\t"
"s_cbranch_scc1 L_sback0_%=\n\t"
"s_memrealtime s[90:91]\n\t"
"L_slow0_%=_l:\n\t"
"buffer_load_dword v43, off, %[srdC], s70 sc0 sc1\n\t"
"buffer_load_dword v52, off, %[srdC], 0 offset:256 sc1\n\t"
"s_memrealtime s[92:93]\n\t"
"s_waitcnt vmcnt(0) lgkmcnt(0)\n\t"
"v_readfirstlane_b32 s89, v43\n\t"
"s_cmp_ge_u32 s89, s71\n\t"
"s_cbranch_scc1 L_sback0_%=\n\t"
"v_readfirstlane_b32 s89, v52\n\t"
"s_cmp_eq_u32 s89, s94\n\t"
"s_cbranch_scc1 L_slow0_%=_d\n\t"
"s_sub_u32 s89, s92, s90\n\t"
"s_cmp_gt_u32 s89, 100000000\n\t"
"s_cbranch_scc1 L_slow0_%=_d\n\t"
"s_sleep 1\n\t"
"s_branch L_slow0_%=_l\n\t"
"L_slow0_%=_d:\n\t"
"s_mov_b32 s88, 1\n\t"
"v_mov_b32 v52, s94\n\t"
"buffer_store_dword v52, off, %[srdC], 0 offset:256 sc1\n\t"
"s_branch L_sback0_%=\n\t"
"L_slow1_%=:\n\t"
"s_cmp_lg_u32 s88, 0\n\t"
"s_cbranch_scc1 L_sback1_%=\n\t"
"s_memrealtime s[90:91]\n\t"
"L_slow1_%=_l:\n\t"
"buffer_load_dword v43, off, %[srdC], s70 sc0 sc1\n\t"
"buffer_load_dword v52, off, %[srdC], 0 offset:256 sc1\n\t"
"s_memrealtime s[92:93]\n\t"
"s_waitcnt vmcnt(0) lgkmcnt(0)\n\t"
"v_readfirstlane_b32 s89, v43\n\t"
"s_cmp_ge_u32 s89, s71\n\t"
"s_cbranch_scc1 L_sback1_%=\n\t"
"v_readfirstlane_b32 s89, v52\n\t"
"s_cmp_eq_u32 s89, s94\n\t"
"s_cbranch_scc1 L_slow1_%=_d\n\t"
"s_sub_u32 s89, s92, s90\n\t"
"s_cmp_gt_u32 s89, 100000000\n\t"
"s_cbranch_scc1 L_slow1_%=_d\n\t"
"s_sleep 1\n\t"
"s_branch L_slow1_%=_l\n\t"
"L_slow1_%=_d:\n\t"
"s_mov_b32 s88, 1\n\t"
"v_mov_b32 v52, s94\n\t"
"buffer_store_dword v52, off, %[srdC], 0 offset:256 sc1\n\t"
"s_branch L_sback1_%=\n\t"
"L_slow2_%=:\n\t"
"s_cmp_lg_u32 s88, 0\n\t"
"s_cbranch_scc1 L_sback2_%=\n\t"
"s_memrealtime s[90:91]\n\t"
"L_slow2_%=_l:\n\t"
"buffer_load_dword v43, off, %[srdC], s70 sc0 sc1\n\t"
"buffer_load_dword v52, off, %[srdC], 0 offset:256 sc1\n\t"
"s_memrealtime s[92:93]\n\t"
"s_waitcnt vmcnt(0) lgkmcnt(0)\n\t"
"v_readfirstlane_b32 s89, v43\n\t"
"s_cmp_ge_u32 s89, s71\n\t"
"s_cbranch_scc1 L_sback2_%=\n\t"
"v_readfirstlane_b32 s89, v52\n\t"
"s_cmp_eq_u32 s89, s94\n\t"
"s_cbranch_scc1 L_slow2_%=_d\n\t"
"s_sub_u32 s89, s92, s90\n\t"
"s_cmp_gt_u32 s89, 100000000\n\t"
"s_cbranch_scc1 L_slow2_%=_d\n\t"
"s_sleep 1\n\t"
"s_branch L_slow2_%=_l\n\t"
"L_slow2_%=_d:\n\t"
"s_mov_b32 s88, 1\n\t"
"v_mov_b32 v52, s94\n\t"
"buffer_store_dword v52, off, %[srdC], 0 offset:256 sc1\n\t"
"s_branch L_sback2_%=\n\t"
"L_slow3_%=:\n\t"
"s_cmp_lg_u32 s88, 0\n\t"
"s_cbranch_scc1 L_sback3_%=\n\t"
"s_memrealtime s[90:91]\n\t"
"L_slow3_%=_l:\n\t"
"buffer_load_dword v43, off, %[srdC], s70 sc0 sc1\n\t"
"buffer_load_dword v52, off, %[srdC], 0 offset:256 sc1\n\t"
"s_memrealtime s[92:93]\n\t"
"s_waitcnt vmcnt(0) lgkmcnt(0)\n\t"
"v_readfirstlane_b32 s89, v43\n\t"
"s_cmp_ge_u32 s89, s71\n\t"
"s_cbranch_scc1 L_sback3_%=\n\t"
"v_readfirstlane_b32 s89, v52\n\t"
"s_cmp_eq_u32 s89, s94\n\t"
"s_cbranch_scc1 L_slow3_%=_d\n\t"
"s_sub_u32 s89, s92, s90\n\t"
"s_cmp_gt_u32 s89, 100000000\n\t"
"s_cbranch_scc1 L_slow3_%=_d\n\t"
"s_sleep 1\n\t"
"s_branch L_slow3_%=_l\n\t"
"L_slow3_%=_d:\n\t"
"s_mov_b32 s88, 1\n\t"
"v_mov_b32 v52, s94\n\t"
"buffer_store_dword v52, off, %[srdC], 0 offset:256 sc1\n\t"
"s_branch L_sback3_%=\n\t"
"L_pp_%=:\n\t"
"s_cmp_lg_u32 s88, 0\n\t"
"s_cbranch_scc1 L_ppb_%=\n\t"
"s_memrealtime s[90:91]\n\t"
"L_pp_%=_l:\n\t"
"buffer_load_dword v43, v44, %[srdC], s70 offen sc0 sc1\n\t"
"buffer_load_dword v52, off, %[srdC], 0 offset:256 sc1\n\t"
"s_memrealtime s[92:93]\n\t"
"s_waitcnt vmcnt(0) lgkmcnt(0)\n\t"
"v_cmp_gt_u32 vcc, s71, v43\n\t"
"s_nop 1\n\t"
"s_cmp_eq_u64 vcc, 0\n\t"
"s_cbranch_scc1 L_ppb_%=\n\t"
"v_readfirstlane_b32 s89, v52\n\t"
"s_cmp_eq_u32 s89, s94\n\t"
"s_cbranch_scc1 L_pp_%=_d\n\t"
"s_sub_u32 s89, s92, s90\n\t"
"s_cmp_gt_u32 s89, 100000000\n\t"
"s_cbranch_scc1 L_pp_%=_d\n\t"
"s_sleep 1\n\t"
"s_branch L_pp_%=_l\n\t"
"L_pp_%=_d:\n\t"
"s_mov_b32 s88, 1\n\t"
"v_mov_b32 v52, s94\n\t"
"buffer_store_dword v52, off, %[srdC], 0 offset:256 sc1\n\t"
"s_branch L_ppb_%=\n\t"
"L_pace_%=:\n\t"
"s_add_u32 s68, s68, 4\n\t"
"s_addc_u32 s69, s69, 0\n\t"
"L_pace_l_%=:\n\t"
"s_memrealtime s[90:91]\n\t"
"s_waitcnt lgkmcnt(0)\n\t"
"s_sub_u32 s89, s90, s65\n\t"
"s_cmp_lt_i32 s89, 0\n\t"
"s_cbranch_scc1 L_pace_l_%=\n\t"
"s_mov_b32 s90, s65\n\t"
"s_setpc_b64 s[68:69]\n\t"
"L_done_%=:\n\t"
"s_waitcnt vmcnt(0) lgkmcnt(0)\n\t"
"s_nop 7\n\t"
"s_nop 7\n\t"

      :
      : [scall] "s"(call), [swrot] "s"(swrot2), [sk0] "s"(sk0), [srdW] "s"(srdW), [srdx] "s"(srdx), [srdC] "s"(srdC), [sxlo] "s"(sxlo), [sxhi] "s"(sxhi), [sldsb] "s"(sldsb), [swv8k] "s"(swv8k),
        [sm0w] "s"(sm0w), [sxrot] "s"(sxrot), [slflg] "s"(slflg), [vla] "v"(vla), [space] "s"(space), [sslope] "s"(sslope), [sf0] "s"(sf0), [snext0] "s"(snext0), [swsz] "s"(swsz),
        [vlane] "v"(lane), [vt0] "v"(t0), [vt1] "v"(t1), [vt2] "v"(t2), [vt3] "v"(t3), [vt4] "v"(t4), [vt5] "v"(t5)
      : "memory", "scc", "vcc", "m0", "exec", VCLOB, ACLOB, SCLOB);
  }
  {
    int tz = threadIdx.x; asm volatile("" : "+v"(tz));
    const int lane2 = tz & 63, g2 = lane2 >> 4, c2 = lane2 & 15;
    char* xch = myws + WS_XCH;
    const u32x4 srdS = mksrd(xch + (size_t)b * 131072 + wid * 32768, 32768);
    const u32x4 srdR = mksrd(xch + (size_t)partner * 131072 + wid * 32768, 32768);
    const u32x4 srdF = mksrd(myws + WS_PFLG, 256 * 4 * 128);
    const u32x4 srdT = mksrd(myws + WS_TMO, 4);
    const unsigned ooff = (unsigned)((mt * 256 + h * 128) * NG + pn * 256 + wid * 64) * 4u;
    const u32x4 srdO = mksrd((const char*)out + ooff, (unsigned)(ML * (size_t)NG * 4) - ooff);
    const unsigned sfown = (b * 4 + wid) * 128, sfpar = (partner * 4 + wid) * 128;
    const unsigned vl16 = lane2 * 16, vooff = (4 * g2) * 32768 + 16 * c2;
    asm volatile(
"v_add_u32 v8, 0, %[vlane16]\n\t"
"v_add_u32 v9, 4096, %[vlane16]\n\t"
"v_add_u32 v10, 8192, %[vlane16]\n\t"
"v_add_u32 v11, 12288, %[vlane16]\n\t"
"v_add_u32 v12, 16384, %[vlane16]\n\t"
"v_add_u32 v13, 20480, %[vlane16]\n\t"
"v_add_u32 v14, 24576, %[vlane16]\n\t"
"v_add_u32 v15, 28672, %[vlane16]\n\t"
"v_mov_b32 v58, %[scall]\n\t"
"buffer_store_dwordx4 a[128:131], v8, %[srdS], 0 offen offset:0 sc1\n\t"
"buffer_store_dwordx4 a[132:135], v8, %[srdS], 0 offen offset:1024 sc1\n\t"
"buffer_store_dwordx4 a[136:139], v8, %[srdS], 0 offen offset:2048 sc1\n\t"
"buffer_store_dwordx4 a[140:143], v8, %[srdS], 0 offen offset:3072 sc1\n\t"
"buffer_store_dwordx4 a[144:147], v9, %[srdS], 0 offen offset:0 sc1\n\t"
"buffer_store_dwordx4 a[148:151], v9, %[srdS], 0 offen offset:1024 sc1\n\t"
"buffer_store_dwordx4 a[152:155], v9, %[srdS], 0 offen offset:2048 sc1\n\t"
"buffer_store_dwordx4 a[156:159], v9, %[srdS], 0 offen offset:3072 sc1\n\t"
"buffer_store_dwordx4 a[160:163], v10, %[srdS], 0 offen offset:0 sc1\n\t"
"buffer_store_dwordx4 a[164:167], v10, %[srdS], 0 offen offset:1024 sc1\n\t"
"buffer_store_dwordx4 a[168:171], v10, %[srdS], 0 offen offset:2048 sc1\n\t"
"buffer_store_dwordx4 a[172:175], v10, %[srdS], 0 offen offset:3072 sc1\n\t"
"buffer_store_dwordx4 a[176:179], v11, %[srdS], 0 offen offset:0 sc1\n\t"
"buffer_store_dwordx4 a[180:183], v11, %[srdS], 0 offen offset:1024 sc1\n\t"
"buffer_store_dwordx4 a[184:187], v11, %[srdS], 0 offen offset:2048 sc1\n\t"
"buffer_store_dwordx4 a[188:191], v11, %[srdS], 0 offen offset:3072 sc1\n\t"
"buffer_store_dwordx4 a[192:195], v12, %[srdS], 0 offen offset:0 sc1\n\t"
"buffer_store_dwordx4 a[196:199], v12, %[srdS], 0 offen offset:1024 sc1\n\t"
"buffer_store_dwordx4 a[200:203], v12, %[srdS], 0 offen offset:2048 sc1\n\t"
"buffer_store_dwordx4 a[204:207], v12, %[srdS], 0 offen offset:3072 sc1\n\t"
"buffer_store_dwordx4 a[208:211], v13, %[srdS], 0 offen offset:0 sc1\n\t"
"buffer_store_dwordx4 a[212:215], v13, %[srdS], 0 offen offset:1024 sc1\n\t"
"buffer_store_dwordx4 a[216:219], v13, %[srdS], 0 offen offset:2048 sc1\n\t"
"buffer_store_dwordx4 a[220:223], v13, %[srdS], 0 offen offset:3072 sc1\n\t"
"buffer_store_dwordx4 a[224:227], v14, %[srdS], 0 offen offset:0 sc1\n\t"
"buffer_store_dwordx4 a[228:231], v14, %[srdS], 0 offen offset:1024 sc1\n\t"
"buffer_store_dwordx4 a[232:235], v14, %[srdS], 0 offen offset:2048 sc1\n\t"
"buffer_store_dwordx4 a[236:239], v14, %[srdS], 0 offen offset:3072 sc1\n\t"
"buffer_store_dwordx4 a[240:243], v15, %[srdS], 0 offen offset:0 sc1\n\t"
"buffer_store_dwordx4 a[244:247], v15, %[srdS], 0 offen offset:1024 sc1\n\t"
"buffer_store_dwordx4 a[248:251], v15, %[srdS], 0 offen offset:2048 sc1\n\t"
"buffer_store_dwordx4 a[252:255], v15, %[srdS], 0 offen offset:3072 sc1\n\t"
"v_mov_b32 v56, 0x42ad1f97\n\t"
"v_mov_b32 v57, 0xc2ad1f97\n\t"
"s_waitcnt vmcnt(0)\n\t"
"s_mov_b64 exec, 1\n\t"
"buffer_store_dword v58, off, %[srdF], %[sfown] sc1\n\t"
"s_memrealtime s[60:61]\n\t"
"L_poll_%=:\n\t"
"buffer_load_dword v59, off, %[srdF], %[sfpar] sc1\n\t"
"buffer_load_dword v48, off, %[srdT], 0 sc1\n\t"
"s_memrealtime s[62:63]\n\t"
"s_waitcnt vmcnt(0) lgkmcnt(0)\n\t"
"v_readfirstlane_b32 s64, v59\n\t"
"s_cmp_ge_u32 s64, %[scall]\n\t"
"s_cbranch_scc1 L_go_%=\n\t"
"v_readfirstlane_b32 s64, v48\n\t"
"s_cmp_eq_u32 s64, %[scall]\n\t"
"s_cbranch_scc1 L_go_%=\n\t"
"s_sub_u32 s64, s62, s60\n\t"
"s_cmp_gt_u32 s64, 100000000\n\t"
"s_cbranch_scc1 L_dead_%=\n\t"
"s_sleep 1\n\t"
"s_branch L_poll_%=\n\t"
"L_dead_%=:\n\t"
"buffer_store_dword v58, off, %[srdT], 0 sc1\n\t"
"L_go_%=:\n\t"
"s_mov_b64 exec, -1\n\t"
"buffer_load_dwordx4 v[128:131], v8, %[srdR], 0 offen offset:0 sc1\n\t"
"buffer_load_dwordx4 v[132:135], v8, %[srdR], 0 offen offset:1024 sc1\n\t"
"buffer_load_dwordx4 v[136:139], v8, %[srdR], 0 offen offset:2048 sc1\n\t"
"buffer_load_dwordx4 v[140:143], v8, %[srdR], 0 offen offset:3072 sc1\n\t"
"buffer_load_dwordx4 v[144:147], v9, %[srdR], 0 offen offset:0 sc1\n\t"
"buffer_load_dwordx4 v[148:151], v9, %[srdR], 0 offen offset:1024 sc1\n\t"
"buffer_load_dwordx4 v[152:155], v9, %[srdR], 0 offen offset:2048 sc1\n\t"
"buffer_load_dwordx4 v[156:159], v9, %[srdR], 0 offen offset:3072 sc1\n\t"
"buffer_load_dwordx4 v[160:163], v10, %[srdR], 0 offen offset:0 sc1\n\t"
"buffer_load_dwordx4 v[164:167], v10, %[srdR], 0 offen offset:1024 sc1\n\t"
"buffer_load_dwordx4 v[168:171], v10, %[srdR], 0 offen offset:2048 sc1\n\t"
"buffer_load_dwordx4 v[172:175], v10, %[srdR], 0 offen offset:3072 sc1\n\t"
"buffer_load_dwordx4 v[176:179], v11, %[srdR], 0 offen offset:0 sc1\n\t"
"buffer_load_dwordx4 v[180:183], v11, %[srdR], 0 offen offset:1024 sc1\n\t"
"buffer_load_dwordx4 v[184:187], v11, %[srdR], 0 offen offset:2048 sc1\n\t"
"buffer_load_dwordx4 v[188:191], v11, %[srdR], 0 offen offset:3072 sc1\n\t"
"buffer_load_dwordx4 v[192:195], v12, %[srdR], 0 offen offset:0 sc1\n\t"
"buffer_load_dwordx4 v[196:199], v12, %[srdR], 0 offen offset:1024 sc1\n\t"
"buffer_load_dwordx4 v[200:203], v12, %[srdR], 0 offen offset:2048 sc1\n\t"
"buffer_load_dwordx4 v[204:207], v12, %[srdR], 0 offen offset:3072 sc1\n\t"
"buffer_load_dwordx4 v[208:211], v13, %[srdR], 0 offen offset:0 sc1\n\t"
"buffer_load_dwordx4 v[212:215], v13, %[srdR], 0 offen offset:1024 sc1\n\t"
"buffer_load_dwordx4 v[216:219], v13, %[srdR], 0 offen offset:2048 sc1\n\t"
"buffer_load_dwordx4 v[220:223], v13, %[srdR], 0 offen offset:3072 sc1\n\t"
"buffer_load_dwordx4 v[224:227], v14, %[srdR], 0 offen offset:0 sc1\n\t"
"buffer_load_dwordx4 v[228:231], v14, %[srdR], 0 offen offset:1024 sc1\n\t"
"buffer_load_dwordx4 v[232:235], v14, %[srdR], 0 offen offset:2048 sc1\n\t"
"buffer_load_dwordx4 v[236:239], v14, %[srdR], 0 offen offset:3072 sc1\n\t"
"buffer_load_dwordx4 v[240:243], v15, %[srdR], 0 offen offset:0 sc1\n\t"
"buffer_load_dwordx4 v[244:247], v15, %[srdR], 0 offen offset:1024 sc1\n\t"
"buffer_load_dwordx4 v[248:251], v15, %[srdR], 0 offen offset:2048 sc1\n\t"
"buffer_load_dwordx4 v[252:255], v15, %[srdR], 0 offen offset:3072 sc1\n\t"
"v_mov_b32 v50, 0xbfb8aa3b\n\t"
"v_mov_b32 v51, 0xbfb8aa3b\n\t"
"v_mov_b32 v52, 1.0\n\t"
"v_mov_b32 v53, 1.0\n\t"
"s_mov_b32 s70, 0\n\t"
"s_mov_b32 s71, 32768\n\t"
"s_mov_b32 s72, 65536\n\t"
"s_mov_b32 s73, 98304\n\t"
"v_accvgpr_read_b32 v16, a0\n\t"
"v_accvgpr_read_b32 v17, a1\n\t"
"v_accvgpr_read_b32 v18, a2\n\t"
"v_accvgpr_read_b32 v19, a3\n\t"
"v_accvgpr_read_b32 v20, a4\n\t"
"v_accvgpr_read_b32 v21, a5\n\t"
"v_accvgpr_read_b32 v22, a6\n\t"
"v_accvgpr_read_b32 v23, a7\n\t"
"v_accvgpr_read_b32 v24, a8\n\t"
"v_accvgpr_read_b32 v25, a9\n\t"
"v_accvgpr_read_b32 v26, a10\n\t"
"v_accvgpr_read_b32 v27, a11\n\t"
"v_accvgpr_read_b32 v28, a12\n\t"
"v_accvgpr_read_b32 v29, a13\n\t"
"v_accvgpr_read_b32 v30, a14\n\t"
"v_accvgpr_read_b32 v31, a15\n\t"
"s_waitcnt vmcnt(28)\n\t"
"v_pk_add_f32 v[16:17], v[16:17], v[128:129]\n\t"
"v_pk_add_f32 v[18:19], v[18:19], v[130:131]\n\t"
"v_pk_add_f32 v[20:21], v[20:21], v[132:133]\n\t"
"v_pk_add_f32 v[22:23], v[22:23], v[134:135]\n\t"
"v_pk_add_f32 v[24:25], v[24:25], v[136:137]\n\t"
"v_pk_add_f32 v[26:27], v[26:27], v[138:139]\n\t"
"v_pk_add_f32 v[28:29], v[28:29], v[140:141]\n\t"
"v_pk_add_f32 v[30:31], v[30:31], v[142:143]\n\t"
"s_getpc_b64 s[74:75]\n\t"
"s_branch L_silu_%=\n\t"
"v_accvgpr_read_b32 v16, a16\n\t"
"v_accvgpr_read_b32 v17, a17\n\t"
"v_accvgpr_read_b32 v18, a18\n\t"
"v_accvgpr_read_b32 v19, a19\n\t"
"v_accvgpr_read_b32 v20, a20\n\t"
"v_accvgpr_read_b32 v21, a21\n\t"
"v_accvgpr_read_b32 v22, a22\n\t"
"v_accvgpr_read_b32 v23, a23\n\t"
"v_accvgpr_read_b32 v24, a24\n\t"
"v_accvgpr_read_b32 v25, a25\n\t"
"v_accvgpr_read_b32 v26, a26\n\t"
"v_accvgpr_read_b32 v27, a27\n\t"
"v_accvgpr_read_b32 v28, a28\n\t"
"v_accvgpr_read_b32 v29, a29\n\t"
"v_accvgpr_read_b32 v30, a30\n\t"
"v_accvgpr_read_b32 v31, a31\n\t"
"s_waitcnt vmcnt(28)\n\t"
"v_pk_add_f32 v[16:17], v[16:17], v[144:145]\n\t"
"v_pk_add_f32 v[18:19], v[18:19], v[146:147]\n\t"
"v_pk_add_f32 v[20:21], v[20:21], v[148:149]\n\t"
"v_pk_add_f32 v[22:23], v[22:23], v[150:151]\n\t"
"v_pk_add_f32 v[24:25], v[24:25], v[152:153]\n\t"
"v_pk_add_f32 v[26:27], v[26:27], v[154:155]\n\t"
"v_pk_add_f32 v[28:29], v[28:29], v[156:157]\n\t"
"v_pk_add_f32 v[30:31], v[30:31], v[158:159]\n\t"
"s_getpc_b64 s[74:75]\n\t"
"s_branch L_silu_%=\n\t"
"v_accvgpr_read_b32 v16, a32\n\t"
"v_accvgpr_read_b32 v17, a33\n\t"
"v_accvgpr_read_b32 v18, a34\n\t"
"v_accvgpr_read_b32 v19, a35\n\t"
"v_accvgpr_read_b32 v20, a36\n\t"
"v_accvgpr_read_b32 v21, a37\n\t"
"v_accvgpr_read_b32 v22, a38\n\t"
"v_accvgpr_read_b32 v23, a39\n\t"
"v_accvgpr_read_b32 v24, a40\n\t"
"v_accvgpr_read_b32 v25, a41\n\t"
"v_accvgpr_read_b32 v26, a42\n\t"
"v_accvgpr_read_b32 v27, a43\n\t"
"v_accvgpr_read_b32 v28, a44\n\t"
"v_accvgpr_read_b32 v29, a45\n\t"
"v_accvgpr_read_b32 v30, a46\n\t"
"v_accvgpr_read_b32 v31, a47\n\t"
"s_waitcnt vmcnt(28)\n\t"
"v_pk_add_f32 v[16:17], v[16:17], v[160:161]\n\t"
"v_pk_add_f32 v[18:19], v[18:19], v[162:163]\n\t"
"v_pk_add_f32 v[20:21], v[20:21], v[164:165]\n\t"
"v_pk_add_f32 v[22:23], v[22:23], v[166:167]\n\t"
"v_pk_add_f32 v[24:25], v[24:25], v[168:169]\n\t"
"v_pk_add_f32 v[26:27], v[26:27], v[170:171]\n\t"
"v_pk_add_f32 v[28:29], v[28:29], v[172:173]\n\t"
"v_pk_add_f32 v[30:31], v[30:31], v[174:175]\n\t"
"s_getpc_b64 s[74:75]\n\t"
"s_branch L_silu_%=\n\t"
"v_accvgpr_read_b32 v16, a48\n\t"
"v_accvgpr_read_b32 v17, a49\n\t"
"v_accvgpr_read_b32 v18, a50\n\t"
"v_accvgpr_read_b32 v19, a51\n\t"
"v_accvgpr_read_b32 v20, a52\n\t"
"v_accvgpr_read_b32 v21, a53\n\t"
"v_accvgpr_read_b32 v22, a54\n\t"
"v_accvgpr_read_b32 v23, a55\n\t"
"v_accvgpr_read_b32 v24, a56\n\t"
"v_accvgpr_read_b32 v25, a57\n\t"
"v_accvgpr_read_b32 v26, a58\n\t"
"v_accvgpr_read_b32 v27, a59\n\t"
"v_accvgpr_read_b32 v28, a60\n\t"
"v_accvgpr_read_b32 v29, a61\n\t"
"v_accvgpr_read_b32 v30, a62\n\t"
"v_accvgpr_read_b32 v31, a63\n\t"
"s_waitcnt vmcnt(28)\n\t"
"v_pk_add_f32 v[16:17], v[16:17], v[176:177]\n\t"
"v_pk_add_f32 v[18:19], v[18:19], v[178:179]\n\t"
"v_pk_add_f32 v[20:21], v[20:21], v[180:181]\n\t"
"v_pk_add_f32 v[22:23], v[22:23], v[182:183]\n\t"
"v_pk_add_f32 v[24:25], v[24:25], v[184:185]\n\t"
"v_pk_add_f32 v[26:27], v[26:27], v[186:187]\n\t"
"v_pk_add_f32 v[28:29], v[28:29], v[188:189]\n\t"
"v_pk_add_f32 v[30:31], v[30:31], v[190:191]\n\t"
"s_getpc_b64 s[74:75]\n\t"
"s_branch L_silu_%=\n\t"
"v_accvgpr_read_b32 v16, a64\n\t"
"v_accvgpr_read_b32 v17, a65\n\t"
"v_accvgpr_read_b32 v18, a66\n\t"
"v_accvgpr_read_b32 v19, a67\n\t"
"v_accvgpr_read_b32 v20, a68\n\t"
"v_accvgpr_read_b32 v21, a69\n\t"
"v_accvgpr_read_b32 v22, a70\n\t"
"v_accvgpr_read_b32 v23, a71\n\t"
"v_accvgpr_read_b32 v24, a72\n\t"
"v_accvgpr_read_b32 v25, a73\n\t"
"v_accvgpr_read_b32 v26, a74\n\t"
"v_accvgpr_read_b32 v27, a75\n\t"
"v_accvgpr_read_b32 v28, a76\n\t"
"v_accvgpr_read_b32 v29, a77\n\t"
"v_accvgpr_read_b32 v30, a78\n\t"
"v_accvgpr_read_b32 v31, a79\n\t"
"s_waitcnt vmcnt(28)\n\t"
"v_pk_add_f32 v[16:17], v[16:17], v[192:193]\n\t"
"v_pk_add_f32 v[18:19], v[18:19], v[194:195]\n\t"
"v_pk_add_f32 v[20:21], v[20:21], v[196:197]\n\t"
"v_pk_add_f32 v[22:23], v[22:23], v[198:199]\n\t"
"v_pk_add_f32 v[24:25], v[24:25], v[200:201]\n\t"
"v_pk_add_f32 v[26:27], v[26:27], v[202:203]\n\t"
"v_pk_add_f32 v[28:29], v[28:29], v[204:205]\n\t"
"v_pk_add_f32 v[30:31], v[30:31], v[206:207]\n\t"
"s_getpc_b64 s[74:75]\n\t"
"s_branch L_silu_%=\n\t"
"v_accvgpr_read_b32 v16, a80\n\t"
"v_accvgpr_read_b32 v17, a81\n\t"
"v_accvgpr_read_b32 v18, a82\n\t"
"v_accvgpr_read_b32 v19, a83\n\t"
"v_accvgpr_read_b32 v20, a84\n\t"
"v_accvgpr_read_b32 v21, a85\n\t"
"v_accvgpr_read_b32 v22, a86\n\t"
"v_accvgpr_read_b32 v23, a87\n\t"
"v_accvgpr_read_b32 v24, a88\n\t"
"v_accvgpr_read_b32 v25, a89\n\t"
"v_accvgpr_read_b32 v26, a90\n\t"
"v_accvgpr_read_b32 v27, a91\n\t"
"v_accvgpr_read_b32 v28, a92\n\t"
"v_accvgpr_read_b32 v29, a93\n\t"
"v_accvgpr_read_b32 v30, a94\n\t"
"v_accvgpr_read_b32 v31, a95\n\t"
"s_waitcnt vmcnt(28)\n\t"
"v_pk_add_f32 v[16:17], v[16:17], v[208:209]\n\t"
"v_pk_add_f32 v[18:19], v[18:19], v[210:211]\n\t"
"v_pk_add_f32 v[20:21], v[20:21], v[212:213]\n\t"
"v_pk_add_f32 v[22:23], v[22:23], v[214:215]\n\t"
"v_pk_add_f32 v[24:25], v[24:25], v[216:217]\n\t"
"v_pk_add_f32 v[26:27], v[26:27], v[218:219]\n\t"
"v_pk_add_f32 v[28:29], v[28:29], v[220:221]\n\t"
"v_pk_add_f32 v[30:31], v[30:31], v[222:223]\n\t"
"s_getpc_b64 s[74:75]\n\t"
"s_branch L_silu_%=\n\t"
"v_accvgpr_read_b32 v16, a96\n\t"
"v_accvgpr_read_b32 v17, a97\n\t"
"v_accvgpr_read_b32 v18, a98\n\t"
"v_accvgpr_read_b32 v19, a99\n\t"
"v_accvgpr_read_b32 v20, a100\n\t"
"v_accvgpr_read_b32 v21, a101\n\t"
"v_accvgpr_read_b32 v22, a102\n\t"
"v_accvgpr_read_b32 v23, a103\n\t"
"v_accvgpr_read_b32 v24, a104\n\t"
"v_accvgpr_read_b32 v25, a105\n\t"
"v_accvgpr_read_b32 v26, a106\n\t"
"v_accvgpr_read_b32 v27, a107\n\t"
"v_accvgpr_read_b32 v28, a108\n\t"
"v_accvgpr_read_b32 v29, a109\n\t"
"v_accvgpr_read_b32 v30, a110\n\t"
"v_accvgpr_read_b32 v31, a111\n\t"
"s_waitcnt vmcnt(28)\n\t"
"v_pk_add_f32 v[16:17], v[16:17], v[224:225]\n\t"
"v_pk_add_f32 v[18:19], v[18:19], v[226:227]\n\t"
"v_pk_add_f32 v[20:21], v[20:21], v[228:229]\n\t"
"v_pk_add_f32 v[22:23], v[22:23], v[230:231]\n\t"
"v_pk_add_f32 v[24:25], v[24:25], v[232:233]\n\t"
"v_pk_add_f32 v[26:27], v[26:27], v[234:235]\n\t"
"v_pk_add_f32 v[28:29], v[28:29], v[236:237]\n\t"
"v_pk_add_f32 v[30:31], v[30:31], v[238:239]\n\t"
"s_getpc_b64 s[74:75]\n\t"
"s_branch L_silu_%=\n\t"
"v_accvgpr_read_b32 v16, a112\n\t"
"v_accvgpr_read_b32 v17, a113\n\t"
"v_accvgpr_read_b32 v18, a114\n\t"
"v_accvgpr_read_b32 v19, a115\n\t"
"v_accvgpr_read_b32 v20, a116\n\t"
"v_accvgpr_read_b32 v21, a117\n\t"
"v_accvgpr_read_b32 v22, a118\n\t"
"v_accvgpr_read_b32 v23, a119\n\t"
"v_accvgpr_read_b32 v24, a120\n\t"
"v_accvgpr_read_b32 v25, a121\n\t"
"v_accvgpr_read_b32 v26, a122\n\t"
"v_accvgpr_read_b32 v27, a123\n\t"
"v_accvgpr_read_b32 v28, a124\n\t"
"v_accvgpr_read_b32 v29, a125\n\t"
"v_accvgpr_read_b32 v30, a126\n\t"
"v_accvgpr_read_b32 v31, a127\n\t"
"s_waitcnt vmcnt(28)\n\t"
"v_pk_add_f32 v[16:17], v[16:17], v[240:241]\n\t"
"v_pk_add_f32 v[18:19], v[18:19], v[242:243]\n\t"
"v_pk_add_f32 v[20:21], v[20:21], v[244:245]\n\t"
"v_pk_add_f32 v[22:23], v[22:23], v[246:247]\n\t"
"v_pk_add_f32 v[24:25], v[24:25], v[248:249]\n\t"
"v_pk_add_f32 v[26:27], v[26:27], v[250:251]\n\t"
"v_pk_add_f32 v[28:29], v[28:29], v[252:253]\n\t"
"v_pk_add_f32 v[30:31], v[30:31], v[254:255]\n\t"
"s_getpc_b64 s[74:75]\n\t"
"s_branch L_silu_%=\n\t"
"s_branch L_fin_%=\n\t"
"L_silu_%=:\n\t"
"s_add_u32 s74, s74, 4\n\t"
"s_addc_u32 s75, s75, 0\n\t"
"v_pk_mul_f32 v[32:33], v[16:17], v[50:51]\n\t"
"v_pk_mul_f32 v[34:35], v[18:19], v[50:51]\n\t"
"v_pk_mul_f32 v[36:37], v[20:21], v[50:51]\n\t"
"v_pk_mul_f32 v[38:39], v[22:23], v[50:51]\n\t"
"v_pk_mul_f32 v[40:41], v[24:25], v[50:51]\n\t"
"v_pk_mul_f32 v[42:43], v[26:27], v[50:51]\n\t"
"v_pk_mul_f32 v[44:45], v[28:29], v[50:51]\n\t"
"v_pk_mul_f32 v[46:47], v[30:31], v[50:51]\n\t"
"v_med3_f32 v32, v32, v57, v56\n\t"
"v_med3_f32 v33, v33, v57, v56\n\t"
"v_med3_f32 v34, v34, v57, v56\n\t"
"v_med3_f32 v35, v35, v57, v56\n\t"
"v_med3_f32 v36, v36, v57, v56\n\t"
"v_med3_f32 v37, v37, v57, v56\n\t"
"v_med3_f32 v38, v38, v57, v56\n\t"
"v_med3_f32 v39, v39, v57, v56\n\t"
"v_med3_f32 v40, v40, v57, v56\n\t"
"v_med3_f32 v41, v41, v57, v56\n\t"
"v_med3_f32 v42, v42, v57, v56\n\t"
"v_med3_f32 v43, v43, v57, v56\n\t"
"v_med3_f32 v44, v44, v57, v56\n\t"
"v_med3_f32 v45, v45, v57, v56\n\t"
"v_med3_f32 v46, v46, v57, v56\n\t"
"v_med3_f32 v47, v47, v57, v56\n\t"
"v_exp_f32 v32, v32\n\t"
"v_exp_f32 v33, v33\n\t"
"v_exp_f32 v34, v34\n\t"
"v_exp_f32 v35, v35\n\t"
"v_exp_f32 v36, v36\n\t"
"v_exp_f32 v37, v37\n\t"
"v_exp_f32 v38, v38\n\t"
"v_exp_f32 v39, v39\n\t"
"v_exp_f32 v40, v40\n\t"
"v_exp_f32 v41, v41\n\t"
"v_exp_f32 v42, v42\n\t"
"v_exp_f32 v43, v43\n\t"
"v_exp_f32 v44, v44\n\t"
"v_exp_f32 v45, v45\n\t"
"v_exp_f32 v46, v46\n\t"
"v_exp_f32 v47, v47\n\t"
"v_pk_add_f32 v[32:33], v[32:33], v[52:53]\n\t"
"v_pk_add_f32 v[34:35], v[34:35], v[52:53]\n\t"
"v_pk_add_f32 v[36:37], v[36:37], v[52:53]\n\t"
"v_pk_add_f32 v[38:39], v[38:39], v[52:53]\n\t"
"v_pk_add_f32 v[40:41], v[40:41], v[52:53]\n\t"
"v_pk_add_f32 v[42:43], v[42:43], v[52:53]\n\t"
"v_pk_add_f32 v[44:45], v[44:45], v[52:53]\n\t"
"v_pk_add_f32 v[46:47], v[46:47], v[52:53]\n\t"
"v_rcp_f32 v32, v32\n\t"
"v_rcp_f32 v33, v33\n\t"
"v_rcp_f32 v34, v34\n\t"
"v_rcp_f32 v35, v35\n\t"
"v_rcp_f32 v36, v36\n\t"
"v_rcp_f32 v37, v37\n\t"
"v_rcp_f32 v38, v38\n\t"
"v_rcp_f32 v39, v39\n\t"
"v_rcp_f32 v40, v40\n\t"
"v_rcp_f32 v41, v41\n\t"
"v_rcp_f32 v42, v42\n\t"
"v_rcp_f32 v43, v43\n\t"
"v_rcp_f32 v44, v44\n\t"
"v_rcp_f32 v45, v45\n\t"
"v_rcp_f32 v46, v46\n\t"
"v_rcp_f32 v47, v47\n\t"
"v_mul_f32 v64, v16, v32\n\t"
"v_mul_f32 v68, v17, v33\n\t"
"v_mul_f32 v72, v18, v34\n\t"
"v_mul_f32 v76, v19, v35\n\t"
"v_mul_f32 v65, v20, v36\n\t"
"v_mul_f32 v69, v21, v37\n\t"
"v_mul_f32 v73, v22, v38\n\t"
"v_mul_f32 v77, v23, v39\n\t"
"v_mul_f32 v66, v24, v40\n\t"
"v_mul_f32 v70, v25, v41\n\t"
"v_mul_f32 v74, v26, v42\n\t"
"v_mul_f32 v78, v27, v43\n\t"
"v_mul_f32 v67, v28, v44\n\t"
"v_mul_f32 v71, v29, v45\n\t"
"v_mul_f32 v75, v30, v46\n\t"
"v_mul_f32 v79, v31, v47\n\t"
"buffer_store_dwordx4 v[64:67], %[vooff], %[srdO], s70 offen \n\t"
"buffer_store_dwordx4 v[68:71], %[vooff], %[srdO], s71 offen \n\t"
"buffer_store_dwordx4 v[72:75], %[vooff], %[srdO], s72 offen \n\t"
"buffer_store_dwordx4 v[76:79], %[vooff], %[srdO], s73 offen \n\t"
"s_add_u32 s70, s70, 524288\n\t"
"s_add_u32 s71, s71, 524288\n\t"
"s_add_u32 s72, s72, 524288\n\t"
"s_add_u32 s73, s73, 524288\n\t"
"s_setpc_b64 s[74:75]\n\t"
"L_fin_%=:\n\t"
"s_waitcnt vmcnt(0)\n\t"

        :  : [srdT] "s"(srdT), [srdS] "s"(srdS), [srdR] "s"(srdR), [srdF] "s"(srdF), [srdO] "s"(srdO), [sfown] "s"(sfown), [sfpar] "s"(sfpar),
           [vlane16] "v"(vl16), [vooff] "v"(vooff), [scall] "s"(call)
        : "memory", "scc", "exec", VCLOB8, ACLOB, SCLOB);
  }
  if (threadIdx.x == 0 && __hip_atomic_load((unsigned*)(myws + WS_TMO), __ATOMIC_RELAXED, __HIP_MEMORY_SCOPE_AGENT) == call)
    out[(size_t)(mt * 256 + h * 128) * NG + pn * 256] = __builtin_nanf("");
}

extern "C" void kernel_launch(int rank, int nranks, ncclComm_t comm, hipStream_t stream,
                              void* const* d_in, const int* in_sizes, int n_in,
                              void* d_out, int out_size, const int* in_dtypes, int out_dtype,
                              void* const* d_out_all, void* const* d_ws_all, size_t ws_bytes) {
  if (nranks != NR || (unsigned)rank >= (unsigned)NR || ws_bytes < WS_NEED || n_in < 2 ||
      in_sizes[0] != MG * KL || in_sizes[1] != KG * NG || out_size != ML * NG) {
    fprintf(stderr, "unexpected problem shape\n");
    return;
  }
  Ptrs P;
  for (int p = 0; p < NR; ++p) P.ws[p] = (char*)d_ws_all[p];
  fused<<<256, 256, 0, stream>>>(rank, (const float*)d_in[0], (const float*)d_in[1], (float*)d_out, P.ws[rank], P);
  HIP_CHECK(hipGetLastError());
}
```
